```python
import numpy as np

kernel_name = "gpu_gemm_a2a_m4096_k4096_n4096_f32_gelu"


ABSMAX_RTOL = 0.001

COMPUTE_DTYPE = "float32"
EPILOGUE = "gelu"
IN_QUANT = None
OUT_QUANT = None
IN_DTYPES = {"x": "float32", "w_mat": "float32"}
WORLD_SIZE = 8

def setup_inputs():

    x = np.random.randn(4096, 4096).astype(np.float32)
    w_mat = (np.random.randn(4096, 4096) / np.sqrt(4096)).astype(np.float32)
    return {"x": x, "w_mat": w_mat}

def _epilogue(y):
    c = 0.7978845608028654
    return (0.5 * y * (1.0 + np.tanh(c * (y + 0.044715 * y ** 3)))).astype(np.float32)

def reference(x, w_mat):
    y = (x @ w_mat).astype(np.float32)
    return _epilogue(y)
```

```cpp
#include <hip/hip_runtime.h>
#include <rccl/rccl.h>
#include <cstdio>
#include <cstdint>

typedef _Float16 half8 __attribute__((ext_vector_type(8)));
typedef _Float16 half4 __attribute__((ext_vector_type(4)));
typedef float f32x16 __attribute__((ext_vector_type(16)));
typedef float f32x4 __attribute__((ext_vector_type(4)));

constexpr int NR = 8;
constexpr int MLOC = 512;
constexpr int KDIM = 4096;
constexpr int NDIM = 4096;
constexpr int NLOC = 512;
constexpr int BM = 128, BN = 64, BK = 128;
constexpr int LDH = BK + 8;
constexpr int NTHREADS = 256;
constexpr int NSTAGE = KDIM / BK;
constexpr int TILES_PER_PEER = (MLOC / BM) * (NLOC / BN);
constexpr unsigned long long DEADLINE_TICKS = 40000000ull;

__device__ __forceinline__ unsigned* ws_word(void* ws, int idx) { return (unsigned*)((char*)ws + 128 * idx); }

__device__ __forceinline__ float gelu_f(float y) {
    const float c = 0.7978845608028654f;
    float t = tanhf(c * (y + 0.044715f * y * y * y));
    return 0.5f * y * (1.0f + t);
}

__global__ __launch_bounds__(NTHREADS, 1)
void gemm_a2a_kernel(const float* __restrict__ x, const float* __restrict__ w,
                     void* const* __restrict__ out_all, void* const* __restrict__ ws_all, int rank) {
    __shared__ __attribute__((aligned(16))) _Float16 smem[2 * (BM + BN) * LDH];
    _Float16* As = smem;
    _Float16* Bs = smem + 2 * BM * LDH;

    const int tid = threadIdx.x;
    const int lane = tid & 63, wid = tid >> 6;
    const int b = blockIdx.x;
    const int peer = ((b & 7) + rank) & 7;
    const int idx = b >> 3;
    const int rt = idx & 3, ct = idx >> 2;
    const int row0 = rt * BM;
    const int ncol0 = peer * NLOC + ct * BN;

    const int a_k4 = tid & 31, a_r = tid >> 5;
    const float* aptr = x + (size_t)(row0 + a_r) * KDIM + a_k4 * 4;
    const int b_nq = tid & 15, b_kg = tid >> 4;
    const float* bptr = w + (size_t)(b_kg * 8) * NDIM + ncol0 + b_nq * 4;

    f32x4 ra[16], rb[8];
#pragma unroll
    for (int j = 0; j < 16; ++j) ra[j] = *(const f32x4*)(aptr + (size_t)(8 * j) * KDIM);
#pragma unroll
    for (int j = 0; j < 8; ++j) rb[j] = *(const f32x4*)(bptr + (size_t)j * NDIM);

    const int wm = wid >> 1, wn = wid & 1;
    f32x16 acc[2];
#pragma unroll
    for (int i = 0; i < 16; ++i) { acc[0][i] = 0.f; acc[1][i] = 0.f; }

    const int fr = lane & 31, fh = lane >> 5;

    for (int it = 0; it < NSTAGE; ++it) {
        const int buf = it & 1;
        _Float16* Ab = As + buf * BM * LDH;
        _Float16* Bb = Bs + buf * BN * LDH;
        const bool more = (it + 1 < NSTAGE);
        const int knext = (it + 1) * BK;
#pragma unroll
        for (int j = 0; j < 16; ++j) {
            f32x4 v = ra[j];
            half4 h; h[0] = (_Float16)v[0]; h[1] = (_Float16)v[1]; h[2] = (_Float16)v[2]; h[3] = (_Float16)v[3];
            *(half4*)(Ab + (a_r + 8 * j) * LDH + a_k4 * 4) = h;
            if (more) ra[j] = *(const f32x4*)(aptr + (size_t)(8 * j) * KDIM + knext);
        }
        {
            half8 h[4];
#pragma unroll
            for (int j = 0; j < 8; ++j) {
#pragma unroll
                for (int i = 0; i < 4; ++i) h[i][j] = (_Float16)rb[j][i];
            }
#pragma unroll
            for (int i = 0; i < 4; ++i) *(half8*)(Bb + (b_nq * 4 + i) * LDH + b_kg * 8) = h[i];
            if (more) {
#pragma unroll
                for (int j = 0; j < 8; ++j) rb[j] = *(const f32x4*)(bptr + (size_t)(knext + j) * NDIM);
            }
        }
        __syncthreads();
        const _Float16* a0p = Ab + (wm * 64 + fr) * LDH + fh * 8;
        const _Float16* a1p = a0p + 32 * LDH;
        const _Float16* b0p = Bb + (wn * 32 + fr) * LDH + fh * 8;
#pragma unroll
        for (int kk = 0; kk < BK / 16; ++kk) {
            half8 a0 = *(const half8*)(a0p + kk * 16);
            half8 a1 = *(const half8*)(a1p + kk * 16);
            half8 bb = *(const half8*)(b0p + kk * 16);
            acc[0] = __builtin_amdgcn_mfma_f32_32x32x16_f16(a0, bb, acc[0], 0, 0, 0);
            acc[1] = __builtin_amdgcn_mfma_f32_32x32x16_f16(a1, bb, acc[1], 0, 0, 0);
        }
    }
    __syncthreads();

    float* Cs = (float*)smem;
    constexpr int LDC = BN + 4;
#pragma unroll
    for (int m = 0; m < 2; ++m) {
#pragma unroll
        for (int r = 0; r < 16; ++r) {
            int row = wm * 64 + m * 32 + (r & 3) + 8 * (r >> 2) + 4 * fh;
            Cs[row * LDC + wn * 32 + fr] = gelu_f(acc[m][r]);
        }
    }
    __syncthreads();
    float* dst = (float*)out_all[peer] + (size_t)(rank * MLOC + row0) * NLOC + ct * BN;
    {
        const int c4 = tid & 15, r0 = tid >> 4;
#pragma unroll
        for (int j = 0; j < 8; ++j) {
            int row = r0 + 16 * j;
            f32x4 v = *(const f32x4*)(Cs + row * LDC + c4 * 4);
            *(f32x4*)(dst + (size_t)row * NLOC + c4 * 4) = v;
        }
    }
    asm volatile("s_waitcnt vmcnt(0)" ::: "memory");
    __syncthreads();
    if (tid == 0) {
        if (peer != rank) {
            __builtin_amdgcn_fence(__ATOMIC_RELEASE, "");
            asm volatile("s_waitcnt vmcnt(0)" ::: "memory");
            __hip_atomic_fetch_add(ws_word(ws_all[peer], 2 + rank), 1u, __ATOMIC_RELAXED, __HIP_MEMORY_SCOPE_SYSTEM);
        }
        if (b == 0) {
            void* myws = ws_all[rank];
            unsigned call = __hip_atomic_fetch_add(ws_word(myws, 0), 1u, __ATOMIC_RELAXED, __HIP_MEMORY_SCOPE_AGENT) + 1u;
            unsigned need = call * (unsigned)TILES_PER_PEER;
            unsigned long long t0 = __builtin_amdgcn_s_memrealtime();
            bool timed_out = false;
            for (int p = 0; p < NR; ++p) {
                if (p == rank) continue;
                while (__hip_atomic_load(ws_word(myws, 2 + p), __ATOMIC_RELAXED, __HIP_MEMORY_SCOPE_SYSTEM) < need) {
                    if (__builtin_amdgcn_s_memrealtime() - t0 > DEADLINE_TICKS) { timed_out = true; break; }
                    __builtin_amdgcn_s_sleep(2);
                }
            }
            __builtin_amdgcn_fence(__ATOMIC_ACQUIRE, "");
            if (timed_out) {
                __hip_atomic_store(ws_word(myws, 1), call, __ATOMIC_RELAXED, __HIP_MEMORY_SCOPE_AGENT);
                ((float*)out_all[rank])[(size_t)(rank * MLOC) * NLOC] = __builtin_nanf("");
            }
        }
    }
}

extern "C" void kernel_launch(int rank, int nranks, ncclComm_t comm,
                              hipStream_t stream,
                              void* const* d_in, const int* in_sizes, int n_in,
                              void* d_out, int out_size,
                              const int* in_dtypes, int out_dtype,
                              void* const* d_out_all,
                              void* const* d_ws_all, size_t ws_bytes) {
    const float* x = (const float*)d_in[0];
    const float* w = (const float*)d_in[1];
    hipLaunchKernelGGL(gemm_a2a_kernel, dim3(256), dim3(NTHREADS), 0, stream,
                       x, w, d_out_all, d_ws_all, rank);
}
```

```cpp
#include <hip/hip_runtime.h>
#include <rccl/rccl.h>
#include <cstdio>
#include <cstdint>

typedef _Float16 half8 __attribute__((ext_vector_type(8)));
typedef _Float16 half4 __attribute__((ext_vector_type(4)));
typedef float f32x16 __attribute__((ext_vector_type(16)));
typedef float f32x4 __attribute__((ext_vector_type(4)));
typedef short s16x4 __attribute__((ext_vector_type(4)));
typedef short s16x8 __attribute__((ext_vector_type(8)));

constexpr int NR = 8;
constexpr int MLOC = 512;
constexpr int KDIM = 4096;
constexpr int NDIM = 4096;
constexpr int NLOC = 512;
constexpr int BM = 128, BN = 64, BK = 128;
constexpr int LDH = BK + 8;
constexpr int NTHREADS = 512;
constexpr int NSTAGE = KDIM / BK;
constexpr int TILES_PER_PEER = (MLOC / BM) * (NLOC / BN);
constexpr unsigned long long DEADLINE_TICKS = 40000000ull;

__device__ __forceinline__ unsigned* ws_word(void* ws, int idx) { return (unsigned*)((char*)ws + 128 * idx); }

__device__ __forceinline__ float gelu_f(float y) {
    const float c = 0.7978845608028654f;
    float t = tanhf(c * (y + 0.044715f * y * y * y));
    return 0.5f * y * (1.0f + t);
}

__global__ __launch_bounds__(NTHREADS, 2)
void gemm_a2a_kernel(const float* __restrict__ x, const float* __restrict__ w,
                     void* const* __restrict__ out_all, void* const* __restrict__ ws_all, int rank) {
    __shared__ __attribute__((aligned(16))) _Float16 smem[2 * BM * LDH + 2 * BK * BN];
    _Float16* As = smem;
    _Float16* Bs = smem + 2 * BM * LDH;

    const int tid = threadIdx.x;
    const int lane = tid & 63, wid = tid >> 6;
    const int b = blockIdx.x;
    const int peer = ((b & 7) + rank) & 7;
    const int idx = b >> 3;
    const int rt = idx & 3, ct = idx >> 2;
    const int row0 = rt * BM;
    const int ncol0 = peer * NLOC + ct * BN;

    const int a_k4 = tid & 31, a_r = tid >> 5;
    const int b_nq = tid & 15, b_kg = tid >> 4;
    const unsigned a_voff = (unsigned)(a_r * KDIM + a_k4 * 4) * 4u;
    const unsigned b_voff = (unsigned)(b_kg * 4 * NDIM + b_nq * 4) * 4u;
    const char* a_base = (const char*)(x + (size_t)row0 * KDIM);
    const char* b_base = (const char*)(w + ncol0);

#define X_NOLDSW 0
#define X_NOLDSR 0
#define X_NOMFMA 0
#define X_NOBAR 0
#define GLOAD(DST, VOFF, SBASE) asm volatile("global_load_dwordx4 %0, %1, %2" : "=v"(DST) : "v"(VOFF), "s"(SBASE))
#define LDA(RA, J, K) GLOAD(RA[J], a_voff, a_base + (size_t)(16 * (J)) * KDIM * 4 + (size_t)(K) * 4)
#define LDB(RB, J, K) GLOAD(RB[J], b_voff, b_base + (size_t)((K) + (J)) * NDIM * 4)
#define WAIT4(N, R, J) asm volatile("s_waitcnt vmcnt(" #N ")" : "+v"(R[J]), "+v"(R[J+1]), "+v"(R[J+2]), "+v"(R[J+3]))

    f32x4 ra0[8], rb0[4], ra1[8], rb1[4];
#pragma unroll
    for (int j = 0; j < 8; ++j) LDA(ra0, j, 0);
#pragma unroll
    for (int j = 0; j < 4; ++j) LDB(rb0, j, 0);
#pragma unroll
    for (int j = 0; j < 8; ++j) LDA(ra1, j, BK);
#pragma unroll
    for (int j = 0; j < 4; ++j) LDB(rb1, j, BK);

    const int wm = wid >> 1, wn = wid & 1;
    f32x16 acc[2];
#pragma unroll
    for (int i = 0; i < 16; ++i) { acc[0][i] = 0.f; acc[1][i] = 0.f; }

    const int fr = lane & 31, fh = lane >> 5;
    const int bblk = (lane >> 4) & 1, bq = (lane & 15) >> 2, bp = lane & 3;

#define STAGE_BODY(RA, RB, BUF, KNEXT, MODE) do { \
        _Float16* Ab = As + (BUF) * BM * LDH; \
        _Float16* Bb = Bs + (BUF) * BK * BN; \
        _Pragma("unroll") for (int g = 0; g < 2; ++g) { \
            if (MODE == 0) WAIT4(20, RA, 4 * g); else if (MODE == 1) WAIT4(12, RA, 4 * g); else WAIT4(0, RA, 4 * g); \
            _Pragma("unroll") for (int jj = 0; jj < 4; ++jj) { \
                const int j = 4 * g + jj; \
                f32x4 v = RA[j]; \
                half4 h; h[0] = (_Float16)v[0]; h[1] = (_Float16)v[1]; h[2] = (_Float16)v[2]; h[3] = (_Float16)v[3]; \
                if (!X_NOLDSW) *(half4*)(Ab + (a_r + 16 * j) * LDH + a_k4 * 4) = h; else asm volatile("" :: "v"(h)); \
                if (MODE == 0) LDA(RA, j, KNEXT); \
            } \
        } \
        { \
            if (MODE == 0) WAIT4(20, RB, 0); else if (MODE == 1) WAIT4(12, RB, 0); else WAIT4(0, RB, 0); \
            _Pragma("unroll") for (int j = 0; j < 4; ++j) { \
                f32x4 v = RB[j]; \
                half4 h; h[0] = (_Float16)v[0]; h[1] = (_Float16)v[1]; h[2] = (_Float16)v[2]; h[3] = (_Float16)v[3]; \
                if (!X_NOLDSW) *(half4*)(Bb + (b_kg * 4 + j) * BN + ((b_nq * 4) ^ (32 * ((j >> 1) & 1)))) = h; else asm volatile("" :: "v"(h)); \
            } \
            if (MODE == 0) { _Pragma("unroll") for (int j = 0; j < 4; ++j) LDB(RB, j, KNEXT); } \
        } \
        if (!X_NOBAR) __syncthreads(); \
        const _Float16* a0p = Ab + (wm * 32 + fr) * LDH + fh * 8; \
        const _Float16* b0p = Bb + (8 * fh + bq) * BN + ((32 * wn + 16 * bblk + 4 * bp) ^ (32 * ((bq >> 1) & 1))); \
        half8 fa[8], fb[8]; \
        _Pragma("unroll") for (int kk = 0; kk < 8; ++kk) { \
            if (!X_NOLDSR) { fa[kk] = *(const half8*)(a0p + kk * 16); \
            { s16x4 lo = __builtin_amdgcn_ds_read_tr16_b64_v4i16((__attribute__((address_space(3))) s16x4*)(b0p + kk * 16 * BN)); \
              s16x4 hi = __builtin_amdgcn_ds_read_tr16_b64_v4i16((__attribute__((address_space(3))) s16x4*)(b0p + (kk * 16 + 4) * BN)); \
              s16x8 cc = __builtin_shufflevector(lo, hi, 0, 1, 2, 3, 4, 5, 6, 7); fb[kk] = __builtin_bit_cast(half8, cc); } } else { asm volatile("" : "=v"(fa[kk])); asm volatile("" : "=v"(fb[kk])); } \
        } \
        _Pragma("unroll") for (int kk = 0; kk < 8; ++kk) \
            { if (!X_NOMFMA) acc[kk & 1] = __builtin_amdgcn_mfma_f32_32x32x16_f16(fa[kk], fb[kk], acc[kk & 1], 0, 0, 0); else asm volatile("" :: "v"(fa[kk]), "v"(fb[kk])); } \
    } while (0)

    for (int it = 0; it < NSTAGE - 2; it += 2) {
        STAGE_BODY(ra0, rb0, 0, (it + 2) * BK, 0);
        STAGE_BODY(ra1, rb1, 1, (it + 3) * BK, 0);
    }
    STAGE_BODY(ra0, rb0, 0, 0, 1);
    STAGE_BODY(ra1, rb1, 1, 0, 2);
    __syncthreads();

    float* Cs = (float*)smem;
    constexpr int LDC = BN + 4;
#pragma unroll
    for (int r = 0; r < 16; ++r) {
        int row = wm * 32 + (r & 3) + 8 * (r >> 2) + 4 * fh;
        Cs[row * LDC + wn * 32 + fr] = gelu_f(acc[0][r] + acc[1][r]);
    }
    __syncthreads();
    float* dst = (float*)out_all[peer] + (size_t)(rank * MLOC + row0) * NLOC + ct * BN;
    {
        const int c4 = tid & 15, r0 = tid >> 4;
#pragma unroll
        for (int j = 0; j < 4; ++j) {
            int row = r0 + 32 * j;
            f32x4 v = *(const f32x4*)(Cs + row * LDC + c4 * 4);
            *(f32x4*)(dst + (size_t)row * NLOC + c4 * 4) = v;
        }
    }
    asm volatile("s_waitcnt vmcnt(0)" ::: "memory");
    __syncthreads();
    if (tid == 0) {
        if (peer != rank) {
            __builtin_amdgcn_fence(__ATOMIC_RELEASE, "");
            asm volatile("s_waitcnt vmcnt(0)" ::: "memory");
            __hip_atomic_fetch_add(ws_word(ws_all[peer], 2 + rank), 1u, __ATOMIC_RELAXED, __HIP_MEMORY_SCOPE_SYSTEM);
        }
        if (b == 0) {
            void* myws = ws_all[rank];
            unsigned call = __hip_atomic_fetch_add(ws_word(myws, 0), 1u, __ATOMIC_RELAXED, __HIP_MEMORY_SCOPE_AGENT) + 1u;
            unsigned need = call * (unsigned)TILES_PER_PEER;
            unsigned long long t0 = __builtin_amdgcn_s_memrealtime();
            bool timed_out = false;
            for (int p = 0; p < NR; ++p) {
                if (p == rank) continue;
                while (__hip_atomic_load(ws_word(myws, 2 + p), __ATOMIC_RELAXED, __HIP_MEMORY_SCOPE_SYSTEM) < need) {
                    if (__builtin_amdgcn_s_memrealtime() - t0 > DEADLINE_TICKS) { timed_out = true; break; }
                    __builtin_amdgcn_s_sleep(2);
                }
            }
            __builtin_amdgcn_fence(__ATOMIC_ACQUIRE, "");
            if (timed_out) {
                __hip_atomic_store(ws_word(myws, 1), call, __ATOMIC_RELAXED, __HIP_MEMORY_SCOPE_AGENT);
                ((float*)out_all[rank])[(size_t)(rank * MLOC) * NLOC] = __builtin_nanf("");
            }
        }
    }
}

extern "C" void kernel_launch(int rank, int nranks, ncclComm_t comm,
                              hipStream_t stream,
                              void* const* d_in, const int* in_sizes, int n_in,
                              void* d_out, int out_size,
                              const int* in_dtypes, int out_dtype,
                              void* const* d_out_all,
                              void* const* d_ws_all, size_t ws_bytes) {
    const float* x = (const float*)d_in[0];
    const float* w = (const float*)d_in[1];
    hipLaunchKernelGGL(gemm_a2a_kernel, dim3(256), dim3(NTHREADS), 0, stream,
                       x, w, d_out_all, d_ws_all, rank);
}
```

```cpp
#include <hip/hip_runtime.h>
#include <rccl/rccl.h>
#include <cstdio>
#include <cstdint>

typedef _Float16 half8 __attribute__((ext_vector_type(8)));
typedef _Float16 half4 __attribute__((ext_vector_type(4)));
typedef float f32x16 __attribute__((ext_vector_type(16)));
typedef float f32x4 __attribute__((ext_vector_type(4)));
typedef unsigned u32x4 __attribute__((ext_vector_type(4)));
typedef short s16x4 __attribute__((ext_vector_type(4)));
typedef short s16x8 __attribute__((ext_vector_type(8)));

constexpr int NR = 8;
constexpr int MLOC = 512;
constexpr int KDIM = 4096;
constexpr int NDIM = 4096;
constexpr int NLOC = 512;
constexpr int BN = 64;
constexpr int KS = 1024;
constexpr int NCW = 8;
constexpr int NPW = 4;
constexpr int NBLOCK = (NCW + NPW) * 64;
constexpr int BLOCKS_PER_PEER = 32;
constexpr unsigned long long DEADLINE_TICKS = 40000000ull;
constexpr size_t FLAG_OFF = 65536;
#define RBPAD 0
constexpr size_t RBS = 262144 + RBPAD;
constexpr size_t X16_OFF = 262144;
constexpr size_t PART_OFF = X16_OFF + (5u << 20);

__device__ __forceinline__ unsigned* ws_word(void* ws, int idx) { return (unsigned*)((char*)ws + 128 * idx); }

__device__ __forceinline__ float gelu_f(float y) {
    const float c = 0.7978845608028654f;
    float u = c * (y + 0.044715f * y * y * y);
    return y / (1.0f + __expf(-2.0f * u));
}

#define APOL ""
#define BPOL " nt"
#define GLOAD(DST, VOFF, SBASE) asm volatile("global_load_dwordx4 %0, %1, %2" APOL : "=v"(DST) : "v"(VOFF), "s"(SBASE))
#define GLOADNT(DST, VOFF, SBASE) asm volatile("global_load_dwordx4 %0, %1, %2" BPOL : "=v"(DST) : "v"(VOFF), "s"(SBASE))

__global__ __launch_bounds__(NBLOCK, 3)
void gemm_a2a_kernel(const float* __restrict__ x, const float* __restrict__ w,
                     void* const* __restrict__ out_all, void* const* __restrict__ ws_all, int rank, int phase) {
    __shared__ __attribute__((aligned(16))) _Float16 Bs[KS * BN];
    __shared__ __attribute__((aligned(16))) float Ts[NCW][8 * 68];
    __shared__ __attribute__((aligned(16))) unsigned rdy[4];

    const int tid = threadIdx.x;
    const int lane = tid & 63, wid = __builtin_amdgcn_readfirstlane(tid >> 6);
    char* myws = (char*)ws_all[rank];
    _Float16* x16 = (_Float16*)(myws + X16_OFF);
    if (phase == 0) {
        if (blockIdx.x == 0 && tid == 0) __hip_atomic_fetch_add(ws_word(myws, 0), 1u, __ATOMIC_RELAXED, __HIP_MEMORY_SCOPE_AGENT);
        const int W = blockIdx.x * 8 + wid;
        const int rb = W >> 7, v = W & 127, g = v >> 1;
        const int c8 = 8 * g + (lane & 7);
#pragma unroll
        for (int i = 0; i < 2; ++i) {
            const int r = 16 * (v & 1) + 8 * i + (lane >> 3);
            const float* src = x + (size_t)(32 * rb + r) * KDIM + c8 * 8;
            f32x4 v0 = __builtin_nontemporal_load((const f32x4*)src), v1 = __builtin_nontemporal_load((const f32x4*)(src + 4));
            half8 h;
#pragma unroll
            for (int e = 0; e < 4; ++e) { h[e] = (_Float16)v0[e]; h[4 + e] = (_Float16)v1[e]; }
            *(half8*)((char*)x16 + rb * RBS + ((size_t)(c8 >> 1) * 64 + (c8 & 1) * 32 + r) * 16) = h;
        }
        return;
    }
    const int b = blockIdx.x;
    const int q = b & 3;
    const int grp = ((b >> 2) & 1) * 32 + (b >> 3);
    const int peer = ((grp >> 3) + rank) & 7;
    const int ct = grp & 7;
    const int ncol0 = peer * NLOC + ct * BN;

    if (wid >= NCW) {
        const int p = wid - NCW;
        const int kr = lane >> 4, nq = lane & 15;
        const unsigned voff = (unsigned)((4 * p + kr) * NDIM + nq * 4) * 4u;
        const char* base = (const char*)(w + (size_t)q * KS * NDIM + ncol0);
        f32x4 rb[16];
#define PWR(X) X
#define PSTAMP(S)
#define PLD(SLOT, S) _Pragma("unroll") for (int i = 0; i < 4; ++i) GLOADNT(rb[(SLOT) * 4 + i], voff, base + (size_t)((S) * 64 + 16 * i) * NDIM * 4)
#define PWAIT(N, SLOT) asm volatile("s_waitcnt vmcnt(" #N ")" : "+v"(rb[(SLOT) * 4]), "+v"(rb[(SLOT) * 4 + 1]), "+v"(rb[(SLOT) * 4 + 2]), "+v"(rb[(SLOT) * 4 + 3]))
#define PST(SLOT, S) do { \
        _Pragma("unroll") for (int i = 0; i < 4; ++i) { \
            f32x4 v = rb[(SLOT) * 4 + i]; \
            half4 h; h[0] = (_Float16)v[0]; h[1] = (_Float16)v[1]; h[2] = (_Float16)v[2]; h[3] = (_Float16)v[3]; \
            PWR(*(half4*)(Bs + ((S) * 64 + 16 * i + 4 * p + kr) * BN + ((nq * 4) ^ (32 * ((kr >> 1) & 1)))) = h); \
        } \
        asm volatile("" ::: "memory"); \
        if (lane == 0) *(volatile unsigned*)&rdy[p] = (unsigned)(S) + 1u; \
        PSTAMP(S); \
        asm volatile("" ::: "memory"); } while (0)
#define PD 2
#pragma unroll
        for (int s = 0; s < PD; ++s) PLD(s, s);
        if (lane == 0) rdy[p] = 0;
        __syncthreads();
#pragma unroll
        for (int s = 0; s < 16; ++s) {
            const int left = (15 - s < PD - 1) ? 15 - s : PD - 1;
            if (left == 3) PWAIT(12, s % PD); else if (left == 2) PWAIT(8, s % PD); else if (left == 1) PWAIT(4, s % PD); else PWAIT(0, s % PD);
            PST(s % PD, s);
            if (s + PD < 16) PLD(s % PD, s + PD);
        }
        __syncthreads();
        return;
    }

    const unsigned a_voff = (unsigned)lane * 16u;
    const char* a_base = (const char*)x16 + (size_t)(2 * wid) * RBS + (size_t)(64 * q) * 1024;
    f32x4 ra[8][2];
#define ASTEP(S) ((size_t)(S) * 1024)
#define ALD(SLOT, STEP) do { GLOAD(ra[SLOT][0], a_voff, a_base + ASTEP(STEP)); \
                             GLOAD(ra[SLOT][1], a_voff, a_base + ASTEP(STEP) + RBS); } while (0)
#define AWAIT(N, SLOT) asm volatile("s_waitcnt vmcnt(" #N ")" : "+v"(ra[SLOT][0]), "+v"(ra[SLOT][1]))
#pragma unroll
    for (int i = 0; i < 8; ++i) ALD(i, i);
    unsigned call;
    asm volatile("global_load_dword %0, %1, off sc1" : "=v"(call) : "v"(myws));
    __syncthreads();

    f32x16 acc[4];
#pragma unroll
    for (int t = 0; t < 4; ++t)
#pragma unroll
        for (int i = 0; i < 16; ++i) acc[t][i] = 0.f;

    const int fh = lane >> 5;
    const int bblk = (lane >> 4) & 1, bq = (lane & 15) >> 2, bp = lane & 3;
    const _Float16* b0p = Bs + (8 * fh + bq) * BN + ((16 * bblk + 4 * bp) ^ (32 * ((bq >> 1) & 1)));
    const _Float16* b1p = Bs + (8 * fh + bq) * BN + ((32 + 16 * bblk + 4 * bp) ^ (32 * ((bq >> 1) & 1)));
    bool bad = false;
    unsigned seen = 0;
#define BPOLL() do { \
        u32x4 rv; \
        asm volatile("ds_read_b128 %0, %1\n\ts_waitcnt lgkmcnt(0)" : "=v"(rv) : "v"((unsigned)(uintptr_t)(__attribute__((address_space(3))) unsigned*)rdy) : "memory"); \
        seen = __builtin_amdgcn_readfirstlane(min(min(rv[0], rv[1]), min(rv[2], rv[3]))); } while (0)
#define BWAIT(S) do { \
        if (seen <= (unsigned)(S)) { \
            BPOLL(); \
            if (seen <= (unsigned)(S)) { \
                unsigned long long t0 = __builtin_amdgcn_s_memrealtime(); \
                for (unsigned n = 1; ; ++n) { \
                    __builtin_amdgcn_s_sleep(BSLEEP); \
                    BPOLL(); \
                    if (seen > (unsigned)(S)) break; \
                    if ((n & 255u) == 0u && __builtin_amdgcn_s_memrealtime() - t0 > DEADLINE_TICKS) { bad = true; seen = 64; break; } \
                } \
            } \
        } } while (0)
#define BSLEEP 2
#define XNOALD 0
#define TR(P) __builtin_amdgcn_ds_read_tr16_b64_v4i16((__attribute__((address_space(3))) s16x4*)(P))
#define STEP_BODY(SLOT, STEP, NW, RELOAD) do { \
        s16x4 l0 = TR(b0p + (STEP) * 16 * BN), h0 = TR(b0p + ((STEP) * 16 + 4) * BN); \
        s16x4 l1 = TR(b1p + (STEP) * 16 * BN), h1 = TR(b1p + ((STEP) * 16 + 4) * BN); \
        half8 fb0 = __builtin_bit_cast(half8, __builtin_shufflevector(l0, h0, 0, 1, 2, 3, 4, 5, 6, 7)); \
        half8 fb1 = __builtin_bit_cast(half8, __builtin_shufflevector(l1, h1, 0, 1, 2, 3, 4, 5, 6, 7)); \
        AWAIT(NW, SLOT); \
        half8 fa0 = __builtin_bit_cast(half8, ra[SLOT][0]), fa1 = __builtin_bit_cast(half8, ra[SLOT][1]); \
        acc[0] = __builtin_amdgcn_mfma_f32_32x32x16_f16(fa0, fb0, acc[0], 0, 0, 0); \
        acc[1] = __builtin_amdgcn_mfma_f32_32x32x16_f16(fa0, fb1, acc[1], 0, 0, 0); \
        acc[2] = __builtin_amdgcn_mfma_f32_32x32x16_f16(fa1, fb0, acc[2], 0, 0, 0); \
        acc[3] = __builtin_amdgcn_mfma_f32_32x32x16_f16(fa1, fb1, acc[3], 0, 0, 0); \
        if (RELOAD && !XNOALD) ALD(SLOT, (STEP) + 8); \
    } while (0)

    for (int st = 0; st < 56; st += 8) {
        BWAIT(st >> 2);
        STEP_BODY(0, st + 0, 14, 1); STEP_BODY(1, st + 1, 14, 1); STEP_BODY(2, st + 2, 14, 1); STEP_BODY(3, st + 3, 14, 1);
        BWAIT((st >> 2) + 1);
        STEP_BODY(4, st + 4, 14, 1); STEP_BODY(5, st + 5, 14, 1); STEP_BODY(6, st + 6, 14, 1); STEP_BODY(7, st + 7, 14, 1);
    }
    BWAIT(14);
    STEP_BODY(0, 56, 14, 0); STEP_BODY(1, 57, 12, 0); STEP_BODY(2, 58, 10, 0); STEP_BODY(3, 59, 8, 0);
    BWAIT(15);
    STEP_BODY(4, 60, 6, 0); STEP_BODY(5, 61, 4, 0); STEP_BODY(6, 62, 2, 0); STEP_BODY(7, 63, 0, 0);

    char* part = myws + PART_OFF;
    unsigned* flags = (unsigned*)(myws + FLAG_OFF);
    f32x4 own[4];
    {
        char* dstp = part + (size_t)((grp * 4 + q) * 8 + wid) * 12288 + lane * 16;
#pragma unroll
        for (int G = 0; G < 4; ++G) {
            if (G == q) {
#pragma unroll
                for (int t = 0; t < 4; ++t) { own[t][0] = acc[t][4 * G]; own[t][1] = acc[t][4 * G + 1]; own[t][2] = acc[t][4 * G + 2]; own[t][3] = acc[t][4 * G + 3]; }
            } else {
                char* d = dstp + ((G - q - 1) & 3) * 4096;
#pragma unroll
                for (int t = 0; t < 4; ++t) {
                    f32x4 v; v[0] = acc[t][4 * G]; v[1] = acc[t][4 * G + 1]; v[2] = acc[t][4 * G + 2]; v[3] = acc[t][4 * G + 3];
                    asm volatile("global_store_dwordx4 %0, %1, off sc0 sc1" :: "v"(d + t * 1024), "v"(v) : "memory");
                }
            }
        }
        asm volatile("s_waitcnt vmcnt(0)" : "+v"(call) :: "memory");
        if (lane == 0) __hip_atomic_store(flags + ((grp * 4 + q) * 8 + wid) * 16, call, __ATOMIC_RELAXED, __HIP_MEMORY_SCOPE_SYSTEM);
        const int sp = (q + 1 + (lane & 3)) & 3;
        unsigned* fp = flags + ((grp * 4 + sp) * 8 + wid) * 16;
        unsigned long long t0 = __builtin_amdgcn_s_memrealtime();
        while (true) {
            unsigned fv = call;
            if ((lane & 3) != 3 && lane < 4) fv = __hip_atomic_load(fp, __ATOMIC_RELAXED, __HIP_MEMORY_SCOPE_SYSTEM);
            if (__builtin_amdgcn_ballot_w64(fv != call) == 0ull) break;
            if (__builtin_amdgcn_s_memrealtime() - t0 > DEADLINE_TICKS) { bad = true; break; }
            __builtin_amdgcn_s_sleep(1);
        }
        asm volatile("" ::: "memory");
        f32x4 pv[12];
#pragma unroll
        for (int d = 0; d < 3; ++d) {
            const int p = (q + 1 + d) & 3;
            const char* s = part + (size_t)((grp * 4 + p) * 8 + wid) * 12288 + ((q - p - 1) & 3) * 4096 + lane * 16;
#pragma unroll
            for (int t = 0; t < 4; ++t)
                asm volatile("global_load_dwordx4 %0, %1, off sc0 sc1" : "=v"(pv[d * 4 + t]) : "v"(s + t * 1024) : "memory");
        }
        asm volatile("s_waitcnt vmcnt(0)" : "+v"(pv[0]), "+v"(pv[1]), "+v"(pv[2]), "+v"(pv[3]), "+v"(pv[4]), "+v"(pv[5]),
                     "+v"(pv[6]), "+v"(pv[7]), "+v"(pv[8]), "+v"(pv[9]), "+v"(pv[10]), "+v"(pv[11]));
#pragma unroll
        for (int t = 0; t < 4; ++t) own[t] = (own[t] + pv[t]) + (pv[4 + t] + pv[8 + t]);
    }

    float* dst = (float*)out_all[peer] + (size_t)(rank * MLOC) * NLOC + ct * BN;
    float* T = Ts[wid];
#pragma unroll
    for (int mi = 0; mi < 2; ++mi) {
#pragma unroll
        for (int ni = 0; ni < 2; ++ni)
#pragma unroll
            for (int i = 0; i < 4; ++i) {
                float g = gelu_f(own[mi * 2 + ni][i]);
                T[(4 * fh + i) * 68 + ni * 32 + (lane & 31)] = bad ? __builtin_nanf("") : g;
            }
        __builtin_amdgcn_wave_barrier();
#pragma unroll
        for (int j = 0; j < 2; ++j) {
            const int r = 4 * j + (lane >> 4), c4 = lane & 15;
            f32x4 v = *(const f32x4*)(T + r * 68 + c4 * 4);
            *(f32x4*)(dst + (size_t)(64 * wid + 32 * mi + 8 * q + r) * NLOC + c4 * 4) = v;
        }
        __builtin_amdgcn_wave_barrier();
    }
    asm volatile("s_waitcnt vmcnt(0)" ::: "memory");
    __syncthreads();
    if (tid == 0) {
        if (peer != rank) {
            __builtin_amdgcn_fence(__ATOMIC_RELEASE, "");
            asm volatile("s_waitcnt vmcnt(0)" ::: "memory");
            __hip_atomic_fetch_add(ws_word(ws_all[peer], 2 + rank), 1u, __ATOMIC_RELAXED, __HIP_MEMORY_SCOPE_SYSTEM);
        }
        if (b == 0) {
            unsigned need = call * (unsigned)BLOCKS_PER_PEER;
            unsigned long long t0 = __builtin_amdgcn_s_memrealtime();
            bool timed_out = false;
            for (int p = 0; p < NR; ++p) {
                if (p == rank) continue;
                while (__hip_atomic_load(ws_word(myws, 2 + p), __ATOMIC_RELAXED, __HIP_MEMORY_SCOPE_SYSTEM) < need) {
                    if (__builtin_amdgcn_s_memrealtime() - t0 > DEADLINE_TICKS) { timed_out = true; break; }
                    __builtin_amdgcn_s_sleep(2);
                }
            }
            __builtin_amdgcn_fence(__ATOMIC_ACQUIRE, "");
            if (timed_out) {
                __hip_atomic_store(ws_word(myws, 1), call, __ATOMIC_RELAXED, __HIP_MEMORY_SCOPE_AGENT);
                ((float*)out_all[rank])[(size_t)(rank * MLOC) * NLOC] = __builtin_nanf("");
            }
        }
    }
}

extern "C" void kernel_launch(int rank, int nranks, ncclComm_t comm,
                              hipStream_t stream,
                              void* const* d_in, const int* in_sizes, int n_in,
                              void* d_out, int out_size,
                              const int* in_dtypes, int out_dtype,
                              void* const* d_out_all,
                              void* const* d_ws_all, size_t ws_bytes) {
    const float* x = (const float*)d_in[0];
    const float* w = (const float*)d_in[1];
    { static int ncall = 0; if (ncall < 4) fprintf(stderr, "KDBG rank %d call %d\n", rank, ncall); ncall++; }
    hipLaunchKernelGGL(gemm_a2a_kernel, dim3(256), dim3(512), 0, stream,
                       x, w, d_out_all, d_ws_all, rank, 0);
    hipLaunchKernelGGL(gemm_a2a_kernel, dim3(256), dim3(NBLOCK), 0, stream,
                       x, w, d_out_all, d_ws_all, rank, 1);
}
```

```cpp
#include <hip/hip_runtime.h>
#include <rccl/rccl.h>
#include <cstdio>
#include <cstdint>

typedef _Float16 half8 __attribute__((ext_vector_type(8)));
typedef _Float16 half4 __attribute__((ext_vector_type(4)));
typedef float f32x16 __attribute__((ext_vector_type(16)));
typedef float f32x4 __attribute__((ext_vector_type(4)));
typedef unsigned u32x4 __attribute__((ext_vector_type(4)));
typedef short s16x4 __attribute__((ext_vector_type(4)));
typedef short s16x8 __attribute__((ext_vector_type(8)));

constexpr int NR = 8;
constexpr int MLOC = 512;
constexpr int KDIM = 4096;
constexpr int NDIM = 4096;
constexpr int NLOC = 512;
constexpr int BN = 64;
constexpr int KS = 1024;
constexpr int NCW = 8;
constexpr int NPW = 4;
constexpr int NBLOCK = (NCW + NPW) * 64;
constexpr int BLOCKS_PER_PEER = 32;
constexpr unsigned long long DEADLINE_TICKS = 40000000ull;
constexpr size_t XFLAG_OFF = 8192;
constexpr size_t FLAG_OFF = 65536;
constexpr size_t RBS = 262144;
constexpr size_t X16_OFF = 262144;
constexpr size_t PART_OFF = X16_OFF + (5u << 20);

__device__ __forceinline__ unsigned* ws_word(void* ws, int idx) { return (unsigned*)((char*)ws + 128 * idx); }

__device__ __forceinline__ float gelu_f(float y) {
    const float c = 0.7978845608028654f;
    float u = c * (y + 0.044715f * y * y * y);
    return y / (1.0f + __expf(-2.0f * u));
}

#define RST(I)
#define XV 0
#define XDELAY 250
#define FSLEEP 8
#define CSLEEP 4
#define XSLEEP 16
#define GLOAD(DST, VOFF, SBASE) asm volatile("global_load_dwordx4 %0, %1, %2" : "=v"(DST) : "v"(VOFF), "s"(SBASE))
#define GLOADNT(DST, VOFF, SBASE) asm volatile("global_load_dwordx4 %0, %1, %2 nt" : "=v"(DST) : "v"(VOFF), "s"(SBASE))
#define GLOADC(DST, P) asm volatile("global_load_dwordx4 %0, %1, off sc0 sc1" : "=v"(DST) : "v"(P) : "memory")
#define GSTOREC(P, V) asm volatile("global_store_dwordx4 %0, %1, off sc0 sc1\n\ts_nop 1" :: "v"(P), "v"(V) : "memory")
#define LDSRD(DST, P) asm volatile("ds_read_b32 %0, %1\n\ts_waitcnt lgkmcnt(0)" : "=v"(DST) : "v"((unsigned)(uintptr_t)(__attribute__((address_space(3))) unsigned*)(P)) : "memory")

#define AF(V) __builtin_bit_cast(f32x4, V)
#define TAGOK(A, B) (__builtin_amdgcn_ballot_w64((((((A)[0] ^ tag) | ((A)[1] ^ tag)) | (((A)[2] ^ tag) | ((A)[3] ^ tag)) | (((B)[0] ^ tag) | ((B)[1] ^ tag)) | (((B)[2] ^ tag) | ((B)[3] ^ tag))) & 1u) != 0u) == 0ull)
__device__ __forceinline__ float tagged(float f, unsigned tag) { return __uint_as_float((__float_as_uint(f) & ~1u) | tag); }
#define TAGGED(F) tagged(F, tag)
#define FDELAY 0
#define CDELAY 0
#define PF 0
#define CPOLL 1
#define RD 16
#define PACE0 3
#define PACE1 3
#define NTOK 3
template <int PACE>
__device__ __forceinline__ void finish_unit(const f32x4& v0, const f32x4& v1, float* T, float* dst, int lane, bool bad, unsigned* tk) {
    const int fh = lane >> 5;
#pragma unroll
    for (int i = 0; i < 4; ++i) {
        float g0 = gelu_f(v0[i]), g1 = gelu_f(v1[i]);
        T[(4 * fh + i) * 68 + (lane & 31)] = bad ? __builtin_nanf("") : g0;
        T[(4 * fh + i) * 68 + 32 + (lane & 31)] = bad ? __builtin_nanf("") : g1;
    }
    __builtin_amdgcn_wave_barrier();
#pragma unroll
    for (int j = 0; j < 2; ++j) {
        const int r = 4 * j + (lane >> 4), c4 = lane & 15;
        f32x4 v = *(const f32x4*)(T + r * 68 + c4 * 4);
        if (PACE == 1) asm volatile("s_waitcnt vmcnt(0)" ::: "memory");
        if (PACE == 2) asm volatile("s_waitcnt vmcnt(1)" ::: "memory");
        if (PACE == 3) {
            unsigned t = 0;
            if (lane == 0) t = __hip_atomic_fetch_add(tk, 1u, __ATOMIC_RELAXED, __HIP_MEMORY_SCOPE_WORKGROUP);
            t = __builtin_amdgcn_readfirstlane(t);
            unsigned long long t0 = __builtin_amdgcn_s_memrealtime();
            while (true) {
                unsigned d; LDSRD(d, tk + 1);
                if (t < __builtin_amdgcn_readfirstlane(d) + NTOK) break;
                if (__builtin_amdgcn_s_memrealtime() - t0 > DEADLINE_TICKS) break;
                __builtin_amdgcn_s_sleep(1);
            }
        }
        GSTOREC(dst + (size_t)r * NLOC + c4 * 4, v);
        if (PACE == 3) {
            asm volatile("s_waitcnt vmcnt(0)" ::: "memory");
            if (lane == 0) __hip_atomic_fetch_add(tk + 1, 1u, __ATOMIC_RELAXED, __HIP_MEMORY_SCOPE_WORKGROUP);
        }
    }
    __builtin_amdgcn_wave_barrier();
}

__global__ __launch_bounds__(NBLOCK, 3)
void gemm_a2a_kernel(const float* __restrict__ x, const float* __restrict__ w,
                     void* const* __restrict__ out_all, void* const* __restrict__ ws_all, int rank) {
    __shared__ __attribute__((aligned(16))) _Float16 Bs[KS * BN];
    __shared__ __attribute__((aligned(16))) float Ts[NCW + NPW][8 * 68];
    __shared__ __attribute__((aligned(16))) unsigned rdy[4];
    __shared__ __attribute__((aligned(16))) unsigned sv[8];

#define RXT(I)
#define SXT(I)
    RST(0);
    const int tid = threadIdx.x;
    const int lane = tid & 63, wid = __builtin_amdgcn_readfirstlane(tid >> 6);
    char* myws;
    {
        unsigned long long a = (unsigned long long)ws_all[rank];
        unsigned lo = __builtin_amdgcn_readfirstlane((unsigned)a), hi = __builtin_amdgcn_readfirstlane((unsigned)(a >> 32));
        myws = (char*)(((unsigned long long)hi << 32) | lo);
    }
    char* x16 = myws + X16_OFF;
    char* part = myws + PART_OFF;
    const __amdgpu_buffer_rsrc_t prs = __builtin_amdgcn_make_buffer_rsrc(part, 0, 32u << 20, 0x00020000);
    const int b = blockIdx.x;
#define MAP 0
    const int q = b & 3;
    const int grp = ((b >> 2) & 1) * 32 + (b >> 3);
    const int peer = ((grp >> 3) + rank) & 7;
    const int ct = grp & 7;
    const int ncol0 = peer * NLOC + ct * BN;
    float* dst = (float*)out_all[peer] + (size_t)(rank * MLOC) * NLOC + ct * BN;
    bool bad = false;

    if (wid >= NCW) {
        const int p = wid - NCW;
        const int kr = lane >> 4, nq = lane & 15;
        const unsigned voff = (unsigned)((4 * p + kr) * NDIM + nq * 4) * 4u;
        const char* base = (const char*)(w + (size_t)q * KS * NDIM + ncol0);
#define PD 2
        f32x4 rb[4 * PD];
#define PLD(SLOT, S) _Pragma("unroll") for (int i = 0; i < 4; ++i) GLOADNT(rb[(SLOT) * 4 + i], voff, base + (size_t)((S) * 64 + 16 * i) * NDIM * 4)
#define PWAIT(N, SLOT) asm volatile("s_waitcnt vmcnt(" #N ")" : "+v"(rb[(SLOT) * 4]), "+v"(rb[(SLOT) * 4 + 1]), "+v"(rb[(SLOT) * 4 + 2]), "+v"(rb[(SLOT) * 4 + 3]))
#define PST(SLOT, S) do { \
        _Pragma("unroll") for (int i = 0; i < 4; ++i) { \
            f32x4 v = rb[(SLOT) * 4 + i]; \
            half4 h; h[0] = (_Float16)v[0]; h[1] = (_Float16)v[1]; h[2] = (_Float16)v[2]; h[3] = (_Float16)v[3]; \
            *(half4*)(Bs + ((S) * 64 + 16 * i + 4 * p + kr) * BN + ((nq * 4) ^ (32 * ((kr >> 1) & 1)))) = h; \
        } \
        asm volatile("" ::: "memory"); \
        if (lane == 0) *(volatile unsigned*)&rdy[p] = (unsigned)(S) + 1u; \
        asm volatile("" ::: "memory"); } while (0)
#define BDELAY 1
        if (lane == 0) rdy[p] = 0;
        __syncthreads();
        {
            unsigned long long t0 = __builtin_amdgcn_s_memrealtime();
            while (true) {
                unsigned c; LDSRD(c, &sv[0]);
                if (__builtin_amdgcn_readfirstlane(c) >= (unsigned)NCW) break;
                if (__builtin_amdgcn_s_memrealtime() - t0 > DEADLINE_TICKS) break;
                __builtin_amdgcn_s_sleep(1);
            }
        }
        _Pragma("unroll") for (int s = 0; s < PD; ++s) { PLD(s, s); }
#pragma unroll
        for (int s = 0; s < 16; ++s) {
            const int left = (15 - s < PD - 1) ? 15 - s : PD - 1;
            if (left == 5) PWAIT(20, s % PD); else if (left == 4) PWAIT(16, s % PD); else if (left == 3) PWAIT(12, s % PD); else if (left == 2) PWAIT(8, s % PD); else if (left == 1) PWAIT(4, s % PD); else PWAIT(0, s % PD);
            PST(s % PD, s);
            if ((s & 3) == 0) RXT(4 + (s >> 2));
            if (s + PD < 16) PLD(s % PD, s + PD);
        }
        RST(3);
        unsigned call;
        unsigned long long t0 = __builtin_amdgcn_s_memrealtime();
        while (true) {
            LDSRD(call, &sv[1]);
            if (call != 0u) break;
            if (__builtin_amdgcn_s_memrealtime() - t0 > DEADLINE_TICKS) { call = 0x80000000u; break; }
            __builtin_amdgcn_s_sleep(2);
        }
        call = __builtin_amdgcn_readfirstlane(call);
        if (call & 0x80000000u) bad = true;
        call &= 0x7fffffffu;
        while (true) {
            unsigned c; LDSRD(c, &sv[3]);
            if (__builtin_amdgcn_readfirstlane(c) >= (unsigned)NCW) break;
            if (__builtin_amdgcn_s_memrealtime() - t0 > DEADLINE_TICKS) { bad = true; break; }
            __builtin_amdgcn_s_sleep(2);
        }
        const unsigned tag = call & 1u;
        const unsigned voff0 = (unsigned)((grp * 4 * 8 + 2 * p) * 16384 + q * 2048 + lane * 16);
        u32x4 pv[16];
        unsigned pend = 0xffu;
        {
            const unsigned po = (unsigned)((grp * 4 * 8 + 2 * p) * 16384 + q * 2048 + 2044) + (unsigned)(((lane & 3) * 8 + ((lane >> 2) & 1)) * 16384);
            while (true) {
                unsigned fv = tag;
                if (lane < 8) fv = __builtin_amdgcn_raw_buffer_load_b32(prs, po, 0, 17);
                if (__builtin_amdgcn_ballot_w64(((fv ^ tag) & 1u) != 0u) == 0ull) break;
                if (__builtin_amdgcn_s_memrealtime() - t0 > DEADLINE_TICKS) { bad = true; break; }
                __builtin_amdgcn_s_sleep(FSLEEP);
                asm volatile("" ::: "memory");
            }
        }
        SXT(4);
        while (true) {
#pragma unroll
            for (int i = 0; i < 8; ++i)
                if (pend & (1u << i)) {
                    const unsigned o = voff0 + (unsigned)(((i & 3) * 8 + (i >> 2)) * 16384);
                    pv[2 * i] = __builtin_amdgcn_raw_buffer_load_b128(prs, o, 0, 17);
                    pv[2 * i + 1] = __builtin_amdgcn_raw_buffer_load_b128(prs, o + 1024u, 0, 17);
                }
#pragma unroll
            for (int i = 0; i < 8; ++i)
                if (pend & (1u << i)) {
                    if (TAGOK(pv[2 * i], pv[2 * i + 1])) pend &= ~(1u << i);
                }
            if (pend == 0u) break;
            if (__builtin_amdgcn_s_memrealtime() - t0 > DEADLINE_TICKS) { bad = true; break; }
            __builtin_amdgcn_s_sleep(FSLEEP);
            asm volatile("" ::: "memory");
        }
        SXT(5);
        {
            f32x4 v0 = (AF(pv[0]) + AF(pv[2])) + (AF(pv[4]) + AF(pv[6])), v1 = (AF(pv[1]) + AF(pv[3])) + (AF(pv[5]) + AF(pv[7]));
            finish_unit<PACE0>(v0, v1, Ts[wid], dst + (size_t)(64 * (2 * p) + 8 * q) * NLOC, lane, bad, &sv[4]);
        }
        {
            f32x4 v0 = (AF(pv[8]) + AF(pv[10])) + (AF(pv[12]) + AF(pv[14])), v1 = (AF(pv[9]) + AF(pv[11])) + (AF(pv[13]) + AF(pv[15]));
            finish_unit<PACE0>(v0, v1, Ts[wid], dst + (size_t)(64 * (2 * p + 1) + 8 * q) * NLOC, lane, bad, &sv[4]);
        }
        SXT(6); SXT(7);
        asm volatile("s_waitcnt vmcnt(0)" ::: "memory");
        __syncthreads();
        return;
    }

    unsigned call;
    {
        unsigned old = 0;
        if (tid == 0) { sv[0] = 0; sv[1] = 0; sv[2] = 0; sv[3] = 0; sv[4] = 0; sv[5] = 0; }
        const int rbk = grp >> 2, ks = q * 64 + (grp & 3) * 16 + 2 * wid;
        const float* src = x + (size_t)(32 * rbk + (lane & 31)) * KDIM + ks * 16 + 8 * (lane >> 5);
        f32x4 v[4];
#pragma unroll
        for (int i = 0; i < 4; ++i)
            asm volatile("global_load_dwordx4 %0, %1, off nt" : "=v"(v[i]) : "v"(src + (i >> 1) * 16 + (i & 1) * 4));
        __syncthreads();
        if (tid == 0) old = __hip_atomic_load(ws_word(myws, 0), __ATOMIC_RELAXED, __HIP_MEMORY_SCOPE_AGENT);
        RXT(0);
        asm volatile("s_waitcnt vmcnt(0)" : "+v"(v[0]), "+v"(v[1]), "+v"(v[2]), "+v"(v[3]));
        RXT(1);
        if (lane == 0) __hip_atomic_fetch_add(&sv[0], 1u, __ATOMIC_RELAXED, __HIP_MEMORY_SCOPE_WORKGROUP);
        if (tid == 0) *(volatile unsigned*)&sv[2] = (old >> 8) + 1u;
#pragma unroll
        for (int i = 0; i < 2; ++i) {
            half8 h;
#pragma unroll
            for (int e = 0; e < 4; ++e) { h[e] = (_Float16)v[2 * i][e]; h[4 + e] = (_Float16)v[2 * i + 1][e]; }
            GSTOREC(x16 + (size_t)rbk * RBS + (size_t)((ks + i) * 64 + lane) * 16, __builtin_bit_cast(f32x4, h));
        }
        unsigned long long t0 = __builtin_amdgcn_s_memrealtime();
        while (true) {
            LDSRD(call, &sv[2]);
            if (call != 0u) break;
            if (__builtin_amdgcn_s_memrealtime() - t0 > DEADLINE_TICKS) { bad = true; break; }
            __builtin_amdgcn_s_sleep(1);
        }
        call = __builtin_amdgcn_readfirstlane(call);
        RXT(2);
        asm volatile("s_waitcnt vmcnt(0)" ::: "memory");
        RXT(3);
        unsigned* xf = (unsigned*)(myws + XFLAG_OFF) + q * 512;
        if (lane == 0) __hip_atomic_store(xf + grp * 8 + wid, call, __ATOMIC_RELAXED, __HIP_MEMORY_SCOPE_SYSTEM);
        if (wid == 0) {
            while (__builtin_amdgcn_s_memrealtime() - t0 < XDELAY) __builtin_amdgcn_s_sleep(4);
            while (true) {
                f32x4 f0, f1;
                GLOADC(f0, xf + lane * 8); GLOADC(f1, xf + lane * 8 + 4);
                asm volatile("s_waitcnt vmcnt(0)" : "+v"(f0), "+v"(f1));
                u32x4 u0 = __builtin_bit_cast(u32x4, f0), u1 = __builtin_bit_cast(u32x4, f1);
                bool ok = u0[0] == call && u0[1] == call && u0[2] == call && u0[3] == call && u1[0] == call && u1[1] == call && u1[2] == call && u1[3] == call;
                if (__builtin_amdgcn_ballot_w64(!ok) == 0ull) break;
                if (__builtin_amdgcn_s_memrealtime() - t0 > DEADLINE_TICKS) { bad = true; break; }
                __builtin_amdgcn_s_sleep(XSLEEP);
            }
            asm volatile("" ::: "memory");
            if (lane == 0) *(volatile unsigned*)&sv[1] = call | (bad ? 0x80000000u : 0u);
        } else {
            while (true) {
                LDSRD(call, &sv[1]);
                if (call != 0u) break;
                if (__builtin_amdgcn_s_memrealtime() - t0 > DEADLINE_TICKS) { call = 0x80000000u; break; }
                __builtin_amdgcn_s_sleep(1);
            }
            call = __builtin_amdgcn_readfirstlane(call);
            if (call & 0x80000000u) bad = true;
            call &= 0x7fffffffu;
        }
        asm volatile("s_waitcnt vmcnt(0) lgkmcnt(0)" ::: "memory");
    }
    RST(1);

    const unsigned a_voff = (unsigned)lane * 16u;
    const char* a_base = x16 + (size_t)(2 * wid) * RBS + (size_t)(64 * q) * 1024;
    f32x4 ra[RD];
#define AOFF(S) ((size_t)((S) >> 6) * RBS + (size_t)((S) & 63) * 1024)
#pragma unroll
    for (int i = 0; i < RD; ++i) GLOAD(ra[i], a_voff, a_base + AOFF(i));

    f32x16 acc[2];
#pragma unroll
    for (int t = 0; t < 2; ++t)
#pragma unroll
        for (int i = 0; i < 16; ++i) acc[t][i] = 0.f;

    const int fh = lane >> 5;
    const int bblk = (lane >> 4) & 1, bq = (lane & 15) >> 2, bp = lane & 3;
    const unsigned lds0 = (unsigned)(uintptr_t)(__attribute__((address_space(3))) _Float16*)Bs;
    const unsigned b0p = lds0 + 2u * ((8 * fh + bq) * BN + ((16 * bblk + 4 * bp) ^ (32 * ((bq >> 1) & 1))));
    const unsigned b1p = lds0 + 2u * ((8 * fh + bq) * BN + ((32 + 16 * bblk + 4 * bp) ^ (32 * ((bq >> 1) & 1))));
#define BBASE(BLK) unsigned bs0 = b0p + (BLK) * 512u * BN, bs1 = b1p + (BLK) * 512u * BN; asm volatile("" : "+v"(bs0), "+v"(bs1))
    unsigned seen = 0;
#define BPOLL() do { \
        u32x4 rv; \
        asm volatile("ds_read_b128 %0, %1\n\ts_waitcnt lgkmcnt(0)" : "=v"(rv) : "v"((unsigned)(uintptr_t)(__attribute__((address_space(3))) unsigned*)rdy) : "memory"); \
        seen = __builtin_amdgcn_readfirstlane(min(min(rv[0], rv[1]), min(rv[2], rv[3]))); } while (0)
#define BWAIT(S) do { \
        if (seen <= (unsigned)(S)) { \
            BPOLL(); \
            if (seen <= (unsigned)(S)) { \
                unsigned long long t0 = __builtin_amdgcn_s_memrealtime(); \
                for (unsigned n = 1; ; ++n) { \
                    __builtin_amdgcn_s_sleep(2); \
                    BPOLL(); \
                    if (seen > (unsigned)(S)) break; \
                    if ((n & 255u) == 0u && __builtin_amdgcn_s_memrealtime() - t0 > DEADLINE_TICKS) { bad = true; seen = 64; break; } \
                } \
            } \
        } } while (0)
#define TR(P) __builtin_amdgcn_ds_read_tr16_b64_v4i16((__attribute__((address_space(3))) s16x4*)(uintptr_t)(P))
#define STEP(G, J) do { \
        s16x4 l0 = TR(bs0 + (J) * 32 * BN), h0 = TR(bs0 + ((J) * 32 + 8) * BN); \
        s16x4 l1 = TR(bs1 + (J) * 32 * BN), h1 = TR(bs1 + ((J) * 32 + 8) * BN); \
        half8 fb0 = __builtin_bit_cast(half8, __builtin_shufflevector(l0, h0, 0, 1, 2, 3, 4, 5, 6, 7)); \
        half8 fb1 = __builtin_bit_cast(half8, __builtin_shufflevector(l1, h1, 0, 1, 2, 3, 4, 5, 6, 7)); \
        asm volatile("s_waitcnt vmcnt(%1)" : "+v"(ra[(G) % RD]) : "n"(((G) > 127 - RD ? 127 - (G) : RD - 1) + ((G) >= 64 && (G) < 64 + RD ? 8 : 0))); \
        half8 fa = __builtin_bit_cast(half8, ra[(G) % RD]); \
        acc[0] = __builtin_amdgcn_mfma_f32_32x32x16_f16(fa, fb0, acc[0], 0, 0, 0); \
        acc[1] = __builtin_amdgcn_mfma_f32_32x32x16_f16(fa, fb1, acc[1], 0, 0, 0); \
        if ((G) + RD < 128) GLOAD(ra[(G) % RD], a_voff, a_base + AOFF((G) + RD)); \
    } while (0)
#define SB() __builtin_amdgcn_sched_barrier(0)
#define STEP4(G0, J0) do { STEP((G0), (J0)); STEP((G0) + 1, (J0) + 1); STEP((G0) + 2, (J0) + 2); STEP((G0) + 3, (J0) + 3); SB(); } while (0)
#define BLOCK1(B) do { BBASE(B); \
        BWAIT(4 * (B));     STEP4(16 * (B), 0); \
        BWAIT(4 * (B) + 1); STEP4(16 * (B) + 4, 4); \
        BWAIT(4 * (B) + 2); STEP4(16 * (B) + 8, 8); \
        BWAIT(4 * (B) + 3); STEP4(16 * (B) + 12, 12); } while (0)
#define BLOCK2(B) do { BBASE(B); \
        STEP4(64 + 16 * (B), 0); STEP4(64 + 16 * (B) + 4, 4); STEP4(64 + 16 * (B) + 8, 8); STEP4(64 + 16 * (B) + 12, 12); } while (0)

    BLOCK1(0); BLOCK1(1); BLOCK1(2); BLOCK1(3);
    RST(2);
    char* mypart = part + (size_t)((grp * 4 + q) * 8 + wid) * 16384 + lane * 16;
    const unsigned tag = call & 1u;
#pragma unroll
    for (int G = 0; G < 4; ++G)
#pragma unroll
        for (int t = 0; t < 2; ++t) {
            f32x4 v; v[0] = TAGGED(acc[t][4 * G]); v[1] = TAGGED(acc[t][4 * G + 1]); v[2] = TAGGED(acc[t][4 * G + 2]); v[3] = TAGGED(acc[t][4 * G + 3]);
            GSTOREC(mypart + G * 2048 + t * 1024, v);
        }
    if (lane == 0) __hip_atomic_fetch_add(&sv[3], 1u, __ATOMIC_RELAXED, __HIP_MEMORY_SCOPE_WORKGROUP);
#pragma unroll
    for (int t = 0; t < 2; ++t)
#pragma unroll
        for (int i = 0; i < 16; ++i) acc[t][i] = 0.f;

    BLOCK2(0); BLOCK2(1); BLOCK2(2); BLOCK2(3);
    RST(3);

    f32x4 own[2];
    {
#pragma unroll
        for (int G = 0; G < 4; ++G) {
            if (G == q) {
#pragma unroll
                for (int t = 0; t < 2; ++t) { own[t][0] = acc[t][4 * G]; own[t][1] = acc[t][4 * G + 1]; own[t][2] = acc[t][4 * G + 2]; own[t][3] = acc[t][4 * G + 3]; }
            } else {
                char* d = mypart + 8192 + ((G - q - 1) & 3) * 2048;
#pragma unroll
                for (int t = 0; t < 2; ++t) {
                    f32x4 v; v[0] = TAGGED(acc[t][4 * G]); v[1] = TAGGED(acc[t][4 * G + 1]); v[2] = TAGGED(acc[t][4 * G + 2]); v[3] = TAGGED(acc[t][4 * G + 3]);
                    GSTOREC(d + t * 1024, v);
                }
            }
        }
        SXT(0);
        u32x4 pv[6];
        unsigned pend = 7u;
        unsigned long long t0 = __builtin_amdgcn_s_memrealtime();
        if (CDELAY) __builtin_amdgcn_s_sleep(CDELAY);
        while (true) {
#pragma unroll
            for (int d = 0; d < 3; ++d)
                if (pend & (1u << d)) {
                    const int p = (q + 1 + d) & 3;
                    const unsigned o = (unsigned)(((grp * 4 + p) * 8 + wid) * 16384 + 8192 + ((q - p - 1) & 3) * 2048 + lane * 16);
                    pv[2 * d] = __builtin_amdgcn_raw_buffer_load_b128(prs, o, 0, 17);
                    pv[2 * d + 1] = __builtin_amdgcn_raw_buffer_load_b128(prs, o + 1024u, 0, 17);
                }
#pragma unroll
            for (int d = 0; d < 3; ++d)
                if (pend & (1u << d)) {
                    if (TAGOK(pv[2 * d], pv[2 * d + 1])) pend &= ~(1u << d);
                }
            if (pend == 0u) break;
            if (__builtin_amdgcn_s_memrealtime() - t0 > DEADLINE_TICKS) { bad = true; break; }
            __builtin_amdgcn_s_sleep(CSLEEP);
            asm volatile("" ::: "memory");
        }
        SXT(1); SXT(2);
#pragma unroll
        for (int t = 0; t < 2; ++t) own[t] = (own[t] + AF(pv[t])) + (AF(pv[2 + t]) + AF(pv[4 + t]));
    }
    RST(4);
    finish_unit<PACE1>(own[0], own[1], Ts[wid], dst + (size_t)(64 * wid + 32 + 8 * q) * NLOC, lane, bad, &sv[4]);
    asm volatile("s_waitcnt vmcnt(0)" ::: "memory");
    SXT(3);
    __syncthreads();
    RST(5);
    if (tid == 0) __hip_atomic_fetch_add(ws_word(myws, 0), 1u, __ATOMIC_RELAXED, __HIP_MEMORY_SCOPE_AGENT);
    if (tid == 0) {
        if (peer != rank) {
            __builtin_amdgcn_fence(__ATOMIC_RELEASE, "");
            asm volatile("s_waitcnt vmcnt(0)" ::: "memory");
            __hip_atomic_fetch_add(ws_word(ws_all[peer], 2 + rank), 1u, __ATOMIC_RELAXED, __HIP_MEMORY_SCOPE_SYSTEM);
        }
        if (b == 0) {
            unsigned need = call * (unsigned)BLOCKS_PER_PEER;
            unsigned long long t0 = __builtin_amdgcn_s_memrealtime();
            bool timed_out = false;
            for (int p = 0; p < NR; ++p) {
                if (p == rank) continue;
                while (__hip_atomic_load(ws_word(myws, 2 + p), __ATOMIC_RELAXED, __HIP_MEMORY_SCOPE_SYSTEM) < need) {
                    if (__builtin_amdgcn_s_memrealtime() - t0 > DEADLINE_TICKS) { timed_out = true; break; }
                    __builtin_amdgcn_s_sleep(2);
                }
            }
            __builtin_amdgcn_fence(__ATOMIC_ACQUIRE, "");
            if (timed_out) {
                __hip_atomic_store(ws_word(myws, 1), call, __ATOMIC_RELAXED, __HIP_MEMORY_SCOPE_AGENT);
                ((float*)out_all[rank])[(size_t)(rank * MLOC) * NLOC] = __builtin_nanf("");
            }
        }
    }
}

extern "C" void kernel_launch(int rank, int nranks, ncclComm_t comm,
                              hipStream_t stream,
                              void* const* d_in, const int* in_sizes, int n_in,
                              void* d_out, int out_size,
                              const int* in_dtypes, int out_dtype,
                              void* const* d_out_all,
                              void* const* d_ws_all, size_t ws_bytes) {
    const float* x = (const float*)d_in[0];
    const float* w = (const float*)d_in[1];
    hipLaunchKernelGGL(gemm_a2a_kernel, dim3(256), dim3(NBLOCK), 0, stream,
                       x, w, d_out_all, d_ws_all, rank);
}
```

```cpp
#include <hip/hip_runtime.h>
#include <rccl/rccl.h>
#include <cstdio>
#include <cstdint>

typedef _Float16 half8 __attribute__((ext_vector_type(8)));
typedef _Float16 half4 __attribute__((ext_vector_type(4)));
typedef float f32x16 __attribute__((ext_vector_type(16)));
typedef float f32x4 __attribute__((ext_vector_type(4)));
typedef unsigned u32x4 __attribute__((ext_vector_type(4)));
typedef short s16x4 __attribute__((ext_vector_type(4)));
typedef short s16x8 __attribute__((ext_vector_type(8)));

constexpr int NR = 8;
constexpr int MLOC = 512;
constexpr int KDIM = 4096;
constexpr int NDIM = 4096;
constexpr int NLOC = 512;
constexpr int BN = 64;
constexpr int KS = 1024;
constexpr int NCW = 8;
constexpr int NPW = 4;
constexpr int NBLOCK = (NCW + NPW) * 64;
constexpr int BLOCKS_PER_PEER = 32;
constexpr unsigned long long DEADLINE_TICKS = 40000000ull;
constexpr size_t XFLAG_OFF = 8192;
constexpr size_t FLAG_OFF = 65536;
constexpr size_t RBS = 262144;
constexpr size_t X16_OFF = 262144;
constexpr size_t PART_OFF = X16_OFF + (5u << 20);

__device__ __forceinline__ unsigned* ws_word(void* ws, int idx) { return (unsigned*)((char*)ws + 128 * idx); }

__device__ __forceinline__ unsigned long long rtc() {
    unsigned long long t;
    asm volatile("s_memrealtime %0\n\ts_waitcnt lgkmcnt(0)" : "=s"(t) :: "memory");
    return t;
}

__device__ __forceinline__ float gelu_f(float y) {
    const float c = 0.7978845608028654f;
    float u = c * (y + 0.044715f * y * y * y);
    return y / (1.0f + __expf(-2.0f * u));
}

#define RST(I)
#define XV 0
#define XDELAY 250
#define FSLEEP 8
#define CSLEEP 4
#define XSLEEP 16
#define GLOAD(DST, VOFF, SBASE) asm volatile("global_load_dwordx4 %0, %1, %2" : "=v"(DST) : "v"(VOFF), "s"(SBASE))
#define GLOADNT(DST, VOFF, SBASE) asm volatile("global_load_dwordx4 %0, %1, %2 nt" : "=v"(DST) : "v"(VOFF), "s"(SBASE))
#define GLOADC(DST, P) asm volatile("global_load_dwordx4 %0, %1, off sc0 sc1" : "=v"(DST) : "v"(P) : "memory")
#define GSTOREC(P, V) asm volatile("global_store_dwordx4 %0, %1, off sc0 sc1\n\ts_nop 1" :: "v"(P), "v"(V) : "memory")
#define LDSRD(DST, P) asm volatile("ds_read_b32 %0, %1\n\ts_waitcnt lgkmcnt(0)" : "=v"(DST) : "v"((unsigned)(uintptr_t)(__attribute__((address_space(3))) unsigned*)(P)) : "memory")

#define AF(V) __builtin_bit_cast(f32x4, V)
#define TAGOK(A, B) (__builtin_amdgcn_ballot_w64((((((A)[0] ^ tag) | ((A)[1] ^ tag)) | (((A)[2] ^ tag) | ((A)[3] ^ tag)) | (((B)[0] ^ tag) | ((B)[1] ^ tag)) | (((B)[2] ^ tag) | ((B)[3] ^ tag))) & 1u) != 0u) == 0ull)
__device__ __forceinline__ float tagged(float f, unsigned tag) { return __uint_as_float((__float_as_uint(f) & ~1u) | tag); }
#define TAGGED(F) tagged(F, tag)
#define FDELAY 0
#define CDELAY 0
#define PF 0
#define CPOLL 1
#define RD 12
#define PACE0 3
#define PACE1 3
#define NTOK 3
#define GMODE 1
#define PU 0
#define PW 4
template <int PACE>
__device__ __forceinline__ void finish_unit(const f32x4& v0, const f32x4& v1, float* T, float* dst, int lane, bool bad, unsigned* tk) {
    const int fh = lane >> 5;
#pragma unroll
    for (int i = 0; i < 4; ++i) {
        float g0 = gelu_f(v0[i]), g1 = gelu_f(v1[i]);
        T[(4 * fh + i) * 68 + (lane & 31)] = bad ? __builtin_nanf("") : g0;
        T[(4 * fh + i) * 68 + 32 + (lane & 31)] = bad ? __builtin_nanf("") : g1;
    }
    __builtin_amdgcn_wave_barrier();
#pragma unroll
    for (int j = 0; j < 2; ++j) {
        const int r = 4 * j + (lane >> 4), c4 = lane & 15;
        f32x4 v = *(const f32x4*)(T + r * 68 + c4 * 4);
        if (PACE == 1) asm volatile("s_waitcnt vmcnt(0)" ::: "memory");
        if (PACE == 2) asm volatile("s_waitcnt vmcnt(1)" ::: "memory");
        if (PACE == 3) {
            unsigned t = 0;
            if (lane == 0) t = __hip_atomic_fetch_add(tk, 1u, __ATOMIC_RELAXED, __HIP_MEMORY_SCOPE_WORKGROUP);
            t = __builtin_amdgcn_readfirstlane(t);
            unsigned long long t0 = __builtin_amdgcn_s_memrealtime();
            while (true) {
                unsigned d; LDSRD(d, tk + 1);
                if (t < __builtin_amdgcn_readfirstlane(d) + NTOK) break;
                if (__builtin_amdgcn_s_memrealtime() - t0 > DEADLINE_TICKS) break;
                __builtin_amdgcn_s_sleep(1);
            }
        }
        GSTOREC(dst + (size_t)r * NLOC + c4 * 4, v);
        if (PACE == 3) {
            asm volatile("s_waitcnt vmcnt(0)" ::: "memory");
            if (lane == 0) __hip_atomic_fetch_add(tk + 1, 1u, __ATOMIC_RELAXED, __HIP_MEMORY_SCOPE_WORKGROUP);
        }
    }
    __builtin_amdgcn_wave_barrier();
}

__global__ __launch_bounds__(NBLOCK, 3)
void gemm_a2a_kernel(const float* __restrict__ x, const float* __restrict__ w,
                     void* const* __restrict__ out_all, void* const* __restrict__ ws_all, int rank) {
    __shared__ __attribute__((aligned(16))) _Float16 Bs[KS * BN];
    __shared__ __attribute__((aligned(16))) float Ts[NCW + NPW][8 * 68];
    __shared__ __attribute__((aligned(16))) unsigned rdy[4];
    __shared__ __attribute__((aligned(16))) unsigned sv[16];

#define RXT(I)
#define SXT(I)
    RST(0);
    const int tid = threadIdx.x;
    const int lane = tid & 63, wid = __builtin_amdgcn_readfirstlane(tid >> 6);
    char* myws;
    {
        unsigned long long a = (unsigned long long)ws_all[rank];
        unsigned lo = __builtin_amdgcn_readfirstlane((unsigned)a), hi = __builtin_amdgcn_readfirstlane((unsigned)(a >> 32));
        myws = (char*)(((unsigned long long)hi << 32) | lo);
    }
    char* x16 = myws + X16_OFF;
    char* part = myws + PART_OFF;
    const __amdgpu_buffer_rsrc_t prs = __builtin_amdgcn_make_buffer_rsrc(part, 0, 32u << 20, 0x00020000);
    const int b = blockIdx.x;
#define MAP 0
    const int q = b & 3;
    const int grp = ((b >> 2) & 1) * 32 + (b >> 3);
    const int peer = ((grp >> 3) + rank) & 7;
    const int ct = grp & 7;
    const int ncol0 = peer * NLOC + ct * BN;
    float* dst = (float*)out_all[peer] + (size_t)(rank * MLOC) * NLOC + ct * BN;
    bool bad = false;

    if (wid >= NCW) {
        const int p = wid - NCW;
        const int kr = lane >> 4, nq = lane & 15;
        const unsigned voff = (unsigned)((4 * p + kr) * NDIM + nq * 4) * 4u;
        const char* base = (const char*)(w + (size_t)q * KS * NDIM + ncol0);
#define PD 2
        f32x4 rb[4 * PD];
#define PLD(SLOT, S) _Pragma("unroll") for (int i = 0; i < 4; ++i) GLOADNT(rb[(SLOT) * 4 + i], voff, base + (size_t)((S) * 64 + 16 * i) * NDIM * 4)
#define PWAIT(N, SLOT) asm volatile("s_waitcnt vmcnt(" #N ")" : "+v"(rb[(SLOT) * 4]), "+v"(rb[(SLOT) * 4 + 1]), "+v"(rb[(SLOT) * 4 + 2]), "+v"(rb[(SLOT) * 4 + 3]))
#define PST(SLOT, S) do { \
        _Pragma("unroll") for (int i = 0; i < 4; ++i) { \
            f32x4 v = rb[(SLOT) * 4 + i]; \
            half4 h; h[0] = (_Float16)v[0]; h[1] = (_Float16)v[1]; h[2] = (_Float16)v[2]; h[3] = (_Float16)v[3]; \
            *(half4*)(Bs + ((S) * 64 + 16 * i + 4 * p + kr) * BN + ((nq * 4) ^ (32 * ((kr >> 1) & 1)))) = h; \
        } \
        asm volatile("" ::: "memory"); \
        if (lane == 0) *(volatile unsigned*)&rdy[p] = (unsigned)(S) + 1u; \
        asm volatile("" ::: "memory"); } while (0)
#define BDELAY 1
        if (lane == 0) rdy[p] = 0;
        __syncthreads();
        {
            unsigned long long t0 = __builtin_amdgcn_s_memrealtime();
            while (true) {
                unsigned c; LDSRD(c, &sv[0]);
                if (__builtin_amdgcn_readfirstlane(c) >= (unsigned)NCW) break;
                if (__builtin_amdgcn_s_memrealtime() - t0 > DEADLINE_TICKS) break;
                __builtin_amdgcn_s_sleep(1);
            }
        }
        _Pragma("unroll") for (int s = 0; s < PD; ++s) { PLD(s, s); }
#pragma unroll
        for (int s = 0; s < 16; ++s) {
            const int left = (15 - s < PD - 1) ? 15 - s : PD - 1;
            if (left == 5) PWAIT(20, s % PD); else if (left == 4) PWAIT(16, s % PD); else if (left == 3) PWAIT(12, s % PD); else if (left == 2) PWAIT(8, s % PD); else if (left == 1) PWAIT(4, s % PD); else PWAIT(0, s % PD);
            PST(s % PD, s);
            if ((s & 3) == 0) RXT(4 + (s >> 2));
            if (s + PD < 16) PLD(s % PD, s + PD);
        }
        RST(3);
        unsigned call;
        unsigned long long t0 = __builtin_amdgcn_s_memrealtime();
        while (true) {
            LDSRD(call, &sv[1]);
            if (call != 0u) break;
            if (__builtin_amdgcn_s_memrealtime() - t0 > DEADLINE_TICKS) { call = 0x80000000u; break; }
            __builtin_amdgcn_s_sleep(2);
        }
        call = __builtin_amdgcn_readfirstlane(call);
        if (call & 0x80000000u) bad = true;
        call &= 0x7fffffffu;
        const unsigned tag = call & 1u;
#pragma unroll 1
        for (int u = 0; u < 2; ++u) {
            const int wv = p + 4 * u;
            while (true) {
                unsigned c; LDSRD(c, &sv[8 + wv]);
                if (__builtin_amdgcn_readfirstlane(c) != 0u) break;
                if (__builtin_amdgcn_s_memrealtime() - t0 > DEADLINE_TICKS) { bad = true; break; }
                __builtin_amdgcn_s_sleep(2);
            }
            const unsigned voff0 = (unsigned)((grp * 4 * 8 + wv) * 16384 + q * 2048 + lane * 16);
            u32x4 pv[8];
            unsigned pend = 0xfu;
            {
                const unsigned po = (unsigned)((grp * 4 * 8 + wv) * 16384 + q * 2048 + 2044) + (unsigned)((lane & 3) * 8 * 16384);
                while (true) {
                    unsigned fv = tag;
                    if (lane < 4) fv = __builtin_amdgcn_raw_buffer_load_b32(prs, po, 0, 17);
                    if (__builtin_amdgcn_ballot_w64(((fv ^ tag) & 1u) != 0u) == 0ull) break;
                    if (__builtin_amdgcn_s_memrealtime() - t0 > DEADLINE_TICKS) { bad = true; break; }
                    __builtin_amdgcn_s_sleep(FSLEEP);
                    asm volatile("" ::: "memory");
                }
            }
            if (u == 0) SXT(4);
            if (GMODE == 3 && u <= PU && lane == 0) __hip_atomic_fetch_add(&sv[6], 1u, __ATOMIC_RELAXED, __HIP_MEMORY_SCOPE_WORKGROUP);
            while (true) {
#pragma unroll
                for (int i = 0; i < 4; ++i)
                    if (pend & (1u << i)) {
                        const unsigned o = voff0 + (unsigned)(i * 8 * 16384);
                        pv[2 * i] = __builtin_amdgcn_raw_buffer_load_b128(prs, o, 0, 17);
                        pv[2 * i + 1] = __builtin_amdgcn_raw_buffer_load_b128(prs, o + 1024u, 0, 17);
                    }
#pragma unroll
                for (int i = 0; i < 4; ++i)
                    if (pend & (1u << i)) {
                        if (TAGOK(pv[2 * i], pv[2 * i + 1])) pend &= ~(1u << i);
                    }
                if (pend == 0u) break;
                if (__builtin_amdgcn_s_memrealtime() - t0 > DEADLINE_TICKS) { bad = true; break; }
                __builtin_amdgcn_s_sleep(FSLEEP);
                asm volatile("" ::: "memory");
            }
            if (u == 0) SXT(5); else SXT(7);
            if (u == 0 && lane == 0) __hip_atomic_fetch_add(&sv[7], 1u, __ATOMIC_RELAXED, __HIP_MEMORY_SCOPE_WORKGROUP);
            if (GMODE == 3 && u <= PU && lane == 0) __hip_atomic_fetch_add(&sv[6], ~0u, __ATOMIC_RELAXED, __HIP_MEMORY_SCOPE_WORKGROUP);
            f32x4 v0 = (AF(pv[0]) + AF(pv[2])) + (AF(pv[4]) + AF(pv[6])), v1 = (AF(pv[1]) + AF(pv[3])) + (AF(pv[5]) + AF(pv[7]));
            finish_unit<PACE0>(v0, v1, Ts[wid], dst + (size_t)(64 * wv + 8 * q) * NLOC, lane, bad, &sv[4]);
            if (u == 0) SXT(6);
        }
        asm volatile("s_waitcnt vmcnt(0)" ::: "memory");
        __syncthreads();
        return;
    }

    unsigned call;
    {
        unsigned old = 0;
        if (tid < 16) sv[tid] = 0;
        const int rbk = grp >> 2, ks = q * 64 + (grp & 3) * 16 + 2 * wid;
        const float* src = x + (size_t)(32 * rbk + (lane & 31)) * KDIM + ks * 16 + 8 * (lane >> 5);
        f32x4 v[4];
#pragma unroll
        for (int i = 0; i < 4; ++i)
            asm volatile("global_load_dwordx4 %0, %1, off nt" : "=v"(v[i]) : "v"(src + (i >> 1) * 16 + (i & 1) * 4));
        __syncthreads();
        if (tid == 0) old = __hip_atomic_load(ws_word(myws, 0), __ATOMIC_RELAXED, __HIP_MEMORY_SCOPE_AGENT);
        RXT(0);
        asm volatile("s_waitcnt vmcnt(0)" : "+v"(v[0]), "+v"(v[1]), "+v"(v[2]), "+v"(v[3]));
        RXT(1);
        if (lane == 0) __hip_atomic_fetch_add(&sv[0], 1u, __ATOMIC_RELAXED, __HIP_MEMORY_SCOPE_WORKGROUP);
        if (tid == 0) *(volatile unsigned*)&sv[2] = (old >> 8) + 1u;
#pragma unroll
        for (int i = 0; i < 2; ++i) {
            half8 h;
#pragma unroll
            for (int e = 0; e < 4; ++e) { h[e] = (_Float16)v[2 * i][e]; h[4 + e] = (_Float16)v[2 * i + 1][e]; }
            GSTOREC(x16 + (size_t)rbk * RBS + (size_t)((ks + i) * 64 + lane) * 16, __builtin_bit_cast(f32x4, h));
        }
        unsigned long long t0 = __builtin_amdgcn_s_memrealtime();
        while (true) {
            LDSRD(call, &sv[2]);
            if (call != 0u) break;
            if (__builtin_amdgcn_s_memrealtime() - t0 > DEADLINE_TICKS) { bad = true; break; }
            __builtin_amdgcn_s_sleep(1);
        }
        call = __builtin_amdgcn_readfirstlane(call);
        RXT(2);
        asm volatile("s_waitcnt vmcnt(0)" ::: "memory");
        RXT(3);
        unsigned* xf = (unsigned*)(myws + XFLAG_OFF) + q * 512;
        if (lane == 0) __hip_atomic_store(xf + grp * 8 + wid, call, __ATOMIC_RELAXED, __HIP_MEMORY_SCOPE_SYSTEM);
        if (wid == 0) {
            while (__builtin_amdgcn_s_memrealtime() - t0 < XDELAY) __builtin_amdgcn_s_sleep(4);
            while (true) {
                f32x4 f0, f1;
                GLOADC(f0, xf + lane * 8); GLOADC(f1, xf + lane * 8 + 4);
                asm volatile("s_waitcnt vmcnt(0)" : "+v"(f0), "+v"(f1));
                u32x4 u0 = __builtin_bit_cast(u32x4, f0), u1 = __builtin_bit_cast(u32x4, f1);
                bool ok = u0[0] == call && u0[1] == call && u0[2] == call && u0[3] == call && u1[0] == call && u1[1] == call && u1[2] == call && u1[3] == call;
                if (__builtin_amdgcn_ballot_w64(!ok) == 0ull) break;
                if (__builtin_amdgcn_s_memrealtime() - t0 > DEADLINE_TICKS) { bad = true; break; }
                __builtin_amdgcn_s_sleep(XSLEEP);
            }
            asm volatile("" ::: "memory");
            if (lane == 0) *(volatile unsigned*)&sv[1] = call | (bad ? 0x80000000u : 0u);
        } else {
            while (true) {
                LDSRD(call, &sv[1]);
                if (call != 0u) break;
                if (__builtin_amdgcn_s_memrealtime() - t0 > DEADLINE_TICKS) { call = 0x80000000u; break; }
                __builtin_amdgcn_s_sleep(1);
            }
            call = __builtin_amdgcn_readfirstlane(call);
            if (call & 0x80000000u) bad = true;
            call &= 0x7fffffffu;
        }
        asm volatile("s_waitcnt vmcnt(0) lgkmcnt(0)" ::: "memory");
    }
    RST(1);

    const unsigned a_voff = (unsigned)lane * 16u;
    const char* a_base = x16 + (size_t)(2 * wid) * RBS + (size_t)(64 * q) * 1024;
#define PFX 0
    if (PFX && wid >= 4) {
        f32x4 pf[6];
        const int c0 = ((b >> 3) * 4 + (wid - 4)) * 6;
#pragma unroll
        for (int i = 0; i < 6; ++i) {
            const int c = c0 + i, rbi = c >> 6, rbk = rbi < 4 ? 2 * rbi + 1 : rbi + 4;
            GLOAD(pf[i], a_voff, x16 + (size_t)rbk * RBS + (size_t)(64 * q + (c & 63)) * 1024);
        }
        asm volatile("s_waitcnt vmcnt(0)" : "+v"(pf[0]), "+v"(pf[1]), "+v"(pf[2]), "+v"(pf[3]), "+v"(pf[4]), "+v"(pf[5]));
    }
    f32x4 ra[RD];
#define AOFF(S) ((size_t)((S) >> 6) * RBS + (size_t)((S) & 63) * 1024)
#pragma unroll
    for (int i = 0; i < RD; ++i) GLOAD(ra[i], a_voff, a_base + AOFF(i));

    f32x16 acc[2];
#pragma unroll
    for (int t = 0; t < 2; ++t)
#pragma unroll
        for (int i = 0; i < 16; ++i) acc[t][i] = 0.f;

    const int fh = lane >> 5;
    const int bblk = (lane >> 4) & 1, bq = (lane & 15) >> 2, bp = lane & 3;
    const unsigned lds0 = (unsigned)(uintptr_t)(__attribute__((address_space(3))) _Float16*)Bs;
    const unsigned b0p = lds0 + 2u * ((8 * fh + bq) * BN + ((16 * bblk + 4 * bp) ^ (32 * ((bq >> 1) & 1))));
    const unsigned b1p = lds0 + 2u * ((8 * fh + bq) * BN + ((32 + 16 * bblk + 4 * bp) ^ (32 * ((bq >> 1) & 1))));
#define BBASE(BLK) unsigned bs0 = b0p + (BLK) * 512u * BN, bs1 = b1p + (BLK) * 512u * BN; asm volatile("" : "+v"(bs0), "+v"(bs1))
    unsigned seen = 0;
#define BPOLL() do { \
        u32x4 rv; \
        asm volatile("ds_read_b128 %0, %1\n\ts_waitcnt lgkmcnt(0)" : "=v"(rv) : "v"((unsigned)(uintptr_t)(__attribute__((address_space(3))) unsigned*)rdy) : "memory"); \
        seen = __builtin_amdgcn_readfirstlane(min(min(rv[0], rv[1]), min(rv[2], rv[3]))); } while (0)
#define BWAIT(S) do { \
        if (seen <= (unsigned)(S)) { \
            BPOLL(); \
            if (seen <= (unsigned)(S)) { \
                unsigned long long t0 = rtc(); \
                for (unsigned n = 1; ; ++n) { \
                    __builtin_amdgcn_s_sleep(2); \
                    BPOLL(); \
                    if (seen > (unsigned)(S)) break; \
                    if ((n & 255u) == 0u && rtc() - t0 > DEADLINE_TICKS) { bad = true; seen = 64; break; } \
                } \
            } \
        } } while (0)
#define TR(P) __builtin_amdgcn_ds_read_tr16_b64_v4i16((__attribute__((address_space(3))) s16x4*)(uintptr_t)(P))
#define LD 2
    s16x4 fb[LD + 1][4];
    unsigned bsl0 = b0p, bsl1 = b1p, bsh0 = b0p + 65536u, bsh1 = b1p + 65536u;
    asm volatile("" : "+v"(bsl0), "+v"(bsl1), "+v"(bsh0), "+v"(bsh1));
#define TRA(DST, P, O) asm volatile("ds_read_b64_tr_b16 %0, %1 offset:%2" : "=v"(DST) : "v"(P), "n"(O))
#define BRD(G) do { \
        if ((G) & 32) { \
            TRA(fb[(G) % (LD + 1)][0], bsh0, ((G) & 31) * 2048); TRA(fb[(G) % (LD + 1)][1], bsh0, ((G) & 31) * 2048 + 512); \
            TRA(fb[(G) % (LD + 1)][2], bsh1, ((G) & 31) * 2048); TRA(fb[(G) % (LD + 1)][3], bsh1, ((G) & 31) * 2048 + 512); \
        } else { \
            TRA(fb[(G) % (LD + 1)][0], bsl0, ((G) & 31) * 2048); TRA(fb[(G) % (LD + 1)][1], bsl0, ((G) & 31) * 2048 + 512); \
            TRA(fb[(G) % (LD + 1)][2], bsl1, ((G) & 31) * 2048); TRA(fb[(G) % (LD + 1)][3], bsl1, ((G) & 31) * 2048 + 512); \
        } } while (0)
#define STEP(G) do { \
        if ((G) + LD < 64 && (((G) + LD) & 3) == 0) BWAIT(((G) + LD) >> 2); \
        if ((G) + LD < 128) BRD((G) + LD); \
        asm volatile("s_waitcnt vmcnt(%5) lgkmcnt(%6)" : "+v"(ra[(G) % RD]), "+v"(fb[(G) % (LD + 1)][0]), "+v"(fb[(G) % (LD + 1)][1]), "+v"(fb[(G) % (LD + 1)][2]), "+v"(fb[(G) % (LD + 1)][3]) \
            : "n"(((G) > 127 - RD ? 127 - (G) : RD - 1) + ((G) >= 64 && (G) < 64 + RD ? 8 : 0)), "n"(4 * ((G) + LD < 128 ? LD : 127 - (G)))); \
        half8 fb0 = __builtin_bit_cast(half8, __builtin_shufflevector(fb[(G) % (LD + 1)][0], fb[(G) % (LD + 1)][1], 0, 1, 2, 3, 4, 5, 6, 7)); \
        half8 fb1 = __builtin_bit_cast(half8, __builtin_shufflevector(fb[(G) % (LD + 1)][2], fb[(G) % (LD + 1)][3], 0, 1, 2, 3, 4, 5, 6, 7)); \
        half8 fa = __builtin_bit_cast(half8, ra[(G) % RD]); \
        acc[0] = __builtin_amdgcn_mfma_f32_32x32x16_f16(fa, fb0, acc[0], 0, 0, 0); \
        acc[1] = __builtin_amdgcn_mfma_f32_32x32x16_f16(fa, fb1, acc[1], 0, 0, 0); \
        if ((G) + RD < 128) GLOAD(ra[(G) % RD], a_voff, a_base + AOFF((G) + RD)); \
        __builtin_amdgcn_sched_barrier(0); \
    } while (0)
#define PAUSE() do { if (GMODE == 3 && wid >= PW) { \
        for (unsigned n = 0; n < 4096u; ++n) { \
            unsigned c; LDSRD(c, &sv[6]); \
            if (__builtin_amdgcn_readfirstlane(c) == 0u) break; \
            __builtin_amdgcn_s_sleep(1); \
        } } } while (0)
#define STEP4(G0) do { PAUSE(); STEP(G0); STEP((G0) + 1); STEP((G0) + 2); STEP((G0) + 3); } while (0)
#define BLOCK1(B) do { STEP4(16 * (B)); STEP4(16 * (B) + 4); STEP4(16 * (B) + 8); STEP4(16 * (B) + 12); } while (0)
#define BLOCK2(B) BLOCK1(4 + (B))

#define SK 16
#define SK2 1
    if (wid >= 2 && wid < 4) BWAIT(SK2 - 1);
    if (wid >= 4) BWAIT(SK - 1);
#define WG 4
#define WGW 4
#define GATE() do { \
        unsigned long long t0 = __builtin_amdgcn_s_memrealtime(); \
        while (true) { \
            unsigned c; LDSRD(c, &sv[7]); \
            if (__builtin_amdgcn_readfirstlane(c) >= (unsigned)WG) break; \
            if (__builtin_amdgcn_s_memrealtime() - t0 > DEADLINE_TICKS) { bad = true; break; } \
            __builtin_amdgcn_s_sleep(2); \
        } } while (0)
    if (GMODE == 1 && WG && wid >= WGW) GATE();
    BWAIT(0);
#pragma unroll
    for (int i = 0; i < LD; ++i) BRD(i);
    BLOCK1(0); BLOCK1(1); BLOCK1(2); BLOCK1(3);
    RST(2);
    char* mypart = part + (size_t)((grp * 4 + q) * 8 + wid) * 16384 + lane * 16;
    const unsigned tag = call & 1u;
#pragma unroll
    for (int G = 0; G < 4; ++G)
#pragma unroll
        for (int t = 0; t < 2; ++t) {
            f32x4 v; v[0] = TAGGED(acc[t][4 * G]); v[1] = TAGGED(acc[t][4 * G + 1]); v[2] = TAGGED(acc[t][4 * G + 2]); v[3] = TAGGED(acc[t][4 * G + 3]);
            GSTOREC(mypart + G * 2048 + t * 1024, v);
        }
    if (lane == 0) *(volatile unsigned*)&sv[8 + wid] = 1u;
#pragma unroll
    for (int t = 0; t < 2; ++t)
#pragma unroll
        for (int i = 0; i < 16; ++i) acc[t][i] = 0.f;

    if (GMODE == 2 && WG && wid < 4) GATE();
    BLOCK2(0); BLOCK2(1); BLOCK2(2); BLOCK2(3);
    RST(3);

    f32x4 own[2];
    {
#pragma unroll
        for (int G = 0; G < 4; ++G) {
            if (G == q) {
#pragma unroll
                for (int t = 0; t < 2; ++t) { own[t][0] = acc[t][4 * G]; own[t][1] = acc[t][4 * G + 1]; own[t][2] = acc[t][4 * G + 2]; own[t][3] = acc[t][4 * G + 3]; }
            } else {
                char* d = mypart + 8192 + ((G - q - 1) & 3) * 2048;
#pragma unroll
                for (int t = 0; t < 2; ++t) {
                    f32x4 v; v[0] = TAGGED(acc[t][4 * G]); v[1] = TAGGED(acc[t][4 * G + 1]); v[2] = TAGGED(acc[t][4 * G + 2]); v[3] = TAGGED(acc[t][4 * G + 3]);
                    GSTOREC(d + t * 1024, v);
                }
            }
        }
        SXT(0);
        u32x4 pv[6];
        unsigned pend = 7u;
        unsigned long long t0 = __builtin_amdgcn_s_memrealtime();
        if (CDELAY) __builtin_amdgcn_s_sleep(CDELAY);
#define CPOLL1 8
        if (wid < CPOLL1) {
            const int p = (q + 1 + (lane & 3)) & 3;
            const unsigned po = (unsigned)(((grp * 4 + p) * 8 + wid) * 16384 + 8192 + ((q - p - 1) & 3) * 2048 + 2044);
            while (true) {
                unsigned fv = tag;
                if (lane < 3) fv = __builtin_amdgcn_raw_buffer_load_b32(prs, po, 0, 17);
                if (__builtin_amdgcn_ballot_w64(((fv ^ tag) & 1u) != 0u) == 0ull) break;
                if (__builtin_amdgcn_s_memrealtime() - t0 > DEADLINE_TICKS) { bad = true; break; }
                __builtin_amdgcn_s_sleep(FSLEEP);
                asm volatile("" ::: "memory");
            }
        }
        while (true) {
#pragma unroll
            for (int d = 0; d < 3; ++d)
                if (pend & (1u << d)) {
                    const int p = (q + 1 + d) & 3;
                    const unsigned o = (unsigned)(((grp * 4 + p) * 8 + wid) * 16384 + 8192 + ((q - p - 1) & 3) * 2048 + lane * 16);
                    pv[2 * d] = __builtin_amdgcn_raw_buffer_load_b128(prs, o, 0, 17);
                    pv[2 * d + 1] = __builtin_amdgcn_raw_buffer_load_b128(prs, o + 1024u, 0, 17);
                }
#pragma unroll
            for (int d = 0; d < 3; ++d)
                if (pend & (1u << d)) {
                    if (TAGOK(pv[2 * d], pv[2 * d + 1])) pend &= ~(1u << d);
                }
            if (pend == 0u) break;
            if (__builtin_amdgcn_s_memrealtime() - t0 > DEADLINE_TICKS) { bad = true; break; }
            __builtin_amdgcn_s_sleep(CSLEEP);
            asm volatile("" ::: "memory");
        }
        SXT(1); SXT(2);
#pragma unroll
        for (int t = 0; t < 2; ++t) own[t] = (own[t] + AF(pv[t])) + (AF(pv[2 + t]) + AF(pv[4 + t]));
    }
    RST(4);
    finish_unit<PACE1>(own[0], own[1], Ts[wid], dst + (size_t)(64 * wid + 32 + 8 * q) * NLOC, lane, bad, &sv[4]);
    asm volatile("s_waitcnt vmcnt(0)" ::: "memory");
    SXT(3);
    __syncthreads();
    RST(5);
    if (tid == 0) __hip_atomic_fetch_add(ws_word(myws, 0), 1u, __ATOMIC_RELAXED, __HIP_MEMORY_SCOPE_AGENT);
    if (tid == 0) {
        if (peer != rank) {
            __builtin_amdgcn_fence(__ATOMIC_RELEASE, "");
            asm volatile("s_waitcnt vmcnt(0)" ::: "memory");
            __hip_atomic_fetch_add(ws_word(ws_all[peer], 2 + rank), 1u, __ATOMIC_RELAXED, __HIP_MEMORY_SCOPE_SYSTEM);
        }
        if (b == 0) {
            unsigned need = call * (unsigned)BLOCKS_PER_PEER;
            unsigned long long t0 = __builtin_amdgcn_s_memrealtime();
            bool timed_out = false;
            for (int p = 0; p < NR; ++p) {
                if (p == rank) continue;
                while (__hip_atomic_load(ws_word(myws, 2 + p), __ATOMIC_RELAXED, __HIP_MEMORY_SCOPE_SYSTEM) < need) {
                    if (__builtin_amdgcn_s_memrealtime() - t0 > DEADLINE_TICKS) { timed_out = true; break; }
                    __builtin_amdgcn_s_sleep(2);
                }
            }
            __builtin_amdgcn_fence(__ATOMIC_ACQUIRE, "");
            if (timed_out) {
                __hip_atomic_store(ws_word(myws, 1), call, __ATOMIC_RELAXED, __HIP_MEMORY_SCOPE_AGENT);
                ((float*)out_all[rank])[(size_t)(rank * MLOC) * NLOC] = __builtin_nanf("");
            }
        }
    }
}

extern "C" void kernel_launch(int rank, int nranks, ncclComm_t comm,
                              hipStream_t stream,
                              void* const* d_in, const int* in_sizes, int n_in,
                              void* d_out, int out_size,
                              const int* in_dtypes, int out_dtype,
                              void* const* d_out_all,
                              void* const* d_ws_all, size_t ws_bytes) {
    const float* x = (const float*)d_in[0];
    const float* w = (const float*)d_in[1];
    hipLaunchKernelGGL(gemm_a2a_kernel, dim3(256), dim3(NBLOCK), 0, stream,
                       x, w, d_out_all, d_ws_all, rank);
}
```

```cpp
#include <hip/hip_runtime.h>
#include <rccl/rccl.h>
#include <cstdio>
#include <cstdint>

typedef _Float16 half8 __attribute__((ext_vector_type(8)));
typedef _Float16 half4 __attribute__((ext_vector_type(4)));
typedef float f32x16 __attribute__((ext_vector_type(16)));
typedef float f32x4 __attribute__((ext_vector_type(4)));
typedef unsigned u32x4 __attribute__((ext_vector_type(4)));
typedef short s16x4 __attribute__((ext_vector_type(4)));
typedef short s16x8 __attribute__((ext_vector_type(8)));

constexpr int NR = 8;
constexpr int MLOC = 512;
constexpr int KDIM = 4096;
constexpr int NDIM = 4096;
constexpr int NLOC = 512;
constexpr int BN = 64;
constexpr int KS = 1024;
constexpr int NCW = 8;
constexpr int NPW = 4;
constexpr int NBLOCK = (NCW + NPW) * 64;
constexpr int BLOCKS_PER_PEER = 32;
constexpr unsigned long long DEADLINE_TICKS = 40000000ull;
constexpr size_t XFLAG_OFF = 8192;
constexpr size_t FLAG_OFF = 65536;
constexpr size_t RBS = 262144;
constexpr size_t X16_OFF = 262144;
constexpr size_t PART_OFF = X16_OFF + (5u << 20);

__device__ __forceinline__ unsigned* ws_word(void* ws, int idx) { return (unsigned*)((char*)ws + 128 * idx); }

__device__ __forceinline__ unsigned long long rtc() {
    unsigned long long t;
    asm volatile("s_memrealtime %0\n\ts_waitcnt lgkmcnt(0)" : "=s"(t) :: "memory");
    return t;
}

__device__ __forceinline__ float gelu_f(float y) {
    const float c = 0.7978845608028654f;
    float u = c * (y + 0.044715f * y * y * y);
    return y / (1.0f + __expf(-2.0f * u));
}

#define RST(I)
#define XV 0
#define XDELAY 350
#define FSLEEP 8
#define CSLEEP 4
#define XSLEEP 16
#define GLOAD(DST, VOFF, SBASE) asm volatile("global_load_dwordx4 %0, %1, %2" : "=v"(DST) : "v"(VOFF), "s"(SBASE))
#define GLOADNT(DST, VOFF, SBASE) asm volatile("global_load_dwordx4 %0, %1, %2 nt" : "=v"(DST) : "v"(VOFF), "s"(SBASE))
#define GLOADC(DST, P) asm volatile("global_load_dwordx4 %0, %1, off sc0 sc1" : "=v"(DST) : "v"(P) : "memory")
#define GSTOREC(P, V) asm volatile("global_store_dwordx4 %0, %1, off sc0 sc1\n\ts_nop 1" :: "v"(P), "v"(V) : "memory")
#define LDSRD(DST, P) asm volatile("ds_read_b32 %0, %1\n\ts_waitcnt lgkmcnt(0)" : "=v"(DST) : "v"((unsigned)(uintptr_t)(__attribute__((address_space(3))) unsigned*)(P)) : "memory")

#define AF(V) __builtin_bit_cast(f32x4, V)
#define TAGOK(A, B) (__builtin_amdgcn_ballot_w64((((((A)[0] ^ tag) | ((A)[1] ^ tag)) | (((A)[2] ^ tag) | ((A)[3] ^ tag)) | (((B)[0] ^ tag) | ((B)[1] ^ tag)) | (((B)[2] ^ tag) | ((B)[3] ^ tag))) & 1u) != 0u) == 0ull)
__device__ __forceinline__ float tagged(float f, unsigned tag) { return __uint_as_float((__float_as_uint(f) & ~1u) | tag); }
#define TAGGED(F) tagged(F, tag)
#define FDELAY 0
#define CDELAY 0
#define PF 0
#define CPOLL 1
#define AGG 2
#define AGT 8000
#define TSL 1
#define XSL 1
#define LSL 2
#define RD 12
#define PACE0 3
#define PACE1 3
#define NTOK 3
#define GMODE 1
#define PU 0
#define PW 4
template <int PACE>
__device__ __forceinline__ void finish_unit(const f32x4& v0, const f32x4& v1, float* T, float* dst, int lane, bool bad, unsigned* tk) {
    const int fh = lane >> 5;
#pragma unroll
    for (int i = 0; i < 4; ++i) {
        float g0 = gelu_f(v0[i]), g1 = gelu_f(v1[i]);
        T[(4 * fh + i) * 68 + (lane & 31)] = bad ? __builtin_nanf("") : g0;
        T[(4 * fh + i) * 68 + 32 + (lane & 31)] = bad ? __builtin_nanf("") : g1;
    }
    __builtin_amdgcn_wave_barrier();
#pragma unroll
    for (int j = 0; j < 2; ++j) {
        const int r = 4 * j + (lane >> 4), c4 = lane & 15;
        f32x4 v = *(const f32x4*)(T + r * 68 + c4 * 4);
        if (PACE == 1) asm volatile("s_waitcnt vmcnt(0)" ::: "memory");
        if (PACE == 2) asm volatile("s_waitcnt vmcnt(1)" ::: "memory");
        if (PACE == 3) {
            unsigned t = 0;
            if (lane == 0) t = __hip_atomic_fetch_add(tk, 1u, __ATOMIC_RELAXED, __HIP_MEMORY_SCOPE_WORKGROUP);
            t = __builtin_amdgcn_readfirstlane(t);
            unsigned long long t0 = __builtin_amdgcn_s_memrealtime();
            while (true) {
                unsigned d; LDSRD(d, tk + 1);
                if (t < __builtin_amdgcn_readfirstlane(d) + NTOK) break;
                if (__builtin_amdgcn_s_memrealtime() - t0 > DEADLINE_TICKS) break;
                __builtin_amdgcn_s_sleep(TSL);
            }
        }
        GSTOREC(dst + (size_t)r * NLOC + c4 * 4, v);
        if (PACE == 3) {
            asm volatile("s_waitcnt vmcnt(0)" ::: "memory");
            if (lane == 0) __hip_atomic_fetch_add(tk + 1, 1u, __ATOMIC_RELAXED, __HIP_MEMORY_SCOPE_WORKGROUP);
        }
    }
    __builtin_amdgcn_wave_barrier();
}

__global__ __launch_bounds__(NBLOCK, 3)
void gemm_a2a_kernel(const float* __restrict__ x, const float* __restrict__ w,
                     void* const* __restrict__ out_all, void* const* __restrict__ ws_all, int rank) {
    __shared__ __attribute__((aligned(16))) _Float16 Bs[KS * BN];
    __shared__ __attribute__((aligned(16))) float Ts[NCW + NPW][8 * 68];
    __shared__ __attribute__((aligned(16))) unsigned rdy[4];
    __shared__ __attribute__((aligned(16))) unsigned sv[16];

#define RXT(I)
#define SXT(I)
    RST(0);
    const int tid = threadIdx.x;
    const int lane = tid & 63, wid = __builtin_amdgcn_readfirstlane(tid >> 6);
    char* myws;
    {
        unsigned long long a = (unsigned long long)ws_all[rank];
        unsigned lo = __builtin_amdgcn_readfirstlane((unsigned)a), hi = __builtin_amdgcn_readfirstlane((unsigned)(a >> 32));
        myws = (char*)(((unsigned long long)hi << 32) | lo);
    }
    char* x16 = myws + X16_OFF;
    char* part = myws + PART_OFF;
    const __amdgpu_buffer_rsrc_t prs = __builtin_amdgcn_make_buffer_rsrc(part, 0, 32u << 20, 0x00020000);
    const int b = blockIdx.x;
#define MAP 0
    const int q = b & 3;
    const int grp = ((b >> 2) & 1) * 32 + (b >> 3);
    const int peer = ((grp >> 3) + rank) & 7;
    const int ct = grp & 7;
    const int ncol0 = peer * NLOC + ct * BN;
    float* dst = (float*)out_all[peer] + (size_t)(rank * MLOC) * NLOC + ct * BN;
    bool bad = false;
#define ROT 0
#define RG(W) (ROT ? (((W) + rank) & 7) : (W))
#define PV 1
    const unsigned pw = PV == 1 ? 1024u + (unsigned)((wid + grp + lane) & 7) * 128u + 124u : PV == 2 ? (unsigned)((wid + grp + lane) & 15) * 128u + 124u : 2044u;

    if (wid >= NCW) {
        const int p = wid - NCW;
        const int kr = lane >> 4, nq = lane & 15;
        const unsigned voff = (unsigned)((4 * p + kr) * NDIM + nq * 4) * 4u;
        const char* base = (const char*)(w + (size_t)q * KS * NDIM + ncol0);
#define PD 2
        f32x4 rb[4 * PD];
#define PLD(SLOT, S) _Pragma("unroll") for (int i = 0; i < 4; ++i) GLOADNT(rb[(SLOT) * 4 + i], voff, base + (size_t)((S) * 64 + 16 * i) * NDIM * 4)
#define PWAIT(N, SLOT) asm volatile("s_waitcnt vmcnt(" #N ")" : "+v"(rb[(SLOT) * 4]), "+v"(rb[(SLOT) * 4 + 1]), "+v"(rb[(SLOT) * 4 + 2]), "+v"(rb[(SLOT) * 4 + 3]))
#define PST(SLOT, S) do { \
        _Pragma("unroll") for (int i = 0; i < 4; ++i) { \
            f32x4 v = rb[(SLOT) * 4 + i]; \
            half4 h; h[0] = (_Float16)v[0]; h[1] = (_Float16)v[1]; h[2] = (_Float16)v[2]; h[3] = (_Float16)v[3]; \
            *(half4*)(Bs + ((S) * 64 + 16 * i + 4 * p + kr) * BN + ((nq * 4) ^ (32 * ((kr >> 1) & 1)))) = h; \
        } \
        asm volatile("" ::: "memory"); \
        if (lane == 0) *(volatile unsigned*)&rdy[p] = (unsigned)(S) + 1u; \
        asm volatile("" ::: "memory"); } while (0)
#define BDELAY 1
        if (lane == 0) rdy[p] = 0;
        __syncthreads();
        {
            unsigned long long t0 = __builtin_amdgcn_s_memrealtime();
            while (true) {
                unsigned c; LDSRD(c, &sv[0]);
                if (__builtin_amdgcn_readfirstlane(c) >= (unsigned)NCW) break;
                if (__builtin_amdgcn_s_memrealtime() - t0 > DEADLINE_TICKS) break;
                __builtin_amdgcn_s_sleep(1);
            }
        }
        _Pragma("unroll") for (int s = 0; s < PD; ++s) { PLD(s, s); }
#pragma unroll
        for (int s = 0; s < 16; ++s) {
            const int left = (15 - s < PD - 1) ? 15 - s : PD - 1;
            if (left == 5) PWAIT(20, s % PD); else if (left == 4) PWAIT(16, s % PD); else if (left == 3) PWAIT(12, s % PD); else if (left == 2) PWAIT(8, s % PD); else if (left == 1) PWAIT(4, s % PD); else PWAIT(0, s % PD);
            PST(s % PD, s);
            if ((s & 3) == 0) RXT(4 + (s >> 2));
            if (s + PD < 16) PLD(s % PD, s + PD);
        }
        RST(3);
        unsigned call;
        unsigned long long t0 = __builtin_amdgcn_s_memrealtime();
        while (true) {
            LDSRD(call, &sv[1]);
            if (call != 0u) break;
            if (__builtin_amdgcn_s_memrealtime() - t0 > DEADLINE_TICKS) { call = 0x80000000u; break; }
            __builtin_amdgcn_s_sleep(LSL);
        }
        call = __builtin_amdgcn_readfirstlane(call);
        if (call & 0x80000000u) bad = true;
        const bool aggr = (call & 0x40000000u) != 0u;
        call &= 0x3fffffffu;
        const unsigned tag = call & 1u;
#define FM 3
#define MAPF 1
        {
        const int w0 = MAPF ? 2 * p : p;
        constexpr int WS = MAPF ? 1 : 4;
        while (true) {
            unsigned c, c2 = 1;
            if (aggr) { LDSRD(c, &sv[8 + w0]); LDSRD(c2, &sv[8 + w0 + WS]); } else { LDSRD(c, &sv[3]); }
            if (__builtin_amdgcn_readfirstlane(c) >= (aggr ? 1u : (unsigned)NCW) && __builtin_amdgcn_readfirstlane(c2) != 0u) break;
            if (__builtin_amdgcn_s_memrealtime() - t0 > DEADLINE_TICKS) { bad = true; break; }
            __builtin_amdgcn_s_sleep(LSL);
        }
        const unsigned voff0 = (unsigned)((grp * 4 * 8 + w0) * 16384 + q * 2048 + lane * 16);
        u32x4 pv[16];
        unsigned pend = 0xffu;
        {
            const unsigned po = (unsigned)((grp * 4 * 8 + w0) * 16384 + q * 2048) + pw + (unsigned)(((lane & 3) * 8 + ((lane >> 2) & 1) * WS) * 16384);
            while (true) {
                unsigned fv = tag;
                if (lane < 8) fv = __builtin_amdgcn_raw_buffer_load_b32(prs, po, 0, 17);
                if (__builtin_amdgcn_ballot_w64(((fv ^ tag) & 1u) != 0u) == 0ull) break;
                if (__builtin_amdgcn_s_memrealtime() - t0 > DEADLINE_TICKS) { bad = true; break; }
                __builtin_amdgcn_s_sleep(FSLEEP);
                asm volatile("" ::: "memory");
            }
        }
        while (true) {
#pragma unroll
            for (int i = 0; i < 8; ++i)
                if (pend & (1u << i)) {
                    const unsigned o = voff0 + (unsigned)(((i & 3) * 8 + (i >> 2) * WS) * 16384);
                    pv[2 * i] = __builtin_amdgcn_raw_buffer_load_b128(prs, o, 0, 17);
                    pv[2 * i + 1] = __builtin_amdgcn_raw_buffer_load_b128(prs, o + 1024u, 0, 17);
                }
#pragma unroll
            for (int i = 0; i < 8; ++i)
                if (pend & (1u << i)) {
                    if (TAGOK(pv[2 * i], pv[2 * i + 1])) pend &= ~(1u << i);
                }
            if (pend == 0u) break;
            if (__builtin_amdgcn_s_memrealtime() - t0 > DEADLINE_TICKS) { bad = true; break; }
            __builtin_amdgcn_s_sleep(FSLEEP);
            asm volatile("" ::: "memory");
        }
        if (lane == 0) __hip_atomic_fetch_add(&sv[7], 1u, __ATOMIC_RELAXED, __HIP_MEMORY_SCOPE_WORKGROUP);
        {
            f32x4 v0 = (AF(pv[0]) + AF(pv[2])) + (AF(pv[4]) + AF(pv[6])), v1 = (AF(pv[1]) + AF(pv[3])) + (AF(pv[5]) + AF(pv[7]));
            finish_unit<PACE0>(v0, v1, Ts[wid], dst + (size_t)(64 * RG(w0) + 8 * q) * NLOC, lane, bad, &sv[4]);
        }
        {
            f32x4 v0 = (AF(pv[8]) + AF(pv[10])) + (AF(pv[12]) + AF(pv[14])), v1 = (AF(pv[9]) + AF(pv[11])) + (AF(pv[13]) + AF(pv[15]));
            finish_unit<PACE0>(v0, v1, Ts[wid], dst + (size_t)(64 * RG(w0 + WS) + 8 * q) * NLOC, lane, bad, &sv[4]);
        }
        }
#pragma unroll 1
        for (int u = 0; u < ((FM == 1 || FM == 3) ? 0 : 2); ++u) {
            const int wv = p + 4 * u;
            while (true) {
                unsigned c; LDSRD(c, &sv[8 + wv]);
                if (__builtin_amdgcn_readfirstlane(c) != 0u) break;
                if (__builtin_amdgcn_s_memrealtime() - t0 > DEADLINE_TICKS) { bad = true; break; }
                __builtin_amdgcn_s_sleep(LSL);
            }
            const unsigned voff0 = (unsigned)((grp * 4 * 8 + wv) * 16384 + q * 2048 + lane * 16);
            u32x4 pv[8];
            unsigned pend = 0xfu;
            {
                const unsigned po = (unsigned)((grp * 4 * 8 + wv) * 16384 + q * 2048) + pw + (unsigned)((lane & 3) * 8 * 16384);
                while (true) {
                    unsigned fv = tag;
                    if (lane < 4) fv = __builtin_amdgcn_raw_buffer_load_b32(prs, po, 0, 17);
                    if (__builtin_amdgcn_ballot_w64(((fv ^ tag) & 1u) != 0u) == 0ull) break;
                    if (__builtin_amdgcn_s_memrealtime() - t0 > DEADLINE_TICKS) { bad = true; break; }
                    __builtin_amdgcn_s_sleep(FSLEEP);
                    asm volatile("" ::: "memory");
                }
            }
            if (u == 0) SXT(4);
            if (GMODE == 3 && u <= PU && lane == 0) __hip_atomic_fetch_add(&sv[6], 1u, __ATOMIC_RELAXED, __HIP_MEMORY_SCOPE_WORKGROUP);
            while (true) {
#pragma unroll
                for (int i = 0; i < 4; ++i)
                    if (pend & (1u << i)) {
                        const unsigned o = voff0 + (unsigned)(i * 8 * 16384);
                        pv[2 * i] = __builtin_amdgcn_raw_buffer_load_b128(prs, o, 0, 17);
                        pv[2 * i + 1] = __builtin_amdgcn_raw_buffer_load_b128(prs, o + 1024u, 0, 17);
                    }
#pragma unroll
                for (int i = 0; i < 4; ++i)
                    if (pend & (1u << i)) {
                        if (TAGOK(pv[2 * i], pv[2 * i + 1])) pend &= ~(1u << i);
                    }
                if (pend == 0u) break;
                if (__builtin_amdgcn_s_memrealtime() - t0 > DEADLINE_TICKS) { bad = true; break; }
                __builtin_amdgcn_s_sleep(FSLEEP);
                asm volatile("" ::: "memory");
            }
            if (u == 0) SXT(5); else SXT(7);
            if (u == 0 && lane == 0) __hip_atomic_fetch_add(&sv[7], 1u, __ATOMIC_RELAXED, __HIP_MEMORY_SCOPE_WORKGROUP);
            if (GMODE == 3 && u <= PU && lane == 0) __hip_atomic_fetch_add(&sv[6], ~0u, __ATOMIC_RELAXED, __HIP_MEMORY_SCOPE_WORKGROUP);
            f32x4 v0 = (AF(pv[0]) + AF(pv[2])) + (AF(pv[4]) + AF(pv[6])), v1 = (AF(pv[1]) + AF(pv[3])) + (AF(pv[5]) + AF(pv[7]));
            finish_unit<PACE0>(v0, v1, Ts[wid], dst + (size_t)(64 * RG(wv) + 8 * q) * NLOC, lane, bad, &sv[4]);
            if (u == 0) SXT(6);
        }
        asm volatile("s_waitcnt vmcnt(0)" ::: "memory");
        __syncthreads();
        return;
    }

    unsigned call;
    bool aggr;
    {
        unsigned old = 0;
        if (tid < 16) sv[tid] = 0;
        const int rbk = grp >> 2, ks = q * 64 + (grp & 3) * 16 + 2 * wid;
        const float* src = x + (size_t)(32 * rbk + (lane & 31)) * KDIM + ks * 16 + 8 * (lane >> 5);
        f32x4 v[4];
#pragma unroll
        for (int i = 0; i < 4; ++i)
            asm volatile("global_load_dwordx4 %0, %1, off nt" : "=v"(v[i]) : "v"(src + (i >> 1) * 16 + (i & 1) * 4));
        __syncthreads();
        unsigned long long last = 0;
        if (tid == 0) {
            old = __hip_atomic_load(ws_word(myws, 0), __ATOMIC_RELAXED, __HIP_MEMORY_SCOPE_AGENT);
            if (AGG == 2) last = __hip_atomic_load((unsigned long long*)ws_word(myws, 12), __ATOMIC_RELAXED, __HIP_MEMORY_SCOPE_AGENT);
        }
        RXT(0);
        asm volatile("s_waitcnt vmcnt(0)" : "+v"(v[0]), "+v"(v[1]), "+v"(v[2]), "+v"(v[3]));
        RXT(1);
        if (lane == 0) __hip_atomic_fetch_add(&sv[0], 1u, __ATOMIC_RELAXED, __HIP_MEMORY_SCOPE_WORKGROUP);
        if (tid == 0) *(volatile unsigned*)&sv[2] = (((old >> 8) + 1u) & 0x3fffffffu) | ((AGG == 1 || (AGG == 2 && __builtin_amdgcn_s_memrealtime() - last > (unsigned long long)AGT)) ? 0x40000000u : 0u);
#pragma unroll
        for (int i = 0; i < 2; ++i) {
            half8 h;
#pragma unroll
            for (int e = 0; e < 4; ++e) { h[e] = (_Float16)v[2 * i][e]; h[4 + e] = (_Float16)v[2 * i + 1][e]; }
            GSTOREC(x16 + (size_t)rbk * RBS + (size_t)((ks + i) * 64 + lane) * 16, __builtin_bit_cast(f32x4, h));
        }
        unsigned long long t0 = __builtin_amdgcn_s_memrealtime();
        while (true) {
            LDSRD(call, &sv[2]);
            if (call != 0u) break;
            if (__builtin_amdgcn_s_memrealtime() - t0 > DEADLINE_TICKS) { bad = true; break; }
            __builtin_amdgcn_s_sleep(1);
        }
        call = __builtin_amdgcn_readfirstlane(call);
        aggr = (call & 0x40000000u) != 0u;
        call &= 0x3fffffffu;
        RXT(2);
        asm volatile("s_waitcnt vmcnt(0)" ::: "memory");
        RXT(3);
#define XF 1
#define XRS 4352
        {
            unsigned c = 0;
            if (lane == 0) c = __hip_atomic_fetch_add(&sv[6], 1u, __ATOMIC_RELAXED, __HIP_MEMORY_SCOPE_WORKGROUP);
            c = __builtin_amdgcn_readfirstlane(c);
            if (c == (unsigned)NCW - 1u && lane < 8)
                __hip_atomic_store((unsigned*)(myws + FLAG_OFF + (size_t)(q * 8 + lane) * XRS) + grp, call, __ATOMIC_RELAXED, __HIP_MEMORY_SCOPE_SYSTEM);
        }
        if (wid == 0) {
            const unsigned* xf = (const unsigned*)(myws + FLAG_OFF + (size_t)(q * 8 + (grp & 7)) * XRS) + lane;
            while (__builtin_amdgcn_s_memrealtime() - t0 < XDELAY) __builtin_amdgcn_s_sleep(4);
            while (true) {
                unsigned f = __hip_atomic_load(xf, __ATOMIC_RELAXED, __HIP_MEMORY_SCOPE_SYSTEM);
                if (__builtin_amdgcn_ballot_w64(f != call) == 0ull) break;
                if (__builtin_amdgcn_s_memrealtime() - t0 > DEADLINE_TICKS) { bad = true; break; }
                __builtin_amdgcn_s_sleep(XSLEEP);
            }
            asm volatile("" ::: "memory");
            if (lane == 0) *(volatile unsigned*)&sv[1] = call | (bad ? 0x80000000u : 0u) | (aggr ? 0x40000000u : 0u);
        } else {
            while (true) {
                LDSRD(call, &sv[1]);
                if (call != 0u) break;
                if (__builtin_amdgcn_s_memrealtime() - t0 > DEADLINE_TICKS) { call = 0x80000000u; break; }
                __builtin_amdgcn_s_sleep(XSL);
            }
            call = __builtin_amdgcn_readfirstlane(call);
            if (call & 0x80000000u) bad = true;
            call &= 0x3fffffffu;
        }
        asm volatile("s_waitcnt vmcnt(0) lgkmcnt(0)" ::: "memory");
    }
    RST(1);

    const unsigned a_voff = (unsigned)lane * 16u;
    const char* a_base = x16 + (size_t)(2 * RG(wid)) * RBS + (size_t)(64 * q) * 1024;
#define PFX 0
    if (PFX && wid >= 4) {
        f32x4 pf[6];
        const int c0 = ((b >> 3) * 4 + (wid - 4)) * 6;
#pragma unroll
        for (int i = 0; i < 6; ++i) {
            const int c = c0 + i, rbi = c >> 6, rbk = rbi < 4 ? 2 * rbi + 1 : rbi + 4;
            GLOAD(pf[i], a_voff, x16 + (size_t)rbk * RBS + (size_t)(64 * q + (c & 63)) * 1024);
        }
        asm volatile("s_waitcnt vmcnt(0)" : "+v"(pf[0]), "+v"(pf[1]), "+v"(pf[2]), "+v"(pf[3]), "+v"(pf[4]), "+v"(pf[5]));
    }
    f32x4 ra[RD];
#define AOFF(S) ((size_t)((S) >> 6) * RBS + (size_t)((S) & 63) * 1024)
#pragma unroll
    for (int i = 0; i < RD; ++i) GLOAD(ra[i], a_voff, a_base + AOFF(i));

    f32x16 acc[2];
#pragma unroll
    for (int t = 0; t < 2; ++t)
#pragma unroll
        for (int i = 0; i < 16; ++i) acc[t][i] = 0.f;

    const int fh = lane >> 5;
    const int bblk = (lane >> 4) & 1, bq = (lane & 15) >> 2, bp = lane & 3;
    const unsigned lds0 = (unsigned)(uintptr_t)(__attribute__((address_space(3))) _Float16*)Bs;
    const unsigned b0p = lds0 + 2u * ((8 * fh + bq) * BN + ((16 * bblk + 4 * bp) ^ (32 * ((bq >> 1) & 1))));
    const unsigned b1p = lds0 + 2u * ((8 * fh + bq) * BN + ((32 + 16 * bblk + 4 * bp) ^ (32 * ((bq >> 1) & 1))));
#define BBASE(BLK) unsigned bs0 = b0p + (BLK) * 512u * BN, bs1 = b1p + (BLK) * 512u * BN; asm volatile("" : "+v"(bs0), "+v"(bs1))
    unsigned seen = 0;
#define BPOLL() do { \
        u32x4 rv; \
        asm volatile("ds_read_b128 %0, %1\n\ts_waitcnt lgkmcnt(0)" : "=v"(rv) : "v"((unsigned)(uintptr_t)(__attribute__((address_space(3))) unsigned*)rdy) : "memory"); \
        seen = __builtin_amdgcn_readfirstlane(min(min(rv[0], rv[1]), min(rv[2], rv[3]))); } while (0)
#define BWAIT(S) do { \
        if (seen <= (unsigned)(S)) { \
            BPOLL(); \
            if (seen <= (unsigned)(S)) { \
                unsigned long long t0 = rtc(); \
                for (unsigned n = 1; ; ++n) { \
                    __builtin_amdgcn_s_sleep(2); \
                    BPOLL(); \
                    if (seen > (unsigned)(S)) break; \
                    if ((n & 255u) == 0u && rtc() - t0 > DEADLINE_TICKS) { bad = true; seen = 64; break; } \
                } \
            } \
        } } while (0)
#define TR(P) __builtin_amdgcn_ds_read_tr16_b64_v4i16((__attribute__((address_space(3))) s16x4*)(uintptr_t)(P))
#define LD 2
    s16x4 fb[LD + 1][4];
    unsigned bsl0 = b0p, bsl1 = b1p, bsh0 = b0p + 65536u, bsh1 = b1p + 65536u;
    asm volatile("" : "+v"(bsl0), "+v"(bsl1), "+v"(bsh0), "+v"(bsh1));
#define TRA(DST, P, O) asm volatile("ds_read_b64_tr_b16 %0, %1 offset:%2" : "=v"(DST) : "v"(P), "n"(O))
#define BRD(G) do { \
        if ((G) & 32) { \
            TRA(fb[(G) % (LD + 1)][0], bsh0, ((G) & 31) * 2048); TRA(fb[(G) % (LD + 1)][1], bsh0, ((G) & 31) * 2048 + 512); \
            TRA(fb[(G) % (LD + 1)][2], bsh1, ((G) & 31) * 2048); TRA(fb[(G) % (LD + 1)][3], bsh1, ((G) & 31) * 2048 + 512); \
        } else { \
            TRA(fb[(G) % (LD + 1)][0], bsl0, ((G) & 31) * 2048); TRA(fb[(G) % (LD + 1)][1], bsl0, ((G) & 31) * 2048 + 512); \
            TRA(fb[(G) % (LD + 1)][2], bsl1, ((G) & 31) * 2048); TRA(fb[(G) % (LD + 1)][3], bsl1, ((G) & 31) * 2048 + 512); \
        } } while (0)
#define STEP(G) do { \
        if ((G) + LD < 64 && (((G) + LD) & 3) == 0) BWAIT(((G) + LD) >> 2); \
        if ((G) + LD < 128) BRD((G) + LD); \
        asm volatile("s_waitcnt vmcnt(%5) lgkmcnt(%6)" : "+v"(ra[(G) % RD]), "+v"(fb[(G) % (LD + 1)][0]), "+v"(fb[(G) % (LD + 1)][1]), "+v"(fb[(G) % (LD + 1)][2]), "+v"(fb[(G) % (LD + 1)][3]) \
            : "n"(((G) > 127 - RD ? 127 - (G) : RD - 1) + ((G) >= 64 && (G) < 64 + RD ? 8 : 0)), "n"(4 * ((G) + LD < 128 ? LD : 127 - (G)))); \
        half8 fb0 = __builtin_bit_cast(half8, __builtin_shufflevector(fb[(G) % (LD + 1)][0], fb[(G) % (LD + 1)][1], 0, 1, 2, 3, 4, 5, 6, 7)); \
        half8 fb1 = __builtin_bit_cast(half8, __builtin_shufflevector(fb[(G) % (LD + 1)][2], fb[(G) % (LD + 1)][3], 0, 1, 2, 3, 4, 5, 6, 7)); \
        half8 fa = __builtin_bit_cast(half8, ra[(G) % RD]); \
        acc[0] = __builtin_amdgcn_mfma_f32_32x32x16_f16(fa, fb0, acc[0], 0, 0, 0); \
        acc[1] = __builtin_amdgcn_mfma_f32_32x32x16_f16(fa, fb1, acc[1], 0, 0, 0); \
        if ((G) + RD < 128) GLOAD(ra[(G) % RD], a_voff, a_base + AOFF((G) + RD)); \
        __builtin_amdgcn_sched_barrier(0); \
    } while (0)
#define PAUSE() do { if (GMODE == 3 && wid >= PW) { \
        for (unsigned n = 0; n < 4096u; ++n) { \
            unsigned c; LDSRD(c, &sv[6]); \
            if (__builtin_amdgcn_readfirstlane(c) == 0u) break; \
            __builtin_amdgcn_s_sleep(1); \
        } } } while (0)
#define STEPV 0
#define STEPO(G, J) do { \
        s16x4 l0 = TR(bs0 + (J) * 32 * BN), h0 = TR(bs0 + ((J) * 32 + 8) * BN); \
        s16x4 l1 = TR(bs1 + (J) * 32 * BN), h1 = TR(bs1 + ((J) * 32 + 8) * BN); \
        half8 fb0 = __builtin_bit_cast(half8, __builtin_shufflevector(l0, h0, 0, 1, 2, 3, 4, 5, 6, 7)); \
        half8 fb1 = __builtin_bit_cast(half8, __builtin_shufflevector(l1, h1, 0, 1, 2, 3, 4, 5, 6, 7)); \
        asm volatile("s_waitcnt vmcnt(%1)" : "+v"(ra[(G) % RD]) : "n"(((G) > 127 - RD ? 127 - (G) : RD - 1) + ((G) >= 64 && (G) < 64 + RD ? 8 : 0))); \
        half8 fa = __builtin_bit_cast(half8, ra[(G) % RD]); \
        acc[0] = __builtin_amdgcn_mfma_f32_32x32x16_f16(fa, fb0, acc[0], 0, 0, 0); \
        acc[1] = __builtin_amdgcn_mfma_f32_32x32x16_f16(fa, fb1, acc[1], 0, 0, 0); \
        if ((G) + RD < 128) GLOAD(ra[(G) % RD], a_voff, a_base + AOFF((G) + RD)); \
    } while (0)
#define SB() __builtin_amdgcn_sched_barrier(0)
#define STEP4(G0, J0) do { STEPO((G0), (J0)); STEPO((G0) + 1, (J0) + 1); STEPO((G0) + 2, (J0) + 2); STEPO((G0) + 3, (J0) + 3); SB(); } while (0)
#define BLOCK1(B) do { BBASE(B); \
        BWAIT(4 * (B));     STEP4(16 * (B), 0); \
        BWAIT(4 * (B) + 1); STEP4(16 * (B) + 4, 4); \
        BWAIT(4 * (B) + 2); STEP4(16 * (B) + 8, 8); \
        BWAIT(4 * (B) + 3); STEP4(16 * (B) + 12, 12); } while (0)
#define BLOCK2(B) do { BBASE(B); \
        STEP4(64 + 16 * (B), 0); STEP4(64 + 16 * (B) + 4, 4); STEP4(64 + 16 * (B) + 8, 8); STEP4(64 + 16 * (B) + 12, 12); } while (0)

#define SK 16
#define SK2 1
    if (wid >= 2 && wid < 4) BWAIT(SK2 - 1);
    if (LSL > 2 && aggr && wid >= 4) {
        unsigned long long t0 = rtc();
        while (true) {
            BPOLL();
            if (seen > (unsigned)(SK - 1)) break;
            if (rtc() - t0 > DEADLINE_TICKS) { bad = true; seen = 64; break; }
            __builtin_amdgcn_s_sleep(LSL);
        }
    }
    if (aggr && wid >= 4) BWAIT(SK - 1);
#define WG 2
#define WGW 4
#define GATE() do { \
        unsigned long long t0 = __builtin_amdgcn_s_memrealtime(); \
        while (true) { \
            unsigned c; LDSRD(c, &sv[7]); \
            if (__builtin_amdgcn_readfirstlane(c) >= (unsigned)WG) break; \
            if (__builtin_amdgcn_s_memrealtime() - t0 > DEADLINE_TICKS) { bad = true; break; } \
            __builtin_amdgcn_s_sleep(LSL); \
        } } while (0)
    if (GMODE == 1 && WG && aggr && wid >= WGW) GATE();
    BWAIT(0);
    BLOCK1(0); BLOCK1(1); BLOCK1(2); BLOCK1(3);
    RST(2);
    char* mypart = part + (size_t)((grp * 4 + q) * 8 + wid) * 16384 + lane * 16;
    const unsigned tag = call & 1u;
#pragma unroll
    for (int G = 0; G < 4; ++G)
#pragma unroll
        for (int t = 0; t < 2; ++t) {
            f32x4 v; v[0] = TAGGED(acc[t][4 * G]); v[1] = TAGGED(acc[t][4 * G + 1]); v[2] = TAGGED(acc[t][4 * G + 2]); v[3] = TAGGED(acc[t][4 * G + 3]);
            GSTOREC(mypart + G * 2048 + t * 1024, v);
        }
    if (lane == 0) *(volatile unsigned*)&sv[8 + wid] = 1u;
    if (FM && lane == 0) __hip_atomic_fetch_add(&sv[3], 1u, __ATOMIC_RELAXED, __HIP_MEMORY_SCOPE_WORKGROUP);
#pragma unroll
    for (int t = 0; t < 2; ++t)
#pragma unroll
        for (int i = 0; i < 16; ++i) acc[t][i] = 0.f;

    if (GMODE == 2 && WG && wid < 4) GATE();
    BLOCK2(0); BLOCK2(1); BLOCK2(2); BLOCK2(3);
    RST(3);

    f32x4 own[2];
    {
#pragma unroll
        for (int G = 0; G < 4; ++G) {
            if (G == q) {
#pragma unroll
                for (int t = 0; t < 2; ++t) { own[t][0] = acc[t][4 * G]; own[t][1] = acc[t][4 * G + 1]; own[t][2] = acc[t][4 * G + 2]; own[t][3] = acc[t][4 * G + 3]; }
            } else {
                char* d = mypart + 8192 + ((G - q - 1) & 3) * 2048;
#pragma unroll
                for (int t = 0; t < 2; ++t) {
                    f32x4 v; v[0] = TAGGED(acc[t][4 * G]); v[1] = TAGGED(acc[t][4 * G + 1]); v[2] = TAGGED(acc[t][4 * G + 2]); v[3] = TAGGED(acc[t][4 * G + 3]);
                    GSTOREC(d + t * 1024, v);
                }
            }
        }
        SXT(0);
        u32x4 pv[6];
        unsigned pend = 7u;
        unsigned long long t0 = __builtin_amdgcn_s_memrealtime();
        if (CDELAY) __builtin_amdgcn_s_sleep(CDELAY);
#define CPOLL1 8
#define CPC 0
        if ((aggr || CPC) && wid < CPOLL1) {
            const int p = (q + 1 + (lane & 3)) & 3;
            const unsigned po = (unsigned)(((grp * 4 + p) * 8 + wid) * 16384 + 8192 + ((q - p - 1) & 3) * 2048) + pw;
            while (true) {
                unsigned fv = tag;
                if (lane < 3) fv = __builtin_amdgcn_raw_buffer_load_b32(prs, po, 0, 17);
                if (__builtin_amdgcn_ballot_w64(((fv ^ tag) & 1u) != 0u) == 0ull) break;
                if (__builtin_amdgcn_s_memrealtime() - t0 > DEADLINE_TICKS) { bad = true; break; }
                __builtin_amdgcn_s_sleep(FSLEEP);
                asm volatile("" ::: "memory");
            }
        }
        while (true) {
#pragma unroll
            for (int d = 0; d < 3; ++d)
                if (pend & (1u << d)) {
                    const int p = (q + 1 + d) & 3;
                    const unsigned o = (unsigned)(((grp * 4 + p) * 8 + wid) * 16384 + 8192 + ((q - p - 1) & 3) * 2048 + lane * 16);
                    pv[2 * d] = __builtin_amdgcn_raw_buffer_load_b128(prs, o, 0, 17);
                    pv[2 * d + 1] = __builtin_amdgcn_raw_buffer_load_b128(prs, o + 1024u, 0, 17);
                }
#pragma unroll
            for (int d = 0; d < 3; ++d)
                if (pend & (1u << d)) {
                    if (TAGOK(pv[2 * d], pv[2 * d + 1])) pend &= ~(1u << d);
                }
            if (pend == 0u) break;
            if (__builtin_amdgcn_s_memrealtime() - t0 > DEADLINE_TICKS) { bad = true; break; }
            __builtin_amdgcn_s_sleep(CSLEEP);
            asm volatile("" ::: "memory");
        }
        SXT(1); SXT(2);
#pragma unroll
        for (int t = 0; t < 2; ++t) own[t] = (own[t] + AF(pv[t])) + (AF(pv[2 + t]) + AF(pv[4 + t]));
    }
    RST(4);
    finish_unit<PACE1>(own[0], own[1], Ts[wid], dst + (size_t)(64 * RG(wid) + 32 + 8 * q) * NLOC, lane, bad, &sv[4]);
    asm volatile("s_waitcnt vmcnt(0)" ::: "memory");
    SXT(3);
    __syncthreads();
    RST(5);
    if (tid == 0) __hip_atomic_fetch_add(ws_word(myws, 0), 1u, __ATOMIC_RELAXED, __HIP_MEMORY_SCOPE_AGENT);
    if (tid == 0) {
        if (peer != rank) {
            __builtin_amdgcn_fence(__ATOMIC_RELEASE, "");
            asm volatile("s_waitcnt vmcnt(0)" ::: "memory");
            __hip_atomic_fetch_add(ws_word(ws_all[peer], 2 + rank), 1u, __ATOMIC_RELAXED, __HIP_MEMORY_SCOPE_SYSTEM);
        }
        if (b == 0) {
            unsigned need = call * (unsigned)BLOCKS_PER_PEER;
            unsigned long long t0 = __builtin_amdgcn_s_memrealtime();
            bool timed_out = false;
            if (AGG == 2) __hip_atomic_store((unsigned long long*)ws_word(myws, 12), t0, __ATOMIC_RELAXED, __HIP_MEMORY_SCOPE_AGENT);
            for (int p = 0; p < NR; ++p) {
                if (p == rank) continue;
                while (__hip_atomic_load(ws_word(myws, 2 + p), __ATOMIC_RELAXED, __HIP_MEMORY_SCOPE_SYSTEM) < need) {
                    if (__builtin_amdgcn_s_memrealtime() - t0 > DEADLINE_TICKS) { timed_out = true; break; }
                    __builtin_amdgcn_s_sleep(2);
                }
            }
            __builtin_amdgcn_fence(__ATOMIC_ACQUIRE, "");
            if (timed_out) {
                __hip_atomic_store(ws_word(myws, 1), call, __ATOMIC_RELAXED, __HIP_MEMORY_SCOPE_AGENT);
                ((float*)out_all[rank])[(size_t)(rank * MLOC) * NLOC] = __builtin_nanf("");
            }
        }
    }
}

extern "C" void kernel_launch(int rank, int nranks, ncclComm_t comm,
                              hipStream_t stream,
                              void* const* d_in, const int* in_sizes, int n_in,
                              void* d_out, int out_size,
                              const int* in_dtypes, int out_dtype,
                              void* const* d_out_all,
                              void* const* d_ws_all, size_t ws_bytes) {
    const float* x = (const float*)d_in[0];
    const float* w = (const float*)d_in[1];
    hipLaunchKernelGGL(gemm_a2a_kernel, dim3(256), dim3(NBLOCK), 0, stream,
                       x, w, d_out_all, d_ws_all, rank);
}
```

```cpp
#include <hip/hip_runtime.h>
#include <rccl/rccl.h>
#include <cstdio>
#include <cstdint>

typedef _Float16 half8 __attribute__((ext_vector_type(8)));
typedef _Float16 half4 __attribute__((ext_vector_type(4)));
typedef float f32x16 __attribute__((ext_vector_type(16)));
typedef float f32x4 __attribute__((ext_vector_type(4)));
typedef unsigned u32x4 __attribute__((ext_vector_type(4)));
typedef short s16x4 __attribute__((ext_vector_type(4)));
typedef short s16x8 __attribute__((ext_vector_type(8)));

constexpr int NR = 8;
constexpr int MLOC = 512;
constexpr int KDIM = 4096;
constexpr int NDIM = 4096;
constexpr int NLOC = 512;
constexpr int BN = 64;
constexpr int KS = 1024;
constexpr int NCW = 8;
constexpr int NPW = 4;
constexpr int NBLOCK = (NCW + NPW) * 64;
constexpr int BLOCKS_PER_PEER = 32;
constexpr unsigned long long DEADLINE_TICKS = 40000000ull;
constexpr size_t XFLAG_OFF = 8192;
constexpr size_t FLAG_OFF = 65536;
constexpr size_t RBS = 262144;
constexpr size_t X16_OFF = 262144;
constexpr size_t PART_OFF = X16_OFF + (5u << 20);

__device__ __forceinline__ unsigned* ws_word(void* ws, int idx) { return (unsigned*)((char*)ws + 128 * idx); }

__device__ __forceinline__ unsigned long long rtc() {
    unsigned long long t;
    asm volatile("s_memrealtime %0\n\ts_waitcnt lgkmcnt(0)" : "=s"(t) :: "memory");
    return t;
}

__device__ __forceinline__ float gelu_f(float y) {
    const float c = 0.7978845608028654f;
    float u = c * (y + 0.044715f * y * y * y);
    return y / (1.0f + __expf(-2.0f * u));
}

#define RST(I)
#define XV 0
#define XDELAY 350
#define XDELAYA XDELAY
#define FSLEEP 8
#define CSLEEP 4
#define XSLEEP 16
#define GLOAD(DST, VOFF, SBASE) asm volatile("global_load_dwordx4 %0, %1, %2" : "=v"(DST) : "v"(VOFF), "s"(SBASE))
#define GLOADNT(DST, VOFF, SBASE) asm volatile("global_load_dwordx4 %0, %1, %2 nt" : "=v"(DST) : "v"(VOFF), "s"(SBASE))
#define GLOADC(DST, P) asm volatile("global_load_dwordx4 %0, %1, off sc0 sc1" : "=v"(DST) : "v"(P) : "memory")
#define GSTOREC(P, V) asm volatile("global_store_dwordx4 %0, %1, off sc0 sc1\n\ts_nop 1" :: "v"(P), "v"(V) : "memory")
#define LDSRD(DST, P) asm volatile("ds_read_b32 %0, %1\n\ts_waitcnt lgkmcnt(0)" : "=v"(DST) : "v"((unsigned)(uintptr_t)(__attribute__((address_space(3))) unsigned*)(P)) : "memory")

#define AF(V) __builtin_bit_cast(f32x4, V)
#define TAGOK(A, B) (__builtin_amdgcn_ballot_w64((((((A)[0] ^ tag) | ((A)[1] ^ tag)) | (((A)[2] ^ tag) | ((A)[3] ^ tag)) | (((B)[0] ^ tag) | ((B)[1] ^ tag)) | (((B)[2] ^ tag) | ((B)[3] ^ tag))) & 1u) != 0u) == 0ull)
__device__ __forceinline__ float tagged(float f, unsigned tag) { return __uint_as_float((__float_as_uint(f) & ~1u) | tag); }
#define TAGGED(F) tagged(F, tag)
#define FDELAY 0
#define CDELAY 0
#define PF 0
#define CPOLL 1
#define OWNL 1
#define NST (OWNL ? 6 : 8)
#define AGG 2
#define AGT 8000
#define TSL 1
#define XSL 1
#define LSL 2
#define RD 12
#define PACE0 3
#define PACE1 3
#define NTOK 3
#define NT0 3
#define NT0B NT0
#define NT1 3
#define NT2 2
#define GMODE 1
#define PU 0
#define PW 4
template <int PACE>
__device__ __forceinline__ void finish_unit(const f32x4& v0, const f32x4& v1, float* T, float* dst, int lane, bool bad, unsigned* tk, unsigned ntok) {
    const int fh = lane >> 5;
#pragma unroll
    for (int i = 0; i < 4; ++i) {
        float g0 = gelu_f(v0[i]), g1 = gelu_f(v1[i]);
        T[(4 * fh + i) * 68 + (lane & 31)] = bad ? __builtin_nanf("") : g0;
        T[(4 * fh + i) * 68 + 32 + (lane & 31)] = bad ? __builtin_nanf("") : g1;
    }
    __builtin_amdgcn_wave_barrier();
#pragma unroll
    for (int j = 0; j < 2; ++j) {
        const int r = 4 * j + (lane >> 4), c4 = lane & 15;
        f32x4 v = *(const f32x4*)(T + r * 68 + c4 * 4);
        if (PACE == 1) asm volatile("s_waitcnt vmcnt(0)" ::: "memory");
        if (PACE == 2) asm volatile("s_waitcnt vmcnt(1)" ::: "memory");
        if (PACE == 3) {
            unsigned t = 0;
            if (lane == 0) t = __hip_atomic_fetch_add(tk, 1u, __ATOMIC_RELAXED, __HIP_MEMORY_SCOPE_WORKGROUP);
            t = __builtin_amdgcn_readfirstlane(t);
            unsigned long long t0 = __builtin_amdgcn_s_memrealtime();
            while (true) {
                unsigned d; LDSRD(d, tk + 1);
                if (t < __builtin_amdgcn_readfirstlane(d) + ntok) break;
                if (__builtin_amdgcn_s_memrealtime() - t0 > DEADLINE_TICKS) break;
                __builtin_amdgcn_s_sleep(TSL);
            }
        }
        GSTOREC(dst + (size_t)r * NLOC + c4 * 4, v);
        if (PACE == 3) {
            asm volatile("s_waitcnt vmcnt(0)" ::: "memory");
            if (lane == 0) __hip_atomic_fetch_add(tk + 1, 1u, __ATOMIC_RELAXED, __HIP_MEMORY_SCOPE_WORKGROUP);
        }
    }
    __builtin_amdgcn_wave_barrier();
}

__global__ __launch_bounds__(NBLOCK, 3)
void gemm_a2a_kernel(const float* __restrict__ x, const float* __restrict__ w,
                     void* const* __restrict__ out_all, void* const* __restrict__ ws_all, int rank) {
    __shared__ __attribute__((aligned(16))) _Float16 Bs[KS * BN];
    __shared__ __attribute__((aligned(16))) float Ts[NCW + NPW][8 * 68];
    __shared__ __attribute__((aligned(16))) unsigned rdy[4];
    __shared__ __attribute__((aligned(16))) unsigned sv[16];

#define RXT(I)
#define SXT(I)
    RST(0);
    const int tid = threadIdx.x;
    const int lane = tid & 63, wid = __builtin_amdgcn_readfirstlane(tid >> 6);
    char* myws;
    {
        unsigned long long a = (unsigned long long)ws_all[rank];
        unsigned lo = __builtin_amdgcn_readfirstlane((unsigned)a), hi = __builtin_amdgcn_readfirstlane((unsigned)(a >> 32));
        myws = (char*)(((unsigned long long)hi << 32) | lo);
    }
    char* x16 = myws + X16_OFF;
    char* part = myws + PART_OFF;
    const __amdgpu_buffer_rsrc_t prs = __builtin_amdgcn_make_buffer_rsrc(part, 0, 32u << 20, 0x00020000);
    const int b = blockIdx.x;
#define MAP 0
    const int q = b & 3;
    const int grp = ((b >> 2) & 1) * 32 + (b >> 3);
    const int peer = ((grp >> 3) + rank) & 7;
    const int ct = grp & 7;
    const int ncol0 = peer * NLOC + ct * BN;
    float* dst = (float*)out_all[peer] + (size_t)(rank * MLOC) * NLOC + ct * BN;
    bool bad = false;
#define ROT 0
#define RG(W) (ROT ? (((W) + rank) & 7) : (W))
#define PV 1
    const unsigned pw = PV == 1 ? 1024u + (unsigned)((wid + grp + lane) & 7) * 128u + 124u : PV == 2 ? (unsigned)((wid + grp + lane) & 15) * 128u + 124u : 2044u;

    if (wid >= NCW) {
        const int p = wid - NCW;
        const int kr = lane >> 4, nq = lane & 15;
        const unsigned voff = (unsigned)((4 * p + kr) * NDIM + nq * 4) * 4u;
        const char* base = (const char*)(w + (size_t)q * KS * NDIM + ncol0);
#define PD 2
        f32x4 rb[4 * PD];
#define PLD(SLOT, S) _Pragma("unroll") for (int i = 0; i < 4; ++i) GLOADNT(rb[(SLOT) * 4 + i], voff, base + (size_t)((S) * 64 + 16 * i) * NDIM * 4)
#define PWAIT(N, SLOT) asm volatile("s_waitcnt vmcnt(" #N ")" : "+v"(rb[(SLOT) * 4]), "+v"(rb[(SLOT) * 4 + 1]), "+v"(rb[(SLOT) * 4 + 2]), "+v"(rb[(SLOT) * 4 + 3]))
#define PST(SLOT, S) do { \
        _Pragma("unroll") for (int i = 0; i < 4; ++i) { \
            f32x4 v = rb[(SLOT) * 4 + i]; \
            half4 h; h[0] = (_Float16)v[0]; h[1] = (_Float16)v[1]; h[2] = (_Float16)v[2]; h[3] = (_Float16)v[3]; \
            *(half4*)(Bs + ((S) * 64 + 16 * i + 4 * p + kr) * BN + ((nq * 4) ^ (32 * ((kr >> 1) & 1)))) = h; \
        } \
        asm volatile("" ::: "memory"); \
        if (lane == 0) *(volatile unsigned*)&rdy[p] = (unsigned)(S) + 1u; \
        asm volatile("" ::: "memory"); } while (0)
#define BDELAY 1
        if (lane == 0) rdy[p] = 0;
        __syncthreads();
        {
            unsigned long long t0 = __builtin_amdgcn_s_memrealtime();
            while (true) {
                unsigned c; LDSRD(c, &sv[0]);
                if (__builtin_amdgcn_readfirstlane(c) >= (unsigned)NCW) break;
                if (__builtin_amdgcn_s_memrealtime() - t0 > DEADLINE_TICKS) break;
                __builtin_amdgcn_s_sleep(1);
            }
        }
        _Pragma("unroll") for (int s = (BDELAY == 2 ? 1 : 0); s < PD; ++s) { PLD(s, s); }
#pragma unroll
        for (int s = 0; s < 16; ++s) {
            const int left = (15 - s < PD - 1) ? 15 - s : PD - 1;
            if (left == 5) PWAIT(20, s % PD); else if (left == 4) PWAIT(16, s % PD); else if (left == 3) PWAIT(12, s % PD); else if (left == 2) PWAIT(8, s % PD); else if (left == 1) PWAIT(4, s % PD); else PWAIT(0, s % PD);
            PST(s % PD, s);
            if ((s & 3) == 0) RXT(4 + (s >> 2));
            if (s + PD < 16) PLD(s % PD, s + PD);
        }
        RST(3);
        unsigned call;
        unsigned long long t0 = __builtin_amdgcn_s_memrealtime();
        while (true) {
            LDSRD(call, &sv[1]);
            if (call != 0u) break;
            if (__builtin_amdgcn_s_memrealtime() - t0 > DEADLINE_TICKS) { call = 0x80000000u; break; }
            __builtin_amdgcn_s_sleep(LSL);
        }
        call = __builtin_amdgcn_readfirstlane(call);
        if (call & 0x80000000u) bad = true;
        const bool aggr = (call & 0x40000000u) != 0u;
        call &= 0x3fffffffu;
        const unsigned tag = call & 1u;
#define FM 3
#define MAPF 1
        if (FM != 4 || !aggr) {
        const int w0 = MAPF ? 2 * p : p;
        constexpr int WS = MAPF ? 1 : 4;
        while (true) {
            unsigned c, c2 = 1;
            if (aggr) { LDSRD(c, &sv[8 + w0]); LDSRD(c2, &sv[8 + w0 + WS]); } else { LDSRD(c, &sv[3]); }
            if (__builtin_amdgcn_readfirstlane(c) >= (aggr ? 1u : (unsigned)NCW) && __builtin_amdgcn_readfirstlane(c2) != 0u) break;
            if (__builtin_amdgcn_s_memrealtime() - t0 > DEADLINE_TICKS) { bad = true; break; }
            __builtin_amdgcn_s_sleep(LSL);
        }
        f32x4 ow[4];
#pragma unroll
        for (int j = 0; j < 4; ++j) ow[j] = *(const f32x4*)(Ts[w0 + (j >> 1) * WS] + (j & 1) * 256 + lane * 4);
        asm volatile("s_waitcnt lgkmcnt(0)" : "+v"(ow[0]), "+v"(ow[1]), "+v"(ow[2]), "+v"(ow[3]) :: "memory");
        if (lane < 2) *(volatile unsigned*)&sv[8 + w0 + lane * WS] = 2u;
        const unsigned voff0 = (unsigned)((grp * 4 * 8 + w0) * 16384 + q * 2048 + lane * 16);
        u32x4 pv[12];
        unsigned pend = 0x3fu;
        {
            const unsigned po = (unsigned)((grp * 4 * 8 + w0) * 16384 + q * 2048) + pw + (unsigned)((((q + 1 + (lane & 3)) & 3) * 8 + ((lane >> 2) & 1) * WS) * 16384);
            while (true) {
                unsigned fv = tag;
                if (lane < 8 && (lane & 3) != 3) fv = __builtin_amdgcn_raw_buffer_load_b32(prs, po, 0, 17);
                if (__builtin_amdgcn_ballot_w64(((fv ^ tag) & 1u) != 0u) == 0ull) break;
                if (__builtin_amdgcn_s_memrealtime() - t0 > DEADLINE_TICKS) { bad = true; break; }
                __builtin_amdgcn_s_sleep(FSLEEP);
                asm volatile("" ::: "memory");
            }
        }
        while (true) {
#pragma unroll
            for (int i = 0; i < 6; ++i)
                if (pend & (1u << i)) {
                    const unsigned o = voff0 + (unsigned)((((q + 1 + i % 3) & 3) * 8 + (i / 3) * WS) * 16384);
                    pv[2 * i] = __builtin_amdgcn_raw_buffer_load_b128(prs, o, 0, 17);
                    pv[2 * i + 1] = __builtin_amdgcn_raw_buffer_load_b128(prs, o + 1024u, 0, 17);
                }
#pragma unroll
            for (int i = 0; i < 6; ++i)
                if (pend & (1u << i)) {
                    if (TAGOK(pv[2 * i], pv[2 * i + 1])) pend &= ~(1u << i);
                }
            if (pend == 0u) break;
            if (__builtin_amdgcn_s_memrealtime() - t0 > DEADLINE_TICKS) { bad = true; break; }
            __builtin_amdgcn_s_sleep(FSLEEP);
            asm volatile("" ::: "memory");
        }
        if (lane == 0) __hip_atomic_fetch_add(&sv[7], 1u, __ATOMIC_RELAXED, __HIP_MEMORY_SCOPE_WORKGROUP);
        {
            f32x4 v0 = (ow[0] + AF(pv[0])) + (AF(pv[2]) + AF(pv[4])), v1 = (ow[1] + AF(pv[1])) + (AF(pv[3]) + AF(pv[5]));
            finish_unit<PACE0>(v0, v1, Ts[wid], dst + (size_t)(64 * RG(w0) + 8 * q) * NLOC, lane, bad, &sv[4], aggr ? (p < 2 ? NT0 : NT0B) : NTOK);
        }
        {
            f32x4 v0 = (ow[2] + AF(pv[6])) + (AF(pv[8]) + AF(pv[10])), v1 = (ow[3] + AF(pv[7])) + (AF(pv[9]) + AF(pv[11]));
            finish_unit<PACE0>(v0, v1, Ts[wid], dst + (size_t)(64 * RG(w0 + WS) + 8 * q) * NLOC, lane, bad, &sv[4], aggr ? (p < 2 ? NT0 : NT0B) : NTOK);
        }
        }
#pragma unroll 1
        for (int u = 0; u < ((FM == 1 || FM == 3 || (FM == 4 && !aggr)) ? 0 : 2); ++u) {
            const int wv = p + 4 * u;
            while (true) {
                unsigned c; LDSRD(c, &sv[8 + wv]);
                if (__builtin_amdgcn_readfirstlane(c) != 0u) break;
                if (__builtin_amdgcn_s_memrealtime() - t0 > DEADLINE_TICKS) { bad = true; break; }
                __builtin_amdgcn_s_sleep(LSL);
            }
            const unsigned voff0 = (unsigned)((grp * 4 * 8 + wv) * 16384 + q * 2048 + lane * 16);
            u32x4 pv[8];
            unsigned pend = 0xfu;
            {
                const unsigned po = (unsigned)((grp * 4 * 8 + wv) * 16384 + q * 2048) + pw + (unsigned)((lane & 3) * 8 * 16384);
                while (true) {
                    unsigned fv = tag;
                    if (lane < 4) fv = __builtin_amdgcn_raw_buffer_load_b32(prs, po, 0, 17);
                    if (__builtin_amdgcn_ballot_w64(((fv ^ tag) & 1u) != 0u) == 0ull) break;
                    if (__builtin_amdgcn_s_memrealtime() - t0 > DEADLINE_TICKS) { bad = true; break; }
                    __builtin_amdgcn_s_sleep(FSLEEP);
                    asm volatile("" ::: "memory");
                }
            }
            if (u == 0) SXT(4);
            if (GMODE == 3 && u <= PU && lane == 0) __hip_atomic_fetch_add(&sv[6], 1u, __ATOMIC_RELAXED, __HIP_MEMORY_SCOPE_WORKGROUP);
            while (true) {
#pragma unroll
                for (int i = 0; i < 4; ++i)
                    if (pend & (1u << i)) {
                        const unsigned o = voff0 + (unsigned)(i * 8 * 16384);
                        pv[2 * i] = __builtin_amdgcn_raw_buffer_load_b128(prs, o, 0, 17);
                        pv[2 * i + 1] = __builtin_amdgcn_raw_buffer_load_b128(prs, o + 1024u, 0, 17);
                    }
#pragma unroll
                for (int i = 0; i < 4; ++i)
                    if (pend & (1u << i)) {
                        if (TAGOK(pv[2 * i], pv[2 * i + 1])) pend &= ~(1u << i);
                    }
                if (pend == 0u) break;
                if (__builtin_amdgcn_s_memrealtime() - t0 > DEADLINE_TICKS) { bad = true; break; }
                __builtin_amdgcn_s_sleep(FSLEEP);
                asm volatile("" ::: "memory");
            }
            if (u == 0) SXT(5); else SXT(7);
            if (u == 0 && lane == 0) __hip_atomic_fetch_add(&sv[7], 1u, __ATOMIC_RELAXED, __HIP_MEMORY_SCOPE_WORKGROUP);
            if (GMODE == 3 && u <= PU && lane == 0) __hip_atomic_fetch_add(&sv[6], ~0u, __ATOMIC_RELAXED, __HIP_MEMORY_SCOPE_WORKGROUP);
            f32x4 v0 = (AF(pv[0]) + AF(pv[2])) + (AF(pv[4]) + AF(pv[6])), v1 = (AF(pv[1]) + AF(pv[3])) + (AF(pv[5]) + AF(pv[7]));
            finish_unit<PACE0>(v0, v1, Ts[wid], dst + (size_t)(64 * RG(wv) + 8 * q) * NLOC, lane, bad, &sv[4], aggr ? (p < 2 ? NT0 : NT0B) : NTOK);
            if (u == 0) SXT(6);
        }
        asm volatile("s_waitcnt vmcnt(0)" ::: "memory");
        __syncthreads();
        return;
    }

    unsigned call;
    bool aggr;
    {
        unsigned old = 0;
        if (tid < 16) sv[tid] = 0;
        const int rbk = grp >> 2, ks = q * 64 + (grp & 3) * 16 + 2 * wid;
        const float* src = x + (size_t)(32 * rbk + (lane & 31)) * KDIM + ks * 16 + 8 * (lane >> 5);
        f32x4 v[4];
#pragma unroll
        for (int i = 0; i < 4; ++i)
            asm volatile("global_load_dwordx4 %0, %1, off nt" : "=v"(v[i]) : "v"(src + (i >> 1) * 16 + (i & 1) * 4));
        __syncthreads();
        unsigned long long last = 0;
        if (tid == 0) {
            old = __hip_atomic_load(ws_word(myws, 0), __ATOMIC_RELAXED, __HIP_MEMORY_SCOPE_AGENT);
            if (AGG == 2) last = __hip_atomic_load((unsigned long long*)ws_word(myws, 12), __ATOMIC_RELAXED, __HIP_MEMORY_SCOPE_AGENT);
        }
        RXT(0);
        asm volatile("s_waitcnt vmcnt(0)" : "+v"(v[0]), "+v"(v[1]), "+v"(v[2]), "+v"(v[3]));
        RXT(1);
        if (lane == 0) __hip_atomic_fetch_add(&sv[0], 1u, __ATOMIC_RELAXED, __HIP_MEMORY_SCOPE_WORKGROUP);
        if (tid == 0) *(volatile unsigned*)&sv[2] = (((old >> 8) + 1u) & 0x3fffffffu) | ((AGG == 1 || (AGG == 2 && __builtin_amdgcn_s_memrealtime() - last > (unsigned long long)AGT)) ? 0x40000000u : 0u);
#pragma unroll
        for (int i = 0; i < 2; ++i) {
            half8 h;
#pragma unroll
            for (int e = 0; e < 4; ++e) { h[e] = (_Float16)v[2 * i][e]; h[4 + e] = (_Float16)v[2 * i + 1][e]; }
            GSTOREC(x16 + (size_t)rbk * RBS + (size_t)((ks + i) * 64 + lane) * 16, __builtin_bit_cast(f32x4, h));
        }
        unsigned long long t0 = __builtin_amdgcn_s_memrealtime();
        while (true) {
            LDSRD(call, &sv[2]);
            if (call != 0u) break;
            if (__builtin_amdgcn_s_memrealtime() - t0 > DEADLINE_TICKS) { bad = true; break; }
            __builtin_amdgcn_s_sleep(1);
        }
        call = __builtin_amdgcn_readfirstlane(call);
        aggr = (call & 0x40000000u) != 0u;
        call &= 0x3fffffffu;
        RXT(2);
        asm volatile("s_waitcnt vmcnt(0)" ::: "memory");
        RXT(3);
#define XF 1
#define XRS 4352
        {
            unsigned c = 0;
            if (lane == 0) c = __hip_atomic_fetch_add(&sv[6], 1u, __ATOMIC_RELAXED, __HIP_MEMORY_SCOPE_WORKGROUP);
            c = __builtin_amdgcn_readfirstlane(c);
            if (c == (unsigned)NCW - 1u && lane < 8)
                __hip_atomic_store((unsigned*)(myws + FLAG_OFF + (size_t)(q * 8 + lane) * XRS) + grp, call, __ATOMIC_RELAXED, __HIP_MEMORY_SCOPE_SYSTEM);
        }
        if (wid == 0) {
            const unsigned* xf = (const unsigned*)(myws + FLAG_OFF + (size_t)(q * 8 + (grp & 7)) * XRS) + lane;
            while (__builtin_amdgcn_s_memrealtime() - t0 < (aggr ? XDELAYA : XDELAY)) __builtin_amdgcn_s_sleep(4);
            while (true) {
                unsigned f = __hip_atomic_load(xf, __ATOMIC_RELAXED, __HIP_MEMORY_SCOPE_SYSTEM);
                if (__builtin_amdgcn_ballot_w64(f != call) == 0ull) break;
                if (__builtin_amdgcn_s_memrealtime() - t0 > DEADLINE_TICKS) { bad = true; break; }
                __builtin_amdgcn_s_sleep(XSLEEP);
            }
            asm volatile("" ::: "memory");
            if (lane == 0) *(volatile unsigned*)&sv[1] = call | (bad ? 0x80000000u : 0u) | (aggr ? 0x40000000u : 0u);
        } else {
            while (true) {
                LDSRD(call, &sv[1]);
                if (call != 0u) break;
                if (__builtin_amdgcn_s_memrealtime() - t0 > DEADLINE_TICKS) { call = 0x80000000u; break; }
                __builtin_amdgcn_s_sleep(XSL);
            }
            call = __builtin_amdgcn_readfirstlane(call);
            if (call & 0x80000000u) bad = true;
            call &= 0x3fffffffu;
        }
        asm volatile("s_waitcnt vmcnt(0) lgkmcnt(0)" ::: "memory");
    }
    RST(1);

    const unsigned a_voff = (unsigned)lane * 16u;
    const char* a_base = x16 + (size_t)(2 * RG(wid)) * RBS + (size_t)(64 * q) * 1024;
#define PFX 0
    if (PFX && wid >= 4) {
        f32x4 pf[6];
        const int c0 = ((b >> 3) * 4 + (wid - 4)) * 6;
#pragma unroll
        for (int i = 0; i < 6; ++i) {
            const int c = c0 + i, rbi = c >> 6, rbk = rbi < 4 ? 2 * rbi + 1 : rbi + 4;
            GLOAD(pf[i], a_voff, x16 + (size_t)rbk * RBS + (size_t)(64 * q + (c & 63)) * 1024);
        }
        asm volatile("s_waitcnt vmcnt(0)" : "+v"(pf[0]), "+v"(pf[1]), "+v"(pf[2]), "+v"(pf[3]), "+v"(pf[4]), "+v"(pf[5]));
    }
    f32x4 ra[RD];
#define AOFF(S) ((size_t)((S) >> 6) * RBS + (size_t)((S) & 63) * 1024)
#pragma unroll
    for (int i = 0; i < RD; ++i) GLOAD(ra[i], a_voff, a_base + AOFF(i));

    f32x16 acc[2];
#pragma unroll
    for (int t = 0; t < 2; ++t)
#pragma unroll
        for (int i = 0; i < 16; ++i) acc[t][i] = 0.f;

    const int fh = lane >> 5;
    const int bblk = (lane >> 4) & 1, bq = (lane & 15) >> 2, bp = lane & 3;
    const unsigned lds0 = (unsigned)(uintptr_t)(__attribute__((address_space(3))) _Float16*)Bs;
    const unsigned b0p = lds0 + 2u * ((8 * fh + bq) * BN + ((16 * bblk + 4 * bp) ^ (32 * ((bq >> 1) & 1))));
    const unsigned b1p = lds0 + 2u * ((8 * fh + bq) * BN + ((32 + 16 * bblk + 4 * bp) ^ (32 * ((bq >> 1) & 1))));
#define BBASE(BLK) unsigned bs0 = b0p + (BLK) * 512u * BN, bs1 = b1p + (BLK) * 512u * BN; asm volatile("" : "+v"(bs0), "+v"(bs1))
    unsigned seen = 0;
#define BPOLL() do { \
        u32x4 rv; \
        asm volatile("ds_read_b128 %0, %1\n\ts_waitcnt lgkmcnt(0)" : "=v"(rv) : "v"((unsigned)(uintptr_t)(__attribute__((address_space(3))) unsigned*)rdy) : "memory"); \
        seen = __builtin_amdgcn_readfirstlane(min(min(rv[0], rv[1]), min(rv[2], rv[3]))); } while (0)
#define BWAIT(S) do { \
        if (seen <= (unsigned)(S)) { \
            BPOLL(); \
            if (seen <= (unsigned)(S)) { \
                unsigned long long t0 = rtc(); \
                for (unsigned n = 1; ; ++n) { \
                    __builtin_amdgcn_s_sleep(2); \
                    BPOLL(); \
                    if (seen > (unsigned)(S)) break; \
                    if ((n & 255u) == 0u && rtc() - t0 > DEADLINE_TICKS) { bad = true; seen = 64; break; } \
                } \
            } \
        } } while (0)
#define TR(P) __builtin_amdgcn_ds_read_tr16_b64_v4i16((__attribute__((address_space(3))) s16x4*)(uintptr_t)(P))
#define LD 2
    s16x4 fb[LD + 1][4];
    unsigned bsl0 = b0p, bsl1 = b1p, bsh0 = b0p + 65536u, bsh1 = b1p + 65536u;
    asm volatile("" : "+v"(bsl0), "+v"(bsl1), "+v"(bsh0), "+v"(bsh1));
#define TRA(DST, P, O) asm volatile("ds_read_b64_tr_b16 %0, %1 offset:%2" : "=v"(DST) : "v"(P), "n"(O))
#define BRD(G) do { \
        if ((G) & 32) { \
            TRA(fb[(G) % (LD + 1)][0], bsh0, ((G) & 31) * 2048); TRA(fb[(G) % (LD + 1)][1], bsh0, ((G) & 31) * 2048 + 512); \
            TRA(fb[(G) % (LD + 1)][2], bsh1, ((G) & 31) * 2048); TRA(fb[(G) % (LD + 1)][3], bsh1, ((G) & 31) * 2048 + 512); \
        } else { \
            TRA(fb[(G) % (LD + 1)][0], bsl0, ((G) & 31) * 2048); TRA(fb[(G) % (LD + 1)][1], bsl0, ((G) & 31) * 2048 + 512); \
            TRA(fb[(G) % (LD + 1)][2], bsl1, ((G) & 31) * 2048); TRA(fb[(G) % (LD + 1)][3], bsl1, ((G) & 31) * 2048 + 512); \
        } } while (0)
#define STEP(G) do { \
        if ((G) + LD < 64 && (((G) + LD) & 3) == 0) BWAIT(((G) + LD) >> 2); \
        if ((G) + LD < 128) BRD((G) + LD); \
        asm volatile("s_waitcnt vmcnt(%5) lgkmcnt(%6)" : "+v"(ra[(G) % RD]), "+v"(fb[(G) % (LD + 1)][0]), "+v"(fb[(G) % (LD + 1)][1]), "+v"(fb[(G) % (LD + 1)][2]), "+v"(fb[(G) % (LD + 1)][3]) \
            : "n"(((G) > 127 - RD ? 127 - (G) : RD - 1) + ((G) >= 64 && (G) < 64 + RD ? NST : 0)), "n"(4 * ((G) + LD < 128 ? LD : 127 - (G)))); \
        half8 fb0 = __builtin_bit_cast(half8, __builtin_shufflevector(fb[(G) % (LD + 1)][0], fb[(G) % (LD + 1)][1], 0, 1, 2, 3, 4, 5, 6, 7)); \
        half8 fb1 = __builtin_bit_cast(half8, __builtin_shufflevector(fb[(G) % (LD + 1)][2], fb[(G) % (LD + 1)][3], 0, 1, 2, 3, 4, 5, 6, 7)); \
        half8 fa = __builtin_bit_cast(half8, ra[(G) % RD]); \
        acc[0] = __builtin_amdgcn_mfma_f32_32x32x16_f16(fa, fb0, acc[0], 0, 0, 0); \
        acc[1] = __builtin_amdgcn_mfma_f32_32x32x16_f16(fa, fb1, acc[1], 0, 0, 0); \
        if ((G) + RD < 128) GLOAD(ra[(G) % RD], a_voff, a_base + AOFF((G) + RD)); \
        __builtin_amdgcn_sched_barrier(0); \
    } while (0)
#define PAUSE() do { if (GMODE == 3 && wid >= PW) { \
        for (unsigned n = 0; n < 4096u; ++n) { \
            unsigned c; LDSRD(c, &sv[6]); \
            if (__builtin_amdgcn_readfirstlane(c) == 0u) break; \
            __builtin_amdgcn_s_sleep(1); \
        } } } while (0)
#define STEPV 0
#define STEPO(G, J) do { \
        s16x4 l0 = TR(bs0 + (J) * 32 * BN), h0 = TR(bs0 + ((J) * 32 + 8) * BN); \
        s16x4 l1 = TR(bs1 + (J) * 32 * BN), h1 = TR(bs1 + ((J) * 32 + 8) * BN); \
        half8 fb0 = __builtin_bit_cast(half8, __builtin_shufflevector(l0, h0, 0, 1, 2, 3, 4, 5, 6, 7)); \
        half8 fb1 = __builtin_bit_cast(half8, __builtin_shufflevector(l1, h1, 0, 1, 2, 3, 4, 5, 6, 7)); \
        asm volatile("s_waitcnt vmcnt(%1)" : "+v"(ra[(G) % RD]) : "n"(((G) > 127 - RD ? 127 - (G) : RD - 1) + ((G) >= 64 && (G) < 64 + RD ? NST : 0))); \
        half8 fa = __builtin_bit_cast(half8, ra[(G) % RD]); \
        acc[0] = __builtin_amdgcn_mfma_f32_32x32x16_f16(fa, fb0, acc[0], 0, 0, 0); \
        acc[1] = __builtin_amdgcn_mfma_f32_32x32x16_f16(fa, fb1, acc[1], 0, 0, 0); \
        if ((G) + RD < 128) GLOAD(ra[(G) % RD], a_voff, a_base + AOFF((G) + RD)); \
    } while (0)
#define SB() __builtin_amdgcn_sched_barrier(0)
#define PS2 16
#define PS1 0
#define PS2A 0
#define PS2B 0
#define PZ() do { if (aggr) { if (wid < 4) { if (PS2A) __builtin_amdgcn_s_sleep(PS2A); } else { if (PS2B) __builtin_amdgcn_s_sleep(PS2B); } } else { if (PS2) __builtin_amdgcn_s_sleep(PS2); } } while (0)
#define PY() do { if (PS1 && !aggr) __builtin_amdgcn_s_sleep(PS1); } while (0)
#define STEP4(G0, J0) do { STEPO((G0), (J0)); STEPO((G0) + 1, (J0) + 1); STEPO((G0) + 2, (J0) + 2); STEPO((G0) + 3, (J0) + 3); SB(); } while (0)
#define BLOCK1(B) do { BBASE(B); \
        BWAIT(4 * (B));     PY(); STEP4(16 * (B), 0); \
        BWAIT(4 * (B) + 1); PY(); STEP4(16 * (B) + 4, 4); \
        BWAIT(4 * (B) + 2); PY(); STEP4(16 * (B) + 8, 8); \
        BWAIT(4 * (B) + 3); PY(); STEP4(16 * (B) + 12, 12); } while (0)
#define BLOCK2(B) do { BBASE(B); \
        PZ(); STEP4(64 + 16 * (B), 0); PZ(); STEP4(64 + 16 * (B) + 4, 4); PZ(); STEP4(64 + 16 * (B) + 8, 8); PZ(); STEP4(64 + 16 * (B) + 12, 12); } while (0)

#define SK 16
#define SK2 1
    if (aggr && wid >= 2 && wid < 4) BWAIT(SK2 - 1);
    if (LSL > 2 && aggr && wid >= 4) {
        unsigned long long t0 = rtc();
        while (true) {
            BPOLL();
            if (seen > (unsigned)(SK - 1)) break;
            if (rtc() - t0 > DEADLINE_TICKS) { bad = true; seen = 64; break; }
            __builtin_amdgcn_s_sleep(LSL);
        }
    }
    if (aggr && wid >= 4) BWAIT(SK - 1);
#define WG 2
#define WGW 4
#define GATE() do { \
        unsigned long long t0 = __builtin_amdgcn_s_memrealtime(); \
        while (true) { \
            unsigned c; LDSRD(c, &sv[7]); \
            if (__builtin_amdgcn_readfirstlane(c) >= (unsigned)WG) break; \
            if (__builtin_amdgcn_s_memrealtime() - t0 > DEADLINE_TICKS) { bad = true; break; } \
            __builtin_amdgcn_s_sleep(LSL); \
        } } while (0)
    if (GMODE == 1 && WG && aggr && wid >= WGW) GATE();
    BWAIT(0);
    BLOCK1(0); BLOCK1(1); BLOCK1(2); BLOCK1(3);
    RST(2);
    char* mypart = part + (size_t)((grp * 4 + q) * 8 + wid) * 16384 + lane * 16;
    const unsigned tag = call & 1u;
#pragma unroll
    for (int G = 0; G < 4; ++G) {
        if (OWNL && G == q) {
#pragma unroll
            for (int t = 0; t < 2; ++t) {
                f32x4 v; v[0] = acc[t][4 * G]; v[1] = acc[t][4 * G + 1]; v[2] = acc[t][4 * G + 2]; v[3] = acc[t][4 * G + 3];
                *(f32x4*)(Ts[wid] + t * 256 + lane * 4) = v;
            }
        } else {
#pragma unroll
            for (int t = 0; t < 2; ++t) {
                f32x4 v; v[0] = TAGGED(acc[t][4 * G]); v[1] = TAGGED(acc[t][4 * G + 1]); v[2] = TAGGED(acc[t][4 * G + 2]); v[3] = TAGGED(acc[t][4 * G + 3]);
                GSTOREC(mypart + G * 2048 + t * 1024, v);
            }
        }
    }
    asm volatile("s_waitcnt lgkmcnt(0)" ::: "memory");
    if (lane == 0) *(volatile unsigned*)&sv[8 + wid] = 1u;
    if (FM && lane == 0) __hip_atomic_fetch_add(&sv[3], 1u, __ATOMIC_RELAXED, __HIP_MEMORY_SCOPE_WORKGROUP);
#pragma unroll
    for (int t = 0; t < 2; ++t)
#pragma unroll
        for (int i = 0; i < 16; ++i) acc[t][i] = 0.f;

    if (GMODE == 2 && WG && wid < 4) GATE();
    BLOCK2(0); BLOCK2(1); BLOCK2(2); BLOCK2(3);
    RST(3);

    f32x4 own[2];
    {
#pragma unroll
        for (int G = 0; G < 4; ++G) {
            if (G == q) {
#pragma unroll
                for (int t = 0; t < 2; ++t) { own[t][0] = acc[t][4 * G]; own[t][1] = acc[t][4 * G + 1]; own[t][2] = acc[t][4 * G + 2]; own[t][3] = acc[t][4 * G + 3]; }
            } else {
                char* d = mypart + 8192 + ((G - q - 1) & 3) * 2048;
#pragma unroll
                for (int t = 0; t < 2; ++t) {
                    f32x4 v; v[0] = TAGGED(acc[t][4 * G]); v[1] = TAGGED(acc[t][4 * G + 1]); v[2] = TAGGED(acc[t][4 * G + 2]); v[3] = TAGGED(acc[t][4 * G + 3]);
                    GSTOREC(d + t * 1024, v);
                }
            }
        }
        SXT(0);
        u32x4 pv[6];
        unsigned pend = 7u;
        unsigned long long t0 = __builtin_amdgcn_s_memrealtime();
        if (CDELAY) __builtin_amdgcn_s_sleep(CDELAY);
#define CPOLL1 8
#define CPC 0
        if ((aggr || CPC) && wid < CPOLL1) {
            const int p = (q + 1 + (lane & 3)) & 3;
            const unsigned po = (unsigned)(((grp * 4 + p) * 8 + wid) * 16384 + 8192 + ((q - p - 1) & 3) * 2048) + pw;
            while (true) {
                unsigned fv = tag;
                if (lane < 3) fv = __builtin_amdgcn_raw_buffer_load_b32(prs, po, 0, 17);
                if (__builtin_amdgcn_ballot_w64(((fv ^ tag) & 1u) != 0u) == 0ull) break;
                if (__builtin_amdgcn_s_memrealtime() - t0 > DEADLINE_TICKS) { bad = true; break; }
                __builtin_amdgcn_s_sleep(FSLEEP);
                asm volatile("" ::: "memory");
            }
        }
        while (true) {
#pragma unroll
            for (int d = 0; d < 3; ++d)
                if (pend & (1u << d)) {
                    const int p = (q + 1 + d) & 3;
                    const unsigned o = (unsigned)(((grp * 4 + p) * 8 + wid) * 16384 + 8192 + ((q - p - 1) & 3) * 2048 + lane * 16);
                    pv[2 * d] = __builtin_amdgcn_raw_buffer_load_b128(prs, o, 0, 17);
                    pv[2 * d + 1] = __builtin_amdgcn_raw_buffer_load_b128(prs, o + 1024u, 0, 17);
                }
#pragma unroll
            for (int d = 0; d < 3; ++d)
                if (pend & (1u << d)) {
                    if (TAGOK(pv[2 * d], pv[2 * d + 1])) pend &= ~(1u << d);
                }
            if (pend == 0u) break;
            if (__builtin_amdgcn_s_memrealtime() - t0 > DEADLINE_TICKS) { bad = true; break; }
            __builtin_amdgcn_s_sleep(CSLEEP);
            asm volatile("" ::: "memory");
        }
        SXT(1); SXT(2);
#pragma unroll
        for (int t = 0; t < 2; ++t) own[t] = (own[t] + AF(pv[t])) + (AF(pv[2 + t]) + AF(pv[4 + t]));
    }
    if (OWNL) {
        unsigned long long t0 = __builtin_amdgcn_s_memrealtime();
        while (true) {
            unsigned c; LDSRD(c, &sv[8 + wid]);
            if (__builtin_amdgcn_readfirstlane(c) == 2u) break;
            if (__builtin_amdgcn_s_memrealtime() - t0 > DEADLINE_TICKS) { bad = true; break; }
            __builtin_amdgcn_s_sleep(1);
        }
    }
    RST(4);
    finish_unit<PACE1>(own[0], own[1], Ts[wid], dst + (size_t)(64 * RG(wid) + 32 + 8 * q) * NLOC, lane, bad, &sv[4], aggr ? (wid < 4 ? NT1 : NT2) : NTOK);
    asm volatile("s_waitcnt vmcnt(0)" ::: "memory");
    SXT(3);
    __syncthreads();
    RST(5);
    if (tid == 0) __hip_atomic_fetch_add(ws_word(myws, 0), 1u, __ATOMIC_RELAXED, __HIP_MEMORY_SCOPE_AGENT);
    if (tid == 0) {
        if (peer != rank) {
            __builtin_amdgcn_fence(__ATOMIC_RELEASE, "");
            asm volatile("s_waitcnt vmcnt(0)" ::: "memory");
            __hip_atomic_fetch_add(ws_word(ws_all[peer], 2 + rank), 1u, __ATOMIC_RELAXED, __HIP_MEMORY_SCOPE_SYSTEM);
        }
        if (b == 0) {
            unsigned need = call * (unsigned)BLOCKS_PER_PEER;
            unsigned long long t0 = __builtin_amdgcn_s_memrealtime();
            bool timed_out = false;
            if (AGG == 2) __hip_atomic_store((unsigned long long*)ws_word(myws, 12), t0, __ATOMIC_RELAXED, __HIP_MEMORY_SCOPE_AGENT);
            for (int p = 0; p < NR; ++p) {
                if (p == rank) continue;
                while (__hip_atomic_load(ws_word(myws, 2 + p), __ATOMIC_RELAXED, __HIP_MEMORY_SCOPE_SYSTEM) < need) {
                    if (__builtin_amdgcn_s_memrealtime() - t0 > DEADLINE_TICKS) { timed_out = true; break; }
                    __builtin_amdgcn_s_sleep(2);
                }
            }
            __builtin_amdgcn_fence(__ATOMIC_ACQUIRE, "");
            if (timed_out) {
                __hip_atomic_store(ws_word(myws, 1), call, __ATOMIC_RELAXED, __HIP_MEMORY_SCOPE_AGENT);
                ((float*)out_all[rank])[(size_t)(rank * MLOC) * NLOC] = __builtin_nanf("");
            }
        }
    }
}

extern "C" void kernel_launch(int rank, int nranks, ncclComm_t comm,
                              hipStream_t stream,
                              void* const* d_in, const int* in_sizes, int n_in,
                              void* d_out, int out_size,
                              const int* in_dtypes, int out_dtype,
                              void* const* d_out_all,
                              void* const* d_ws_all, size_t ws_bytes) {
    const float* x = (const float*)d_in[0];
    const float* w = (const float*)d_in[1];
    hipLaunchKernelGGL(gemm_a2a_kernel, dim3(256), dim3(NBLOCK), 0, stream,
                       x, w, d_out_all, d_ws_all, rank);
}
```

```cpp
#include <hip/hip_runtime.h>
#include <rccl/rccl.h>
#include <cstdio>
#include <cstdint>

typedef _Float16 half8 __attribute__((ext_vector_type(8)));
typedef _Float16 half4 __attribute__((ext_vector_type(4)));
typedef float f32x16 __attribute__((ext_vector_type(16)));
typedef float f32x4 __attribute__((ext_vector_type(4)));
typedef unsigned u32x4 __attribute__((ext_vector_type(4)));
typedef short s16x4 __attribute__((ext_vector_type(4)));
typedef short s16x8 __attribute__((ext_vector_type(8)));

constexpr int NR = 8;
constexpr int MLOC = 512;
constexpr int KDIM = 4096;
constexpr int NDIM = 4096;
constexpr int NLOC = 512;
constexpr int BN = 64;
constexpr int KS = 1024;
constexpr int NCW = 8;
constexpr int NPW = 4;
constexpr int NBLOCK = (NCW + NPW) * 64;
constexpr int BLOCKS_PER_PEER = 32;
constexpr unsigned long long DEADLINE_TICKS = 40000000ull;
constexpr size_t XFLAG_OFF = 8192;
constexpr size_t FLAG_OFF = 65536;
constexpr size_t RBS = 262144;
constexpr size_t X16_OFF = 262144;
constexpr size_t PART_OFF = X16_OFF + (5u << 20);

__device__ __forceinline__ unsigned* ws_word(void* ws, int idx) { return (unsigned*)((char*)ws + 128 * idx); }

__device__ __forceinline__ unsigned long long rtc() {
    unsigned long long t;
    asm volatile("s_memrealtime %0\n\ts_waitcnt lgkmcnt(0)" : "=s"(t) :: "memory");
    return t;
}

__device__ __forceinline__ float gelu_f(float y) {
    const float c = 0.7978845608028654f;
    float u = c * (y + 0.044715f * y * y * y);
    return y / (1.0f + __expf(-2.0f * u));
}

#define RST(I)
#define XV 0
#define XDELAY 350
#define XDELAYA XDELAY
#define FSLEEP 8
#define CSLEEP 4
#define XSLEEP 16
#define GLOAD(DST, VOFF, SBASE) asm volatile("global_load_dwordx4 %0, %1, %2" : "=v"(DST) : "v"(VOFF), "s"(SBASE))
#define GLOADNT(DST, VOFF, SBASE) asm volatile("global_load_dwordx4 %0, %1, %2 nt" : "=v"(DST) : "v"(VOFF), "s"(SBASE))
#define GLOADC(DST, P) asm volatile("global_load_dwordx4 %0, %1, off sc0 sc1" : "=v"(DST) : "v"(P) : "memory")
#define GSTOREC(P, V) asm volatile("global_store_dwordx4 %0, %1, off sc0 sc1\n\ts_nop 1" :: "v"(P), "v"(V) : "memory")
#define ASC 0
#define GSTOREA(P, V) GSTOREC(P, V)
#define GSTOREX(P, V) GSTOREC(P, V)
#define LAUX ((ASC & 2) ? 16 : 17)
#define LDSRD(DST, P) asm volatile("ds_read_b32 %0, %1\n\ts_waitcnt lgkmcnt(0)" : "=v"(DST) : "v"((unsigned)(uintptr_t)(__attribute__((address_space(3))) unsigned*)(P)) : "memory")

#define AF(V) __builtin_bit_cast(f32x4, V)
#define TAGOK(A, B) (__builtin_amdgcn_ballot_w64((((((A)[0] ^ tag) | ((A)[1] ^ tag)) | (((A)[2] ^ tag) | ((A)[3] ^ tag)) | (((B)[0] ^ tag) | ((B)[1] ^ tag)) | (((B)[2] ^ tag) | ((B)[3] ^ tag))) & 1u) != 0u) == 0ull)
__device__ __forceinline__ float tagged(float f, unsigned tag) { return __uint_as_float((__float_as_uint(f) & ~1u) | tag); }
#define TAGGED(F) tagged(F, tag)
#define FDELAY 0
#define CDELAY 0
#define PF 0
#define CPOLL 1
#define OWNL 1
#define NST (OWNL ? 6 : 8)
#define AGG 2
#define AGT 8000
#define TSL 1
#define XSL 1
#define LSL 2
#define RD 12
#define PACE0 3
#define PACE1 3
#define NTOK 3
#define NT0 3
#define NT0B NT0
#define NT1 3
#define NT2 2
#define GMODE 1
#define PU 0
#define PW 4
template <int PACE>
__device__ __forceinline__ void finish_unit(const f32x4& v0, const f32x4& v1, float* T, float* dst, int lane, bool bad, unsigned* tk, unsigned ntok) {
    const int fh = lane >> 5;
#pragma unroll
    for (int i = 0; i < 4; ++i) {
        float g0 = gelu_f(v0[i]), g1 = gelu_f(v1[i]);
        T[(4 * fh + i) * 68 + (lane & 31)] = bad ? __builtin_nanf("") : g0;
        T[(4 * fh + i) * 68 + 32 + (lane & 31)] = bad ? __builtin_nanf("") : g1;
    }
    __builtin_amdgcn_wave_barrier();
#pragma unroll
    for (int j = 0; j < 2; ++j) {
        const int r = 4 * j + (lane >> 4), c4 = lane & 15;
        f32x4 v = *(const f32x4*)(T + r * 68 + c4 * 4);
        if (PACE == 1) asm volatile("s_waitcnt vmcnt(0)" ::: "memory");
        if (PACE == 2) asm volatile("s_waitcnt vmcnt(1)" ::: "memory");
        if (PACE == 3) {
            unsigned t = 0;
            if (lane == 0) t = __hip_atomic_fetch_add(tk, 1u, __ATOMIC_RELAXED, __HIP_MEMORY_SCOPE_WORKGROUP);
            t = __builtin_amdgcn_readfirstlane(t);
            unsigned long long t0 = __builtin_amdgcn_s_memrealtime();
            while (true) {
                unsigned d; LDSRD(d, tk + 1);
                if (t < __builtin_amdgcn_readfirstlane(d) + ntok) break;
                if (__builtin_amdgcn_s_memrealtime() - t0 > DEADLINE_TICKS) break;
                __builtin_amdgcn_s_sleep(TSL);
            }
        }
        GSTOREC(dst + (size_t)r * NLOC + c4 * 4, v);
        if (PACE == 3) {
            asm volatile("s_waitcnt vmcnt(0)" ::: "memory");
            if (lane == 0) __hip_atomic_fetch_add(tk + 1, 1u, __ATOMIC_RELAXED, __HIP_MEMORY_SCOPE_WORKGROUP);
        }
    }
    __builtin_amdgcn_wave_barrier();
}

__global__ __launch_bounds__(NBLOCK, 3)
void gemm_a2a_kernel(const float* __restrict__ x, const float* __restrict__ w,
                     void* const* __restrict__ out_all, void* const* __restrict__ ws_all, int rank) {
    __shared__ __attribute__((aligned(16))) _Float16 Bs[KS * BN];
    __shared__ __attribute__((aligned(16))) float Ts[NCW + NPW][8 * 68];
    __shared__ __attribute__((aligned(16))) unsigned rdy[4];
    __shared__ __attribute__((aligned(16))) unsigned sv[16];

#define RXT(I)
#define SXT(I)
    RST(0);
    const int tid = threadIdx.x;
    const int lane = tid & 63, wid = __builtin_amdgcn_readfirstlane(tid >> 6);
#define HOIST 1
    f32x4 v[4];
    {
        const int b_ = blockIdx.x, q_ = b_ & 3, g_ = ((b_ >> 2) & 1) * 32 + (b_ >> 3);
        const float* src = x + (size_t)(32 * (g_ >> 2) + (lane & 31)) * KDIM + (q_ * 64 + (g_ & 3) * 16 + 2 * (wid & 7)) * 16 + 8 * (lane >> 5);
        if (wid < NCW) {
#pragma unroll
            for (int i = 0; i < 4; ++i)
                asm volatile("global_load_dwordx4 %0, %1, off nt" : "=v"(v[i]) : "v"(src + (i >> 1) * 16 + (i & 1) * 4));
        }
        __builtin_amdgcn_sched_barrier(0);
    }
    char* myws;
    {
        unsigned long long a = (unsigned long long)ws_all[rank];
        unsigned lo = __builtin_amdgcn_readfirstlane((unsigned)a), hi = __builtin_amdgcn_readfirstlane((unsigned)(a >> 32));
        myws = (char*)(((unsigned long long)hi << 32) | lo);
    }
    char* x16 = myws + X16_OFF;
    char* part = myws + PART_OFF;
    const __amdgpu_buffer_rsrc_t prs = __builtin_amdgcn_make_buffer_rsrc(part, 0, 32u << 20, 0x00020000);
    const int b = blockIdx.x;
#define MAP 0
    const int q = b & 3;
    const int grp = ((b >> 2) & 1) * 32 + (b >> 3);
    const int peer = ((grp >> 3) + rank) & 7;
    const int ct = grp & 7;
    const int ncol0 = peer * NLOC + ct * BN;
    float* dst = (float*)out_all[peer] + (size_t)(rank * MLOC) * NLOC + ct * BN;
    bool bad = false;
#define ROT 0
#define RG(W) (ROT ? (((W) + rank) & 7) : (W))
#define PV 1
    const unsigned pw = PV == 1 ? 1024u + (unsigned)((wid + grp + lane) & 7) * 128u + 124u : PV == 2 ? (unsigned)((wid + grp + lane) & 15) * 128u + 124u : 2044u;

    if (wid >= NCW) {
        const int p = wid - NCW;
        const int kr = lane >> 4, nq = lane & 15;
        const unsigned voff = (unsigned)((4 * p + kr) * NDIM + nq * 4) * 4u;
        const char* base = (const char*)(w + (size_t)q * KS * NDIM + ncol0);
#define PD 2
        f32x4 rb[4 * PD];
#define PLD(SLOT, S) _Pragma("unroll") for (int i = 0; i < 4; ++i) GLOADNT(rb[(SLOT) * 4 + i], voff, base + (size_t)((S) * 64 + 16 * i) * NDIM * 4)
#define PWAIT(N, SLOT) asm volatile("s_waitcnt vmcnt(" #N ")" : "+v"(rb[(SLOT) * 4]), "+v"(rb[(SLOT) * 4 + 1]), "+v"(rb[(SLOT) * 4 + 2]), "+v"(rb[(SLOT) * 4 + 3]))
#define PST(SLOT, S) do { \
        _Pragma("unroll") for (int i = 0; i < 4; ++i) { \
            f32x4 v = rb[(SLOT) * 4 + i]; \
            half4 h; h[0] = (_Float16)v[0]; h[1] = (_Float16)v[1]; h[2] = (_Float16)v[2]; h[3] = (_Float16)v[3]; \
            *(half4*)(Bs + ((S) * 64 + 16 * i + 4 * p + kr) * BN + ((nq * 4) ^ (32 * ((kr >> 1) & 1)))) = h; \
        } \
        asm volatile("" ::: "memory"); \
        if (lane == 0) *(volatile unsigned*)&rdy[p] = (unsigned)(S) + 1u; \
        asm volatile("" ::: "memory"); } while (0)
#define BDELAY 1
#define PCALL 1
        unsigned old = 0;
        unsigned long long last = 0;
        if (p == 0) {
            asm volatile("global_load_dword %0, %1, off sc1" : "=v"(old) : "v"(ws_word(myws, 0)) : "memory");
            asm volatile("global_load_dwordx2 %0, %1, off sc1" : "=v"(last) : "v"(ws_word(myws, 12)) : "memory");
        }
        if (lane == 0) rdy[p] = 0;
        __syncthreads();
        {
            unsigned long long t0 = __builtin_amdgcn_s_memrealtime();
            while (true) {
                unsigned c; LDSRD(c, &sv[0]);
                if (__builtin_amdgcn_readfirstlane(c) >= (unsigned)NCW) break;
                if (__builtin_amdgcn_s_memrealtime() - t0 > DEADLINE_TICKS) break;
                __builtin_amdgcn_s_sleep(1);
            }
        }
        _Pragma("unroll") for (int s = (BDELAY == 2 ? 1 : 0); s < PD; ++s) { PLD(s, s); }
        static_assert(PD == 2, "wait count below");
        if (p == 0) {
            asm volatile("s_waitcnt vmcnt(8)" : "+v"(old), "+v"(last));
            if (lane == 0) *(volatile unsigned*)&sv[2] = (((old >> 8) + 1u) & 0x3fffffffu) | ((AGG == 1 || (AGG == 2 && __builtin_amdgcn_s_memrealtime() - last > (unsigned long long)AGT)) ? 0x40000000u : 0u);
        }
#pragma unroll
        for (int s = 0; s < 16; ++s) {
            const int left = (15 - s < PD - 1) ? 15 - s : PD - 1;
            if (left == 5) PWAIT(20, s % PD); else if (left == 4) PWAIT(16, s % PD); else if (left == 3) PWAIT(12, s % PD); else if (left == 2) PWAIT(8, s % PD); else if (left == 1) PWAIT(4, s % PD); else PWAIT(0, s % PD);
            PST(s % PD, s);
            if ((s & 3) == 0) RXT(4 + (s >> 2));
            if (s + PD < 16) PLD(s % PD, s + PD);
        }
        RST(3);
        unsigned call;
        unsigned long long t0 = __builtin_amdgcn_s_memrealtime();
        while (true) {
            LDSRD(call, &sv[1]);
            if (call != 0u) break;
            if (__builtin_amdgcn_s_memrealtime() - t0 > DEADLINE_TICKS) { call = 0x80000000u; break; }
            __builtin_amdgcn_s_sleep(LSL);
        }
        call = __builtin_amdgcn_readfirstlane(call);
        if (call & 0x80000000u) bad = true;
        const bool aggr = (call & 0x40000000u) != 0u;
        call &= 0x3fffffffu;
        const unsigned tag = call & 1u;
#define FM 3
#define MAPF 1
        if (FM != 4 || !aggr) {
        const int w0 = MAPF ? 2 * p : p;
        constexpr int WS = MAPF ? 1 : 4;
        while (true) {
            unsigned c, c2 = 1;
            if (aggr) { LDSRD(c, &sv[8 + w0]); LDSRD(c2, &sv[8 + w0 + WS]); } else { LDSRD(c, &sv[3]); }
            if (__builtin_amdgcn_readfirstlane(c) >= (aggr ? 1u : (unsigned)NCW) && __builtin_amdgcn_readfirstlane(c2) != 0u) break;
            if (__builtin_amdgcn_s_memrealtime() - t0 > DEADLINE_TICKS) { bad = true; break; }
            __builtin_amdgcn_s_sleep(LSL);
        }
        f32x4 ow[4];
#pragma unroll
        for (int j = 0; j < 4; ++j) ow[j] = *(const f32x4*)(Ts[w0 + (j >> 1) * WS] + (j & 1) * 256 + lane * 4);
        asm volatile("s_waitcnt lgkmcnt(0)" : "+v"(ow[0]), "+v"(ow[1]), "+v"(ow[2]), "+v"(ow[3]) :: "memory");
        if (lane < 2) *(volatile unsigned*)&sv[8 + w0 + lane * WS] = 2u;
        const unsigned voff0 = (unsigned)((grp * 4 * 8 + w0) * 16384 + q * 2048 + lane * 16);
        u32x4 pv[12];
        unsigned pend = 0x3fu;
#define FCP0 0
        if (!(aggr && p < FCP0)) {
            const unsigned po = (unsigned)((grp * 4 * 8 + w0) * 16384 + q * 2048) + pw + (unsigned)((((q + 1 + (lane & 3)) & 3) * 8 + ((lane >> 2) & 1) * WS) * 16384);
            while (true) {
                unsigned fv = tag;
                if (lane < 8 && (lane & 3) != 3) fv = __builtin_amdgcn_raw_buffer_load_b32(prs, po, 0, LAUX);
                if (__builtin_amdgcn_ballot_w64(((fv ^ tag) & 1u) != 0u) == 0ull) break;
                if (__builtin_amdgcn_s_memrealtime() - t0 > DEADLINE_TICKS) { bad = true; break; }
                __builtin_amdgcn_s_sleep(FSLEEP);
                asm volatile("" ::: "memory");
            }
        }
        while (true) {
#pragma unroll
            for (int i = 0; i < 6; ++i)
                if (pend & (1u << i)) {
                    const unsigned o = voff0 + (unsigned)((((q + 1 + i % 3) & 3) * 8 + (i / 3) * WS) * 16384);
                    pv[2 * i] = __builtin_amdgcn_raw_buffer_load_b128(prs, o, 0, LAUX);
                    pv[2 * i + 1] = __builtin_amdgcn_raw_buffer_load_b128(prs, o + 1024u, 0, LAUX);
                }
#pragma unroll
            for (int i = 0; i < 6; ++i)
                if (pend & (1u << i)) {
                    if (TAGOK(pv[2 * i], pv[2 * i + 1])) pend &= ~(1u << i);
                }
            if (pend == 0u) break;
            if (__builtin_amdgcn_s_memrealtime() - t0 > DEADLINE_TICKS) { bad = true; break; }
            __builtin_amdgcn_s_sleep(FSLEEP);
            asm volatile("" ::: "memory");
        }
        if (lane == 0) __hip_atomic_fetch_add(&sv[7], 1u, __ATOMIC_RELAXED, __HIP_MEMORY_SCOPE_WORKGROUP);
        {
            f32x4 v0 = (ow[0] + AF(pv[0])) + (AF(pv[2]) + AF(pv[4])), v1 = (ow[1] + AF(pv[1])) + (AF(pv[3]) + AF(pv[5]));
            finish_unit<PACE0>(v0, v1, Ts[wid], dst + (size_t)(64 * RG(w0) + 8 * q) * NLOC, lane, bad, &sv[4], aggr ? (p < 2 ? NT0 : NT0B) : NTOK);
        }
        {
            f32x4 v0 = (ow[2] + AF(pv[6])) + (AF(pv[8]) + AF(pv[10])), v1 = (ow[3] + AF(pv[7])) + (AF(pv[9]) + AF(pv[11]));
            finish_unit<PACE0>(v0, v1, Ts[wid], dst + (size_t)(64 * RG(w0 + WS) + 8 * q) * NLOC, lane, bad, &sv[4], aggr ? (p < 2 ? NT0 : NT0B) : NTOK);
        }
        }
#pragma unroll 1
        for (int u = 0; u < ((FM == 1 || FM == 3 || (FM == 4 && !aggr)) ? 0 : 2); ++u) {
            const int wv = p + 4 * u;
            while (true) {
                unsigned c; LDSRD(c, &sv[8 + wv]);
                if (__builtin_amdgcn_readfirstlane(c) != 0u) break;
                if (__builtin_amdgcn_s_memrealtime() - t0 > DEADLINE_TICKS) { bad = true; break; }
                __builtin_amdgcn_s_sleep(LSL);
            }
            const unsigned voff0 = (unsigned)((grp * 4 * 8 + wv) * 16384 + q * 2048 + lane * 16);
            u32x4 pv[8];
            unsigned pend = 0xfu;
            {
                const unsigned po = (unsigned)((grp * 4 * 8 + wv) * 16384 + q * 2048) + pw + (unsigned)((lane & 3) * 8 * 16384);
                while (true) {
                    unsigned fv = tag;
                    if (lane < 4) fv = __builtin_amdgcn_raw_buffer_load_b32(prs, po, 0, LAUX);
                    if (__builtin_amdgcn_ballot_w64(((fv ^ tag) & 1u) != 0u) == 0ull) break;
                    if (__builtin_amdgcn_s_memrealtime() - t0 > DEADLINE_TICKS) { bad = true; break; }
                    __builtin_amdgcn_s_sleep(FSLEEP);
                    asm volatile("" ::: "memory");
                }
            }
            if (u == 0) SXT(4);
            if (GMODE == 3 && u <= PU && lane == 0) __hip_atomic_fetch_add(&sv[6], 1u, __ATOMIC_RELAXED, __HIP_MEMORY_SCOPE_WORKGROUP);
            while (true) {
#pragma unroll
                for (int i = 0; i < 4; ++i)
                    if (pend & (1u << i)) {
                        const unsigned o = voff0 + (unsigned)(i * 8 * 16384);
                        pv[2 * i] = __builtin_amdgcn_raw_buffer_load_b128(prs, o, 0, LAUX);
                        pv[2 * i + 1] = __builtin_amdgcn_raw_buffer_load_b128(prs, o + 1024u, 0, LAUX);
                    }
#pragma unroll
                for (int i = 0; i < 4; ++i)
                    if (pend & (1u << i)) {
                        if (TAGOK(pv[2 * i], pv[2 * i + 1])) pend &= ~(1u << i);
                    }
                if (pend == 0u) break;
                if (__builtin_amdgcn_s_memrealtime() - t0 > DEADLINE_TICKS) { bad = true; break; }
                __builtin_amdgcn_s_sleep(FSLEEP);
                asm volatile("" ::: "memory");
            }
            if (u == 0) SXT(5); else SXT(7);
            if (u == 0 && lane == 0) __hip_atomic_fetch_add(&sv[7], 1u, __ATOMIC_RELAXED, __HIP_MEMORY_SCOPE_WORKGROUP);
            if (GMODE == 3 && u <= PU && lane == 0) __hip_atomic_fetch_add(&sv[6], ~0u, __ATOMIC_RELAXED, __HIP_MEMORY_SCOPE_WORKGROUP);
            f32x4 v0 = (AF(pv[0]) + AF(pv[2])) + (AF(pv[4]) + AF(pv[6])), v1 = (AF(pv[1]) + AF(pv[3])) + (AF(pv[5]) + AF(pv[7]));
            finish_unit<PACE0>(v0, v1, Ts[wid], dst + (size_t)(64 * RG(wv) + 8 * q) * NLOC, lane, bad, &sv[4], aggr ? (p < 2 ? NT0 : NT0B) : NTOK);
            if (u == 0) SXT(6);
        }
        asm volatile("s_waitcnt vmcnt(0)" ::: "memory");
        __syncthreads();
        return;
    }

    unsigned call;
    bool aggr;
    {
        unsigned old = 0;
        if (tid < 16) sv[tid] = 0;
        const int rbk = grp >> 2, ks = q * 64 + (grp & 3) * 16 + 2 * wid;
        __syncthreads();
        unsigned long long last = 0;
        if (!PCALL && tid == 0) {
            old = __hip_atomic_load(ws_word(myws, 0), __ATOMIC_RELAXED, __HIP_MEMORY_SCOPE_AGENT);
            if (AGG == 2) last = __hip_atomic_load((unsigned long long*)ws_word(myws, 12), __ATOMIC_RELAXED, __HIP_MEMORY_SCOPE_AGENT);
        }
        RXT(0);
        asm volatile("s_waitcnt vmcnt(0)" : "+v"(v[0]), "+v"(v[1]), "+v"(v[2]), "+v"(v[3]));
        RXT(1);
        if (lane == 0) __hip_atomic_fetch_add(&sv[0], 1u, __ATOMIC_RELAXED, __HIP_MEMORY_SCOPE_WORKGROUP);
        if (!PCALL && tid == 0) *(volatile unsigned*)&sv[2] = (((old >> 8) + 1u) & 0x3fffffffu) | ((AGG == 1 || (AGG == 2 && __builtin_amdgcn_s_memrealtime() - last > (unsigned long long)AGT)) ? 0x40000000u : 0u);
#pragma unroll
        for (int i = 0; i < 2; ++i) {
            half8 h;
#pragma unroll
            for (int e = 0; e < 4; ++e) { h[e] = (_Float16)v[2 * i][e]; h[4 + e] = (_Float16)v[2 * i + 1][e]; }
            GSTOREX(x16 + (size_t)rbk * RBS + (size_t)((ks + i) * 64 + lane) * 16, __builtin_bit_cast(f32x4, h));
        }
        unsigned long long t0 = __builtin_amdgcn_s_memrealtime();
        while (true) {
            LDSRD(call, &sv[2]);
            if (call != 0u) break;
            if (__builtin_amdgcn_s_memrealtime() - t0 > DEADLINE_TICKS) { bad = true; break; }
            __builtin_amdgcn_s_sleep(1);
        }
        call = __builtin_amdgcn_readfirstlane(call);
        aggr = (call & 0x40000000u) != 0u;
        call &= 0x3fffffffu;
        RXT(2);
        asm volatile("s_waitcnt vmcnt(0)" ::: "memory");
        RXT(3);
#define XF 1
#define XRS 4352
        {
            unsigned c = 0;
            if (lane == 0) c = __hip_atomic_fetch_add(&sv[6], 1u, __ATOMIC_RELAXED, __HIP_MEMORY_SCOPE_WORKGROUP);
            c = __builtin_amdgcn_readfirstlane(c);
            if (c == (unsigned)NCW - 1u && lane < 8)
                __hip_atomic_store((unsigned*)(myws + FLAG_OFF + (size_t)(q * 8 + lane) * XRS) + grp, call, __ATOMIC_RELAXED, __HIP_MEMORY_SCOPE_SYSTEM);
        }
        if (wid == 0) {
            const unsigned* xf = (const unsigned*)(myws + FLAG_OFF + (size_t)(q * 8 + (grp & 7)) * XRS) + lane;
            while (__builtin_amdgcn_s_memrealtime() - t0 < (aggr ? XDELAYA : XDELAY)) __builtin_amdgcn_s_sleep(4);
            while (true) {
                unsigned f = __hip_atomic_load(xf, __ATOMIC_RELAXED, __HIP_MEMORY_SCOPE_SYSTEM);
                if (__builtin_amdgcn_ballot_w64(f != call) == 0ull) break;
                if (__builtin_amdgcn_s_memrealtime() - t0 > DEADLINE_TICKS) { bad = true; break; }
                __builtin_amdgcn_s_sleep(XSLEEP);
            }
            asm volatile("" ::: "memory");
            if (lane == 0) *(volatile unsigned*)&sv[1] = call | (bad ? 0x80000000u : 0u) | (aggr ? 0x40000000u : 0u);
        } else {
            while (true) {
                LDSRD(call, &sv[1]);
                if (call != 0u) break;
                if (__builtin_amdgcn_s_memrealtime() - t0 > DEADLINE_TICKS) { call = 0x80000000u; break; }
                __builtin_amdgcn_s_sleep(XSL);
            }
            call = __builtin_amdgcn_readfirstlane(call);
            if (call & 0x80000000u) bad = true;
            call &= 0x3fffffffu;
        }
        asm volatile("s_waitcnt vmcnt(0) lgkmcnt(0)" ::: "memory");
    }
    RST(1);

    const unsigned a_voff = (unsigned)lane * 16u;
    const char* a_base = x16 + (size_t)(2 * RG(wid)) * RBS + (size_t)(64 * q) * 1024;
#define PFX 0
    if (PFX && wid >= 4) {
        f32x4 pf[6];
        const int c0 = ((b >> 3) * 4 + (wid - 4)) * 6;
#pragma unroll
        for (int i = 0; i < 6; ++i) {
            const int c = c0 + i, rbi = c >> 6, rbk = rbi < 4 ? 2 * rbi + 1 : rbi + 4;
            GLOAD(pf[i], a_voff, x16 + (size_t)rbk * RBS + (size_t)(64 * q + (c & 63)) * 1024);
        }
        asm volatile("s_waitcnt vmcnt(0)" : "+v"(pf[0]), "+v"(pf[1]), "+v"(pf[2]), "+v"(pf[3]), "+v"(pf[4]), "+v"(pf[5]));
    }
    f32x4 ra[RD];
#define AOFF(S) ((size_t)((S) >> 6) * RBS + (size_t)((S) & 63) * 1024)
#pragma unroll
    for (int i = 0; i < RD; ++i) GLOAD(ra[i], a_voff, a_base + AOFF(i));

    f32x16 acc[2];
#pragma unroll
    for (int t = 0; t < 2; ++t)
#pragma unroll
        for (int i = 0; i < 16; ++i) acc[t][i] = 0.f;

    const int fh = lane >> 5;
    const int bblk = (lane >> 4) & 1, bq = (lane & 15) >> 2, bp = lane & 3;
    const unsigned lds0 = (unsigned)(uintptr_t)(__attribute__((address_space(3))) _Float16*)Bs;
    const unsigned b0p = lds0 + 2u * ((8 * fh + bq) * BN + ((16 * bblk + 4 * bp) ^ (32 * ((bq >> 1) & 1))));
    const unsigned b1p = lds0 + 2u * ((8 * fh + bq) * BN + ((32 + 16 * bblk + 4 * bp) ^ (32 * ((bq >> 1) & 1))));
#define BBASE(BLK) unsigned bs0 = b0p + (BLK) * 512u * BN, bs1 = b1p + (BLK) * 512u * BN; asm volatile("" : "+v"(bs0), "+v"(bs1))
    unsigned seen = 0;
#define BPOLL() do { \
        u32x4 rv; \
        asm volatile("ds_read_b128 %0, %1\n\ts_waitcnt lgkmcnt(0)" : "=v"(rv) : "v"((unsigned)(uintptr_t)(__attribute__((address_space(3))) unsigned*)rdy) : "memory"); \
        seen = __builtin_amdgcn_readfirstlane(min(min(rv[0], rv[1]), min(rv[2], rv[3]))); } while (0)
#define BWAIT(S) do { \
        if (seen <= (unsigned)(S)) { \
            BPOLL(); \
            if (seen <= (unsigned)(S)) { \
                unsigned long long t0 = rtc(); \
                for (unsigned n = 1; ; ++n) { \
                    __builtin_amdgcn_s_sleep(2); \
                    BPOLL(); \
                    if (seen > (unsigned)(S)) break; \
                    if ((n & 255u) == 0u && rtc() - t0 > DEADLINE_TICKS) { bad = true; seen = 64; break; } \
                } \
            } \
        } } while (0)
#define TR(P) __builtin_amdgcn_ds_read_tr16_b64_v4i16((__attribute__((address_space(3))) s16x4*)(uintptr_t)(P))
#define LD 2
    s16x4 fb[LD + 1][4];
    unsigned bsl0 = b0p, bsl1 = b1p, bsh0 = b0p + 65536u, bsh1 = b1p + 65536u;
    asm volatile("" : "+v"(bsl0), "+v"(bsl1), "+v"(bsh0), "+v"(bsh1));
#define TRA(DST, P, O) asm volatile("ds_read_b64_tr_b16 %0, %1 offset:%2" : "=v"(DST) : "v"(P), "n"(O))
#define BRD(G) do { \
        if ((G) & 32) { \
            TRA(fb[(G) % (LD + 1)][0], bsh0, ((G) & 31) * 2048); TRA(fb[(G) % (LD + 1)][1], bsh0, ((G) & 31) * 2048 + 512); \
            TRA(fb[(G) % (LD + 1)][2], bsh1, ((G) & 31) * 2048); TRA(fb[(G) % (LD + 1)][3], bsh1, ((G) & 31) * 2048 + 512); \
        } else { \
            TRA(fb[(G) % (LD + 1)][0], bsl0, ((G) & 31) * 2048); TRA(fb[(G) % (LD + 1)][1], bsl0, ((G) & 31) * 2048 + 512); \
            TRA(fb[(G) % (LD + 1)][2], bsl1, ((G) & 31) * 2048); TRA(fb[(G) % (LD + 1)][3], bsl1, ((G) & 31) * 2048 + 512); \
        } } while (0)
#define STEP(G) do { \
        if ((G) + LD < 64 && (((G) + LD) & 3) == 0) BWAIT(((G) + LD) >> 2); \
        if ((G) + LD < 128) BRD((G) + LD); \
        asm volatile("s_waitcnt vmcnt(%5) lgkmcnt(%6)" : "+v"(ra[(G) % RD]), "+v"(fb[(G) % (LD + 1)][0]), "+v"(fb[(G) % (LD + 1)][1]), "+v"(fb[(G) % (LD + 1)][2]), "+v"(fb[(G) % (LD + 1)][3]) \
            : "n"(((G) > 127 - RD ? 127 - (G) : RD - 1) + ((G) >= 64 && (G) < 64 + RD ? NST : 0)), "n"(4 * ((G) + LD < 128 ? LD : 127 - (G)))); \
        half8 fb0 = __builtin_bit_cast(half8, __builtin_shufflevector(fb[(G) % (LD + 1)][0], fb[(G) % (LD + 1)][1], 0, 1, 2, 3, 4, 5, 6, 7)); \
        half8 fb1 = __builtin_bit_cast(half8, __builtin_shufflevector(fb[(G) % (LD + 1)][2], fb[(G) % (LD + 1)][3], 0, 1, 2, 3, 4, 5, 6, 7)); \
        half8 fa = __builtin_bit_cast(half8, ra[(G) % RD]); \
        acc[0] = __builtin_amdgcn_mfma_f32_32x32x16_f16(fa, fb0, acc[0], 0, 0, 0); \
        acc[1] = __builtin_amdgcn_mfma_f32_32x32x16_f16(fa, fb1, acc[1], 0, 0, 0); \
        if ((G) + RD < 128) GLOAD(ra[(G) % RD], a_voff, a_base + AOFF((G) + RD)); \
        __builtin_amdgcn_sched_barrier(0); \
    } while (0)
#define PAUSE() do { if (GMODE == 3 && wid >= PW) { \
        for (unsigned n = 0; n < 4096u; ++n) { \
            unsigned c; LDSRD(c, &sv[6]); \
            if (__builtin_amdgcn_readfirstlane(c) == 0u) break; \
            __builtin_amdgcn_s_sleep(1); \
        } } } while (0)
#define STEPV 0
#define STEPO(G, J) do { \
        s16x4 l0 = TR(bs0 + (J) * 32 * BN), h0 = TR(bs0 + ((J) * 32 + 8) * BN); \
        s16x4 l1 = TR(bs1 + (J) * 32 * BN), h1 = TR(bs1 + ((J) * 32 + 8) * BN); \
        half8 fb0 = __builtin_bit_cast(half8, __builtin_shufflevector(l0, h0, 0, 1, 2, 3, 4, 5, 6, 7)); \
        half8 fb1 = __builtin_bit_cast(half8, __builtin_shufflevector(l1, h1, 0, 1, 2, 3, 4, 5, 6, 7)); \
        asm volatile("s_waitcnt vmcnt(%1)" : "+v"(ra[(G) % RD]) : "n"(((G) > 127 - RD ? 127 - (G) : RD - 1) + ((G) >= 64 && (G) < 64 + RD ? NST : 0))); \
        half8 fa = __builtin_bit_cast(half8, ra[(G) % RD]); \
        acc[0] = __builtin_amdgcn_mfma_f32_32x32x16_f16(fa, fb0, acc[0], 0, 0, 0); \
        acc[1] = __builtin_amdgcn_mfma_f32_32x32x16_f16(fa, fb1, acc[1], 0, 0, 0); \
        if ((G) + RD < 128) GLOAD(ra[(G) % RD], a_voff, a_base + AOFF((G) + RD)); \
    } while (0)
#define SB() __builtin_amdgcn_sched_barrier(0)
#define PS2 16
#define PS1 0
#define PS2A 0
#define PS2B 0
#define PZ() do { if (aggr) { if (wid < 4) { if (PS2A) __builtin_amdgcn_s_sleep(PS2A); } else { if (PS2B) __builtin_amdgcn_s_sleep(PS2B); } } else { if (PS2) __builtin_amdgcn_s_sleep(PS2); } } while (0)
#define PY() do { if (PS1 && !aggr) __builtin_amdgcn_s_sleep(PS1); } while (0)
#define STEP4(G0, J0) do { STEPO((G0), (J0)); STEPO((G0) + 1, (J0) + 1); STEPO((G0) + 2, (J0) + 2); STEPO((G0) + 3, (J0) + 3); SB(); } while (0)
#define BLOCK1(B) do { BBASE(B); \
        BWAIT(4 * (B));     PY(); STEP4(16 * (B), 0); \
        BWAIT(4 * (B) + 1); PY(); STEP4(16 * (B) + 4, 4); \
        BWAIT(4 * (B) + 2); PY(); STEP4(16 * (B) + 8, 8); \
        BWAIT(4 * (B) + 3); PY(); STEP4(16 * (B) + 12, 12); } while (0)
#define BLOCK2(B) do { BBASE(B); \
        PZ(); STEP4(64 + 16 * (B), 0); PZ(); STEP4(64 + 16 * (B) + 4, 4); PZ(); STEP4(64 + 16 * (B) + 8, 8); PZ(); STEP4(64 + 16 * (B) + 12, 12); } while (0)

#define SK 16
#define SK2 1
    if (aggr && wid >= 2 && wid < 4) BWAIT(SK2 - 1);
    if (LSL > 2 && aggr && wid >= 4) {
        unsigned long long t0 = rtc();
        while (true) {
            BPOLL();
            if (seen > (unsigned)(SK - 1)) break;
            if (rtc() - t0 > DEADLINE_TICKS) { bad = true; seen = 64; break; }
            __builtin_amdgcn_s_sleep(LSL);
        }
    }
    if (aggr && wid >= 4) BWAIT(SK - 1);
#define WG 2
#define WGW 4
#define GATE() do { \
        unsigned long long t0 = __builtin_amdgcn_s_memrealtime(); \
        while (true) { \
            unsigned c; LDSRD(c, &sv[7]); \
            if (__builtin_amdgcn_readfirstlane(c) >= (unsigned)WG) break; \
            if (__builtin_amdgcn_s_memrealtime() - t0 > DEADLINE_TICKS) { bad = true; break; } \
            __builtin_amdgcn_s_sleep(LSL); \
        } } while (0)
    if (GMODE == 1 && WG && aggr && wid >= WGW) GATE();
    BWAIT(0);
    BLOCK1(0); BLOCK1(1); BLOCK1(2); BLOCK1(3);
    RST(2);
    char* mypart = part + (size_t)((grp * 4 + q) * 8 + wid) * 16384 + lane * 16;
    const unsigned tag = call & 1u;
#pragma unroll
    for (int G = 0; G < 4; ++G) {
        if (OWNL && G == q) {
#pragma unroll
            for (int t = 0; t < 2; ++t) {
                f32x4 v; v[0] = acc[t][4 * G]; v[1] = acc[t][4 * G + 1]; v[2] = acc[t][4 * G + 2]; v[3] = acc[t][4 * G + 3];
                *(f32x4*)(Ts[wid] + t * 256 + lane * 4) = v;
            }
        } else {
#pragma unroll
            for (int t = 0; t < 2; ++t) {
                f32x4 v; v[0] = TAGGED(acc[t][4 * G]); v[1] = TAGGED(acc[t][4 * G + 1]); v[2] = TAGGED(acc[t][4 * G + 2]); v[3] = TAGGED(acc[t][4 * G + 3]);
                GSTOREA(mypart + G * 2048 + t * 1024, v);
            }
        }
    }
    asm volatile("s_waitcnt lgkmcnt(0)" ::: "memory");
    if (lane == 0) *(volatile unsigned*)&sv[8 + wid] = 1u;
    if (FM && lane == 0) __hip_atomic_fetch_add(&sv[3], 1u, __ATOMIC_RELAXED, __HIP_MEMORY_SCOPE_WORKGROUP);
#pragma unroll
    for (int t = 0; t < 2; ++t)
#pragma unroll
        for (int i = 0; i < 16; ++i) acc[t][i] = 0.f;

    if (GMODE == 2 && WG && wid < 4) GATE();
    BLOCK2(0); BLOCK2(1); BLOCK2(2); BLOCK2(3);
    RST(3);

    f32x4 own[2];
    {
#pragma unroll
        for (int G = 0; G < 4; ++G) {
            if (G == q) {
#pragma unroll
                for (int t = 0; t < 2; ++t) { own[t][0] = acc[t][4 * G]; own[t][1] = acc[t][4 * G + 1]; own[t][2] = acc[t][4 * G + 2]; own[t][3] = acc[t][4 * G + 3]; }
            } else {
                char* d = mypart + 8192 + ((G - q - 1) & 3) * 2048;
#pragma unroll
                for (int t = 0; t < 2; ++t) {
                    f32x4 v; v[0] = TAGGED(acc[t][4 * G]); v[1] = TAGGED(acc[t][4 * G + 1]); v[2] = TAGGED(acc[t][4 * G + 2]); v[3] = TAGGED(acc[t][4 * G + 3]);
                    GSTOREA(d + t * 1024, v);
                }
            }
        }
        SXT(0);
        u32x4 pv[6];
        unsigned pend = 7u;
        unsigned long long t0 = __builtin_amdgcn_s_memrealtime();
        if (CDELAY) __builtin_amdgcn_s_sleep(CDELAY);
#define CPOLL1 8
#define CPC 0
        if ((aggr || CPC) && wid < CPOLL1) {
            const int p = (q + 1 + (lane & 3)) & 3;
            const unsigned po = (unsigned)(((grp * 4 + p) * 8 + wid) * 16384 + 8192 + ((q - p - 1) & 3) * 2048) + pw;
            while (true) {
                unsigned fv = tag;
                if (lane < 3) fv = __builtin_amdgcn_raw_buffer_load_b32(prs, po, 0, LAUX);
                if (__builtin_amdgcn_ballot_w64(((fv ^ tag) & 1u) != 0u) == 0ull) break;
                if (__builtin_amdgcn_s_memrealtime() - t0 > DEADLINE_TICKS) { bad = true; break; }
                __builtin_amdgcn_s_sleep(FSLEEP);
                asm volatile("" ::: "memory");
            }
        }
        while (true) {
#pragma unroll
            for (int d = 0; d < 3; ++d)
                if (pend & (1u << d)) {
                    const int p = (q + 1 + d) & 3;
                    const unsigned o = (unsigned)(((grp * 4 + p) * 8 + wid) * 16384 + 8192 + ((q - p - 1) & 3) * 2048 + lane * 16);
                    pv[2 * d] = __builtin_amdgcn_raw_buffer_load_b128(prs, o, 0, LAUX);
                    pv[2 * d + 1] = __builtin_amdgcn_raw_buffer_load_b128(prs, o + 1024u, 0, LAUX);
                }
#pragma unroll
            for (int d = 0; d < 3; ++d)
                if (pend & (1u << d)) {
                    if (TAGOK(pv[2 * d], pv[2 * d + 1])) pend &= ~(1u << d);
                }
            if (pend == 0u) break;
            if (__builtin_amdgcn_s_memrealtime() - t0 > DEADLINE_TICKS) { bad = true; break; }
            __builtin_amdgcn_s_sleep(CSLEEP);
            asm volatile("" ::: "memory");
        }
        SXT(1); SXT(2);
#pragma unroll
        for (int t = 0; t < 2; ++t) own[t] = (own[t] + AF(pv[t])) + (AF(pv[2 + t]) + AF(pv[4 + t]));
    }
    if (OWNL) {
        unsigned long long t0 = __builtin_amdgcn_s_memrealtime();
        while (true) {
            unsigned c; LDSRD(c, &sv[8 + wid]);
            if (__builtin_amdgcn_readfirstlane(c) == 2u) break;
            if (__builtin_amdgcn_s_memrealtime() - t0 > DEADLINE_TICKS) { bad = true; break; }
            __builtin_amdgcn_s_sleep(1);
        }
    }
    RST(4);
    finish_unit<PACE1>(own[0], own[1], Ts[wid], dst + (size_t)(64 * RG(wid) + 32 + 8 * q) * NLOC, lane, bad, &sv[4], aggr ? (wid < 4 ? NT1 : NT2) : NTOK);
    asm volatile("s_waitcnt vmcnt(0)" ::: "memory");
    SXT(3);
    __syncthreads();
    RST(5);
    if (tid == 0) __hip_atomic_fetch_add(ws_word(myws, 0), 1u, __ATOMIC_RELAXED, __HIP_MEMORY_SCOPE_AGENT);
    if (tid == 0) {
        if (peer != rank) {
            __builtin_amdgcn_fence(__ATOMIC_RELEASE, "");
            asm volatile("s_waitcnt vmcnt(0)" ::: "memory");
            __hip_atomic_fetch_add(ws_word(ws_all[peer], 2 + rank), 1u, __ATOMIC_RELAXED, __HIP_MEMORY_SCOPE_SYSTEM);
        }
        if (b == 0) {
            unsigned need = call * (unsigned)BLOCKS_PER_PEER;
            unsigned long long t0 = __builtin_amdgcn_s_memrealtime();
            bool timed_out = false;
            if (AGG == 2) __hip_atomic_store((unsigned long long*)ws_word(myws, 12), t0, __ATOMIC_RELAXED, __HIP_MEMORY_SCOPE_AGENT);
            for (int p = 0; p < NR; ++p) {
                if (p == rank) continue;
                while (__hip_atomic_load(ws_word(myws, 2 + p), __ATOMIC_RELAXED, __HIP_MEMORY_SCOPE_SYSTEM) < need) {
                    if (__builtin_amdgcn_s_memrealtime() - t0 > DEADLINE_TICKS) { timed_out = true; break; }
                    __builtin_amdgcn_s_sleep(2);
                }
            }
            __builtin_amdgcn_fence(__ATOMIC_ACQUIRE, "");
            if (timed_out) {
                __hip_atomic_store(ws_word(myws, 1), call, __ATOMIC_RELAXED, __HIP_MEMORY_SCOPE_AGENT);
                ((float*)out_all[rank])[(size_t)(rank * MLOC) * NLOC] = __builtin_nanf("");
            }
        }
    }
}

extern "C" void kernel_launch(int rank, int nranks, ncclComm_t comm,
                              hipStream_t stream,
                              void* const* d_in, const int* in_sizes, int n_in,
                              void* d_out, int out_size,
                              const int* in_dtypes, int out_dtype,
                              void* const* d_out_all,
                              void* const* d_ws_all, size_t ws_bytes) {
    const float* x = (const float*)d_in[0];
    const float* w = (const float*)d_in[1];
    hipLaunchKernelGGL(gemm_a2a_kernel, dim3(256), dim3(NBLOCK), 0, stream,
                       x, w, d_out_all, d_ws_all, rank);
}
```

```cpp
#include <hip/hip_runtime.h>
#include <rccl/rccl.h>
#include <cstdio>
#include <cstdint>

typedef _Float16 half8 __attribute__((ext_vector_type(8)));
typedef _Float16 half4 __attribute__((ext_vector_type(4)));
typedef float f32x16 __attribute__((ext_vector_type(16)));
typedef float f32x4 __attribute__((ext_vector_type(4)));
typedef unsigned u32x4 __attribute__((ext_vector_type(4)));
typedef short s16x4 __attribute__((ext_vector_type(4)));
typedef short s16x8 __attribute__((ext_vector_type(8)));

constexpr int NR = 8;
constexpr int MLOC = 512;
constexpr int KDIM = 4096;
constexpr int NDIM = 4096;
constexpr int NLOC = 512;
constexpr int BN = 64;
constexpr int KS = 1024;
constexpr int NCW = 8;
constexpr int NPW = 4;
constexpr int NBLOCK = (NCW + NPW) * 64;
constexpr int BLOCKS_PER_PEER = 32;
constexpr unsigned long long DEADLINE_TICKS = 40000000ull;
constexpr size_t XFLAG_OFF = 8192;
constexpr size_t FLAG_OFF = 65536;
constexpr size_t RBS = 262144;
constexpr size_t X16_OFF = 262144;
constexpr size_t PART_OFF = X16_OFF + (5u << 20);

__device__ __forceinline__ unsigned* ws_word(void* ws, int idx) { return (unsigned*)((char*)ws + 128 * idx); }

__device__ __forceinline__ unsigned long long rtc() {
    unsigned long long t;
    asm volatile("s_memrealtime %0\n\ts_waitcnt lgkmcnt(0)" : "=s"(t) :: "memory");
    return t;
}

__device__ __forceinline__ float gelu_f(float y) {
    const float c = 0.7978845608028654f;
    float u = c * (y + 0.044715f * y * y * y);
    return y / (1.0f + __expf(-2.0f * u));
}

#define RST(I)
#define XV 0
#define XDELAY 350
#define XDELAYA XDELAY
#define FSLEEP 8
#define CSLEEP 4
#define XSLEEP 16
#define GLOAD(DST, VOFF, SBASE) asm volatile("global_load_dwordx4 %0, %1, %2" : "=v"(DST) : "v"(VOFF), "s"(SBASE))
#define GLOADNT(DST, VOFF, SBASE) asm volatile("global_load_dwordx4 %0, %1, %2 nt" : "=v"(DST) : "v"(VOFF), "s"(SBASE))
#define GLOADC(DST, P) asm volatile("global_load_dwordx4 %0, %1, off sc0 sc1" : "=v"(DST) : "v"(P) : "memory")
#define GSTOREC(P, V) asm volatile("global_store_dwordx4 %0, %1, off sc0 sc1\n\ts_nop 1" :: "v"(P), "v"(V) : "memory")
#define ASC 0
#define GSTOREA(P, V) GSTOREC(P, V)
#define GSTOREX(P, V) GSTOREC(P, V)
#define LAUX ((ASC & 2) ? 16 : 17)
#define LDSRD(DST, P) asm volatile("ds_read_b32 %0, %1\n\ts_waitcnt lgkmcnt(0)" : "=v"(DST) : "v"((unsigned)(uintptr_t)(__attribute__((address_space(3))) unsigned*)(P)) : "memory")

#define AF(V) __builtin_bit_cast(f32x4, V)
#define TAGOK(A, B) (__builtin_amdgcn_ballot_w64((((((A)[0] ^ tag) | ((A)[1] ^ tag)) | (((A)[2] ^ tag) | ((A)[3] ^ tag)) | (((B)[0] ^ tag) | ((B)[1] ^ tag)) | (((B)[2] ^ tag) | ((B)[3] ^ tag))) & 1u) != 0u) == 0ull)
__device__ __forceinline__ float tagged(float f, unsigned tag) { return __uint_as_float((__float_as_uint(f) & ~1u) | tag); }
#define TAGGED(F) tagged(F, tag)
#define FDELAY 0
#define CDELAY 0
#define PF 0
#define CPOLL 1
#define OWNL 1
#define NST (OWNL ? 6 : 8)
#define AGG 2
#define AGT 8000
#define TSL 1
#define XSL 1
#define LSL 2
#define RD 12
#define PACE0 3
#define PACE1 3
#define NTOK 3
#define NT0 3
#define NT0B NT0
#define NT1 3
#define NT2 2
#define GMODE 1
#define PU 0
#define PW 4
template <int PACE>
__device__ __forceinline__ void finish_unit(const f32x4& v0, const f32x4& v1, float* T, float* dst, int lane, bool bad, unsigned* tk, unsigned ntok) {
    const int fh = lane >> 5;
#pragma unroll
    for (int i = 0; i < 4; ++i) {
        float g0 = gelu_f(v0[i]), g1 = gelu_f(v1[i]);
        T[(4 * fh + i) * 68 + (lane & 31)] = bad ? __builtin_nanf("") : g0;
        T[(4 * fh + i) * 68 + 32 + (lane & 31)] = bad ? __builtin_nanf("") : g1;
    }
    __builtin_amdgcn_wave_barrier();
#pragma unroll
    for (int j = 0; j < 2; ++j) {
        const int r = 4 * j + (lane >> 4), c4 = lane & 15;
        f32x4 v = *(const f32x4*)(T + r * 68 + c4 * 4);
        if (PACE == 1) asm volatile("s_waitcnt vmcnt(0)" ::: "memory");
        if (PACE == 2) asm volatile("s_waitcnt vmcnt(1)" ::: "memory");
        if (PACE == 3) {
            unsigned t = 0;
            if (lane == 0) t = __hip_atomic_fetch_add(tk, 1u, __ATOMIC_RELAXED, __HIP_MEMORY_SCOPE_WORKGROUP);
            t = __builtin_amdgcn_readfirstlane(t);
            unsigned long long t0 = __builtin_amdgcn_s_memrealtime();
            while (true) {
                unsigned d; LDSRD(d, tk + 1);
                if (t < __builtin_amdgcn_readfirstlane(d) + ntok) break;
                if (__builtin_amdgcn_s_memrealtime() - t0 > DEADLINE_TICKS) break;
                __builtin_amdgcn_s_sleep(TSL);
            }
        }
        GSTOREC(dst + (size_t)r * NLOC + c4 * 4, v);
        if (PACE == 3) {
            asm volatile("s_waitcnt vmcnt(0)" ::: "memory");
            if (lane == 0) __hip_atomic_fetch_add(tk + 1, 1u, __ATOMIC_RELAXED, __HIP_MEMORY_SCOPE_WORKGROUP);
        }
    }
    __builtin_amdgcn_wave_barrier();
}

__global__ __launch_bounds__(NBLOCK, 3)
void gemm_a2a_kernel(const float* __restrict__ x, const float* __restrict__ w,
                     void* const* __restrict__ out_all, void* const* __restrict__ ws_all, int rank) {
    __shared__ __attribute__((aligned(16))) _Float16 Bs[KS * BN];
    __shared__ __attribute__((aligned(16))) float Ts[NCW + NPW][8 * 68];
    __shared__ __attribute__((aligned(16))) unsigned rdy[4];
    __shared__ __attribute__((aligned(16))) unsigned sv[16];

#define RXT(I)
#define SXT(I)
    RST(0);
    const int tid = threadIdx.x;
    const int lane = tid & 63, wid = __builtin_amdgcn_readfirstlane(tid >> 6);
#define HOIST 1
#define SL 1
    unsigned long long wsp, outp, wpp;
    {
        const int g_ = ((blockIdx.x >> 2) & 1) * 32 + (blockIdx.x >> 3);
        const int pe_ = ((g_ >> 3) + rank) & 7;
        asm volatile("s_load_dwordx2 %0, %3, 0x0\n\ts_load_dwordx2 %1, %4, 0x0\n\ts_load_dwordx2 %2, %5, 0x0" : "=&s"(wsp), "=&s"(outp), "=&s"(wpp) : "s"(ws_all + rank), "s"(out_all + pe_), "s"(ws_all + pe_), "s"(x), "s"(w));
    }
    f32x4 v[4];
    {
        const int b_ = blockIdx.x, q_ = b_ & 3, g_ = ((b_ >> 2) & 1) * 32 + (b_ >> 3);
        const float* src = x + (size_t)(32 * (g_ >> 2) + (lane & 31)) * KDIM + (q_ * 64 + (g_ & 3) * 16 + 2 * (wid & 7)) * 16 + 8 * (lane >> 5);
        if (wid < NCW) {
#pragma unroll
            for (int i = 0; i < 4; ++i)
                asm volatile("global_load_dwordx4 %0, %1, off nt" : "=v"(v[i]) : "v"(src + (i >> 1) * 16 + (i & 1) * 4));
        }
        __builtin_amdgcn_sched_barrier(0);
    }
    char* myws;
    asm volatile("s_waitcnt lgkmcnt(0)" : "+s"(wsp), "+s"(outp), "+s"(wpp));
    myws = (char*)wsp;
    char* x16 = myws + X16_OFF;
    char* part = myws + PART_OFF;
    const __amdgpu_buffer_rsrc_t prs = __builtin_amdgcn_make_buffer_rsrc(part, 0, 32u << 20, 0x00020000);
    const int b = blockIdx.x;
#define MAP 0
    const int q = b & 3;
    const int grp = ((b >> 2) & 1) * 32 + (b >> 3);
    const int peer = ((grp >> 3) + rank) & 7;
    const int ct = grp & 7;
    const int ncol0 = peer * NLOC + ct * BN;
    static_assert(MAP == 0, "table index above");
    float* dst = (float*)outp + (size_t)(rank * MLOC) * NLOC + ct * BN;
    bool bad = false;
#define ROT 0
#define RG(W) (ROT ? (((W) + rank) & 7) : (W))
#define PV 1
    const unsigned pw = PV == 1 ? 1024u + (unsigned)((wid + grp + lane) & 7) * 128u + 124u : PV == 2 ? (unsigned)((wid + grp + lane) & 15) * 128u + 124u : 2044u;

    if (wid >= NCW) {
        const int p = wid - NCW;
        const int kr = lane >> 4, nq = lane & 15;
        const unsigned voff = (unsigned)((4 * p + kr) * NDIM + nq * 4) * 4u;
        const char* base = (const char*)(w + (size_t)q * KS * NDIM + ncol0);
#define PD 2
        f32x4 rb[4 * PD];
#define PLD(SLOT, S) _Pragma("unroll") for (int i = 0; i < 4; ++i) GLOADNT(rb[(SLOT) * 4 + i], voff, base + (size_t)((S) * 64 + 16 * i) * NDIM * 4)
#define PWAIT(N, SLOT) asm volatile("s_waitcnt vmcnt(" #N ")" : "+v"(rb[(SLOT) * 4]), "+v"(rb[(SLOT) * 4 + 1]), "+v"(rb[(SLOT) * 4 + 2]), "+v"(rb[(SLOT) * 4 + 3]))
#define PST(SLOT, S) do { \
        _Pragma("unroll") for (int i = 0; i < 4; ++i) { \
            f32x4 v = rb[(SLOT) * 4 + i]; \
            half4 h; h[0] = (_Float16)v[0]; h[1] = (_Float16)v[1]; h[2] = (_Float16)v[2]; h[3] = (_Float16)v[3]; \
            *(half4*)(Bs + ((S) * 64 + 16 * i + 4 * p + kr) * BN + ((nq * 4) ^ (32 * ((kr >> 1) & 1)))) = h; \
        } \
        asm volatile("" ::: "memory"); \
        if (lane == 0) *(volatile unsigned*)&rdy[p] = (unsigned)(S) + 1u; \
        asm volatile("" ::: "memory"); } while (0)
#define BDELAY 1
#define PCALL 2
        unsigned old = 0;
        unsigned long long last = 0;
        if (p == 0) {
            asm volatile("global_load_dword %0, %1, off sc1" : "=v"(old) : "v"(ws_word(myws, 0)) : "memory");
            asm volatile("global_load_dwordx2 %0, %1, off sc1" : "=v"(last) : "v"(ws_word(myws, 12)) : "memory");
        }
        if (lane == 0) rdy[p] = 0;
        __syncthreads();
        if (p == 0) {
            asm volatile("s_waitcnt vmcnt(0)" : "+v"(old), "+v"(last));
            if (lane == 0) *(volatile unsigned*)&sv[2] = (((old >> 8) + 1u) & 0x3fffffffu) | ((AGG == 1 || (AGG == 2 && __builtin_amdgcn_s_memrealtime() - last > (unsigned long long)AGT)) ? 0x40000000u : 0u);
        }
#define EB 1
#define BTH NCW
        if (!EB) {
            unsigned long long t0 = __builtin_amdgcn_s_memrealtime();
            while (true) {
                unsigned c; LDSRD(c, &sv[0]);
                if (__builtin_amdgcn_readfirstlane(c) >= (unsigned)BTH) break;
                if (__builtin_amdgcn_s_memrealtime() - t0 > DEADLINE_TICKS) break;
                __builtin_amdgcn_s_sleep(1);
            }
        }
#define BX 0
        if (BX > 0) {
            unsigned long long t0 = __builtin_amdgcn_s_memrealtime();
            while (__builtin_amdgcn_s_memrealtime() - t0 < (unsigned long long)BX) __builtin_amdgcn_s_sleep(1);
        }
        if (BX < 0) {
            unsigned long long t0 = __builtin_amdgcn_s_memrealtime();
            while (true) {
                unsigned c; LDSRD(c, &sv[6]);
                if (__builtin_amdgcn_readfirstlane(c) >= (unsigned)(-BX)) break;
                if (__builtin_amdgcn_s_memrealtime() - t0 > DEADLINE_TICKS) break;
                __builtin_amdgcn_s_sleep(1);
            }
        }
        _Pragma("unroll") for (int s = (BDELAY == 2 ? 1 : 0); s < PD; ++s) { PLD(s, s); }
#pragma unroll
        for (int s = 0; s < 16; ++s) {
            const int left = (15 - s < PD - 1) ? 15 - s : PD - 1;
            if (left == 5) PWAIT(20, s % PD); else if (left == 4) PWAIT(16, s % PD); else if (left == 3) PWAIT(12, s % PD); else if (left == 2) PWAIT(8, s % PD); else if (left == 1) PWAIT(4, s % PD); else PWAIT(0, s % PD);
            PST(s % PD, s);
            if ((s & 3) == 0) RXT(4 + (s >> 2));
            if (s + PD < 16) PLD(s % PD, s + PD);
        }
        RST(3);
        unsigned call;
        unsigned long long t0 = __builtin_amdgcn_s_memrealtime();
        while (true) {
            LDSRD(call, &sv[1]);
            if (call != 0u) break;
            if (__builtin_amdgcn_s_memrealtime() - t0 > DEADLINE_TICKS) { call = 0x80000000u; break; }
            __builtin_amdgcn_s_sleep(LSL);
        }
        call = __builtin_amdgcn_readfirstlane(call);
        if (call & 0x80000000u) bad = true;
        const bool aggr = (call & 0x40000000u) != 0u;
        call &= 0x3fffffffu;
        const unsigned tag = call & 1u;
#define FM 3
#define MAPF 1
        if (FM != 4 || !aggr) {
        const int w0 = MAPF ? 2 * p : p;
        constexpr int WS = MAPF ? 1 : 4;
        while (true) {
            unsigned c, c2 = 1;
            if (aggr) { LDSRD(c, &sv[8 + w0]); LDSRD(c2, &sv[8 + w0 + WS]); } else { LDSRD(c, &sv[3]); }
            if (__builtin_amdgcn_readfirstlane(c) >= (aggr ? 1u : (unsigned)NCW) && __builtin_amdgcn_readfirstlane(c2) != 0u) break;
            if (__builtin_amdgcn_s_memrealtime() - t0 > DEADLINE_TICKS) { bad = true; break; }
            __builtin_amdgcn_s_sleep(LSL);
        }
        f32x4 ow[4];
#pragma unroll
        for (int j = 0; j < 4; ++j) ow[j] = *(const f32x4*)(Ts[w0 + (j >> 1) * WS] + (j & 1) * 256 + lane * 4);
        asm volatile("s_waitcnt lgkmcnt(0)" : "+v"(ow[0]), "+v"(ow[1]), "+v"(ow[2]), "+v"(ow[3]) :: "memory");
        if (lane < 2) *(volatile unsigned*)&sv[8 + w0 + lane * WS] = 2u;
        const unsigned voff0 = (unsigned)((grp * 4 * 8 + w0) * 16384 + q * 2048 + lane * 16);
        u32x4 pv[12];
        unsigned pend = 0x3fu;
#define FCP0 0
        if (!(aggr && p < FCP0)) {
            const unsigned po = (unsigned)((grp * 4 * 8 + w0) * 16384 + q * 2048) + pw + (unsigned)((((q + 1 + (lane & 3)) & 3) * 8 + ((lane >> 2) & 1) * WS) * 16384);
            while (true) {
                unsigned fv = tag;
                if (lane < 8 && (lane & 3) != 3) fv = __builtin_amdgcn_raw_buffer_load_b32(prs, po, 0, LAUX);
                if (__builtin_amdgcn_ballot_w64(((fv ^ tag) & 1u) != 0u) == 0ull) break;
                if (__builtin_amdgcn_s_memrealtime() - t0 > DEADLINE_TICKS) { bad = true; break; }
                __builtin_amdgcn_s_sleep(FSLEEP);
                asm volatile("" ::: "memory");
            }
        }
        while (true) {
#pragma unroll
            for (int i = 0; i < 6; ++i)
                if (pend & (1u << i)) {
                    const unsigned o = voff0 + (unsigned)((((q + 1 + i % 3) & 3) * 8 + (i / 3) * WS) * 16384);
                    pv[2 * i] = __builtin_amdgcn_raw_buffer_load_b128(prs, o, 0, LAUX);
                    pv[2 * i + 1] = __builtin_amdgcn_raw_buffer_load_b128(prs, o + 1024u, 0, LAUX);
                }
#pragma unroll
            for (int i = 0; i < 6; ++i)
                if (pend & (1u << i)) {
                    if (TAGOK(pv[2 * i], pv[2 * i + 1])) pend &= ~(1u << i);
                }
            if (pend == 0u) break;
            if (__builtin_amdgcn_s_memrealtime() - t0 > DEADLINE_TICKS) { bad = true; break; }
            __builtin_amdgcn_s_sleep(FSLEEP);
            asm volatile("" ::: "memory");
        }
        if (lane == 0) __hip_atomic_fetch_add(&sv[7], 1u, __ATOMIC_RELAXED, __HIP_MEMORY_SCOPE_WORKGROUP);
        {
            f32x4 v0 = (ow[0] + AF(pv[0])) + (AF(pv[2]) + AF(pv[4])), v1 = (ow[1] + AF(pv[1])) + (AF(pv[3]) + AF(pv[5]));
            finish_unit<PACE0>(v0, v1, Ts[wid], dst + (size_t)(64 * RG(w0) + 8 * q) * NLOC, lane, bad, &sv[4], aggr ? (p < 2 ? NT0 : NT0B) : NTOK);
        }
        {
            f32x4 v0 = (ow[2] + AF(pv[6])) + (AF(pv[8]) + AF(pv[10])), v1 = (ow[3] + AF(pv[7])) + (AF(pv[9]) + AF(pv[11]));
            finish_unit<PACE0>(v0, v1, Ts[wid], dst + (size_t)(64 * RG(w0 + WS) + 8 * q) * NLOC, lane, bad, &sv[4], aggr ? (p < 2 ? NT0 : NT0B) : NTOK);
        }
        }
#pragma unroll 1
        for (int u = 0; u < ((FM == 1 || FM == 3 || (FM == 4 && !aggr)) ? 0 : 2); ++u) {
            const int wv = p + 4 * u;
            while (true) {
                unsigned c; LDSRD(c, &sv[8 + wv]);
                if (__builtin_amdgcn_readfirstlane(c) != 0u) break;
                if (__builtin_amdgcn_s_memrealtime() - t0 > DEADLINE_TICKS) { bad = true; break; }
                __builtin_amdgcn_s_sleep(LSL);
            }
            const unsigned voff0 = (unsigned)((grp * 4 * 8 + wv) * 16384 + q * 2048 + lane * 16);
            u32x4 pv[8];
            unsigned pend = 0xfu;
            {
                const unsigned po = (unsigned)((grp * 4 * 8 + wv) * 16384 + q * 2048) + pw + (unsigned)((lane & 3) * 8 * 16384);
                while (true) {
                    unsigned fv = tag;
                    if (lane < 4) fv = __builtin_amdgcn_raw_buffer_load_b32(prs, po, 0, LAUX);
                    if (__builtin_amdgcn_ballot_w64(((fv ^ tag) & 1u) != 0u) == 0ull) break;
                    if (__builtin_amdgcn_s_memrealtime() - t0 > DEADLINE_TICKS) { bad = true; break; }
                    __builtin_amdgcn_s_sleep(FSLEEP);
                    asm volatile("" ::: "memory");
                }
            }
            if (u == 0) SXT(4);
            if (GMODE == 3 && u <= PU && lane == 0) __hip_atomic_fetch_add(&sv[6], 1u, __ATOMIC_RELAXED, __HIP_MEMORY_SCOPE_WORKGROUP);
            while (true) {
#pragma unroll
                for (int i = 0; i < 4; ++i)
                    if (pend & (1u << i)) {
                        const unsigned o = voff0 + (unsigned)(i * 8 * 16384);
                        pv[2 * i] = __builtin_amdgcn_raw_buffer_load_b128(prs, o, 0, LAUX);
                        pv[2 * i + 1] = __builtin_amdgcn_raw_buffer_load_b128(prs, o + 1024u, 0, LAUX);
                    }
#pragma unroll
                for (int i = 0; i < 4; ++i)
                    if (pend & (1u << i)) {
                        if (TAGOK(pv[2 * i], pv[2 * i + 1])) pend &= ~(1u << i);
                    }
                if (pend == 0u) break;
                if (__builtin_amdgcn_s_memrealtime() - t0 > DEADLINE_TICKS) { bad = true; break; }
                __builtin_amdgcn_s_sleep(FSLEEP);
                asm volatile("" ::: "memory");
            }
            if (u == 0) SXT(5); else SXT(7);
            if (u == 0 && lane == 0) __hip_atomic_fetch_add(&sv[7], 1u, __ATOMIC_RELAXED, __HIP_MEMORY_SCOPE_WORKGROUP);
            if (GMODE == 3 && u <= PU && lane == 0) __hip_atomic_fetch_add(&sv[6], ~0u, __ATOMIC_RELAXED, __HIP_MEMORY_SCOPE_WORKGROUP);
            f32x4 v0 = (AF(pv[0]) + AF(pv[2])) + (AF(pv[4]) + AF(pv[6])), v1 = (AF(pv[1]) + AF(pv[3])) + (AF(pv[5]) + AF(pv[7]));
            finish_unit<PACE0>(v0, v1, Ts[wid], dst + (size_t)(64 * RG(wv) + 8 * q) * NLOC, lane, bad, &sv[4], aggr ? (p < 2 ? NT0 : NT0B) : NTOK);
            if (u == 0) SXT(6);
        }
        asm volatile("s_waitcnt vmcnt(0)" ::: "memory");
        __syncthreads();
        return;
    }

    unsigned call;
    bool aggr;
    {
        unsigned old = 0;
        if (tid < 16) sv[tid] = 0;
        const int rbk = grp >> 2, ks = q * 64 + (grp & 3) * 16 + 2 * wid;
        static_assert(!EB || PCALL >= 1, "EB needs the call number from the producer wave");
        if (!EB) __syncthreads();
        unsigned long long last = 0;
        if (!PCALL && tid == 0) {
            old = __hip_atomic_load(ws_word(myws, 0), __ATOMIC_RELAXED, __HIP_MEMORY_SCOPE_AGENT);
            if (AGG == 2) last = __hip_atomic_load((unsigned long long*)ws_word(myws, 12), __ATOMIC_RELAXED, __HIP_MEMORY_SCOPE_AGENT);
        }
        RXT(0);
        asm volatile("s_waitcnt vmcnt(0)" : "+v"(v[0]), "+v"(v[1]), "+v"(v[2]), "+v"(v[3]));
        RXT(1);
        if (EB == 2) __syncthreads();
        if (!EB && lane == 0) __hip_atomic_fetch_add(&sv[0], 1u, __ATOMIC_RELAXED, __HIP_MEMORY_SCOPE_WORKGROUP);
        if (!PCALL && tid == 0) *(volatile unsigned*)&sv[2] = (((old >> 8) + 1u) & 0x3fffffffu) | ((AGG == 1 || (AGG == 2 && __builtin_amdgcn_s_memrealtime() - last > (unsigned long long)AGT)) ? 0x40000000u : 0u);
#pragma unroll
        for (int i = 0; i < 2; ++i) {
            half8 h;
#pragma unroll
            for (int e = 0; e < 4; ++e) { h[e] = (_Float16)v[2 * i][e]; h[4 + e] = (_Float16)v[2 * i + 1][e]; }
            GSTOREX(x16 + (size_t)rbk * RBS + (size_t)((ks + i) * 64 + lane) * 16, __builtin_bit_cast(f32x4, h));
        }
        if (EB == 1) __syncthreads();
        unsigned long long t0 = __builtin_amdgcn_s_memrealtime();
        while (true) {
            LDSRD(call, &sv[2]);
            if (call != 0u) break;
            if (__builtin_amdgcn_s_memrealtime() - t0 > DEADLINE_TICKS) { bad = true; break; }
            __builtin_amdgcn_s_sleep(1);
        }
        call = __builtin_amdgcn_readfirstlane(call);
        aggr = (call & 0x40000000u) != 0u;
        call &= 0x3fffffffu;
        RXT(2);
        asm volatile("s_waitcnt vmcnt(0)" ::: "memory");
        RXT(3);
#define XF 1
#define XRS 4352
        {
            unsigned c = 0;
            if (lane == 0) c = __hip_atomic_fetch_add(&sv[6], 1u, __ATOMIC_RELAXED, __HIP_MEMORY_SCOPE_WORKGROUP);
            c = __builtin_amdgcn_readfirstlane(c);
            if (c == (unsigned)NCW - 1u && lane < 8)
                __hip_atomic_store((unsigned*)(myws + FLAG_OFF + (size_t)(q * 8 + lane) * XRS) + grp, call, __ATOMIC_RELAXED, __HIP_MEMORY_SCOPE_SYSTEM);
        }
        if (wid == 0) {
            const unsigned* xf = (const unsigned*)(myws + FLAG_OFF + (size_t)(q * 8 + (grp & 7)) * XRS) + lane;
            while (__builtin_amdgcn_s_memrealtime() - t0 < (aggr ? XDELAYA : XDELAY)) __builtin_amdgcn_s_sleep(4);
            while (true) {
                unsigned f = __hip_atomic_load(xf, __ATOMIC_RELAXED, __HIP_MEMORY_SCOPE_SYSTEM);
                if (__builtin_amdgcn_ballot_w64(f != call) == 0ull) break;
                if (__builtin_amdgcn_s_memrealtime() - t0 > DEADLINE_TICKS) { bad = true; break; }
                __builtin_amdgcn_s_sleep(XSLEEP);
            }
            asm volatile("" ::: "memory");
            if (lane == 0) *(volatile unsigned*)&sv[1] = call | (bad ? 0x80000000u : 0u) | (aggr ? 0x40000000u : 0u);
        } else {
            while (true) {
                LDSRD(call, &sv[1]);
                if (call != 0u) break;
                if (__builtin_amdgcn_s_memrealtime() - t0 > DEADLINE_TICKS) { call = 0x80000000u; break; }
                __builtin_amdgcn_s_sleep(XSL);
            }
            call = __builtin_amdgcn_readfirstlane(call);
            if (call & 0x80000000u) bad = true;
            call &= 0x3fffffffu;
        }
        asm volatile("s_waitcnt vmcnt(0) lgkmcnt(0)" ::: "memory");
    }
    RST(1);

    const unsigned a_voff = (unsigned)lane * 16u;
    const char* a_base = x16 + (size_t)(2 * RG(wid)) * RBS + (size_t)(64 * q) * 1024;
#define PFX 0
    if (PFX && wid >= 4) {
        f32x4 pf[6];
        const int c0 = ((b >> 3) * 4 + (wid - 4)) * 6;
#pragma unroll
        for (int i = 0; i < 6; ++i) {
            const int c = c0 + i, rbi = c >> 6, rbk = rbi < 4 ? 2 * rbi + 1 : rbi + 4;
            GLOAD(pf[i], a_voff, x16 + (size_t)rbk * RBS + (size_t)(64 * q + (c & 63)) * 1024);
        }
        asm volatile("s_waitcnt vmcnt(0)" : "+v"(pf[0]), "+v"(pf[1]), "+v"(pf[2]), "+v"(pf[3]), "+v"(pf[4]), "+v"(pf[5]));
    }
    f32x4 ra[RD];
#define AOFF(S) ((size_t)((S) >> 6) * RBS + (size_t)((S) & 63) * 1024)
#pragma unroll
    for (int i = 0; i < RD; ++i) GLOAD(ra[i], a_voff, a_base + AOFF(i));

    f32x16 acc[2];
#pragma unroll
    for (int t = 0; t < 2; ++t)
#pragma unroll
        for (int i = 0; i < 16; ++i) acc[t][i] = 0.f;

    const int fh = lane >> 5;
    const int bblk = (lane >> 4) & 1, bq = (lane & 15) >> 2, bp = lane & 3;
    const unsigned lds0 = (unsigned)(uintptr_t)(__attribute__((address_space(3))) _Float16*)Bs;
    const unsigned b0p = lds0 + 2u * ((8 * fh + bq) * BN + ((16 * bblk + 4 * bp) ^ (32 * ((bq >> 1) & 1))));
    const unsigned b1p = lds0 + 2u * ((8 * fh + bq) * BN + ((32 + 16 * bblk + 4 * bp) ^ (32 * ((bq >> 1) & 1))));
#define BBASE(BLK) unsigned bs0 = b0p + (BLK) * 512u * BN, bs1 = b1p + (BLK) * 512u * BN; asm volatile("" : "+v"(bs0), "+v"(bs1))
    unsigned seen = 0;
#define BPOLL() do { \
        u32x4 rv; \
        asm volatile("ds_read_b128 %0, %1\n\ts_waitcnt lgkmcnt(0)" : "=v"(rv) : "v"((unsigned)(uintptr_t)(__attribute__((address_space(3))) unsigned*)rdy) : "memory"); \
        seen = __builtin_amdgcn_readfirstlane(min(min(rv[0], rv[1]), min(rv[2], rv[3]))); } while (0)
#define BWAIT(S) do { \
        if (seen <= (unsigned)(S)) { \
            BPOLL(); \
            if (seen <= (unsigned)(S)) { \
                unsigned long long t0 = rtc(); \
                for (unsigned n = 1; ; ++n) { \
                    __builtin_amdgcn_s_sleep(2); \
                    BPOLL(); \
                    if (seen > (unsigned)(S)) break; \
                    if ((n & 255u) == 0u && rtc() - t0 > DEADLINE_TICKS) { bad = true; seen = 64; break; } \
                } \
            } \
        } } while (0)
#define TR(P) __builtin_amdgcn_ds_read_tr16_b64_v4i16((__attribute__((address_space(3))) s16x4*)(uintptr_t)(P))
#define LD 2
    s16x4 fb[LD + 1][4];
    unsigned bsl0 = b0p, bsl1 = b1p, bsh0 = b0p + 65536u, bsh1 = b1p + 65536u;
    asm volatile("" : "+v"(bsl0), "+v"(bsl1), "+v"(bsh0), "+v"(bsh1));
#define TRA(DST, P, O) asm volatile("ds_read_b64_tr_b16 %0, %1 offset:%2" : "=v"(DST) : "v"(P), "n"(O))
#define BRD(G) do { \
        if ((G) & 32) { \
            TRA(fb[(G) % (LD + 1)][0], bsh0, ((G) & 31) * 2048); TRA(fb[(G) % (LD + 1)][1], bsh0, ((G) & 31) * 2048 + 512); \
            TRA(fb[(G) % (LD + 1)][2], bsh1, ((G) & 31) * 2048); TRA(fb[(G) % (LD + 1)][3], bsh1, ((G) & 31) * 2048 + 512); \
        } else { \
            TRA(fb[(G) % (LD + 1)][0], bsl0, ((G) & 31) * 2048); TRA(fb[(G) % (LD + 1)][1], bsl0, ((G) & 31) * 2048 + 512); \
            TRA(fb[(G) % (LD + 1)][2], bsl1, ((G) & 31) * 2048); TRA(fb[(G) % (LD + 1)][3], bsl1, ((G) & 31) * 2048 + 512); \
        } } while (0)
#define STEP(G) do { \
        if ((G) + LD < 64 && (((G) + LD) & 3) == 0) BWAIT(((G) + LD) >> 2); \
        if ((G) + LD < 128) BRD((G) + LD); \
        asm volatile("s_waitcnt vmcnt(%5) lgkmcnt(%6)" : "+v"(ra[(G) % RD]), "+v"(fb[(G) % (LD + 1)][0]), "+v"(fb[(G) % (LD + 1)][1]), "+v"(fb[(G) % (LD + 1)][2]), "+v"(fb[(G) % (LD + 1)][3]) \
            : "n"(((G) > 127 - RD ? 127 - (G) : RD - 1) + ((G) >= 64 && (G) < 64 + RD ? NST : 0)), "n"(4 * ((G) + LD < 128 ? LD : 127 - (G)))); \
        half8 fb0 = __builtin_bit_cast(half8, __builtin_shufflevector(fb[(G) % (LD + 1)][0], fb[(G) % (LD + 1)][1], 0, 1, 2, 3, 4, 5, 6, 7)); \
        half8 fb1 = __builtin_bit_cast(half8, __builtin_shufflevector(fb[(G) % (LD + 1)][2], fb[(G) % (LD + 1)][3], 0, 1, 2, 3, 4, 5, 6, 7)); \
        half8 fa = __builtin_bit_cast(half8, ra[(G) % RD]); \
        acc[0] = __builtin_amdgcn_mfma_f32_32x32x16_f16(fa, fb0, acc[0], 0, 0, 0); \
        acc[1] = __builtin_amdgcn_mfma_f32_32x32x16_f16(fa, fb1, acc[1], 0, 0, 0); \
        if ((G) + RD < 128) GLOAD(ra[(G) % RD], a_voff, a_base + AOFF((G) + RD)); \
        __builtin_amdgcn_sched_barrier(0); \
    } while (0)
#define PAUSE() do { if (GMODE == 3 && wid >= PW) { \
        for (unsigned n = 0; n < 4096u; ++n) { \
            unsigned c; LDSRD(c, &sv[6]); \
            if (__builtin_amdgcn_readfirstlane(c) == 0u) break; \
            __builtin_amdgcn_s_sleep(1); \
        } } } while (0)
#define STEPV 0
#define STEPO(G, J) do { \
        s16x4 l0 = TR(bs0 + (J) * 32 * BN), h0 = TR(bs0 + ((J) * 32 + 8) * BN); \
        s16x4 l1 = TR(bs1 + (J) * 32 * BN), h1 = TR(bs1 + ((J) * 32 + 8) * BN); \
        half8 fb0 = __builtin_bit_cast(half8, __builtin_shufflevector(l0, h0, 0, 1, 2, 3, 4, 5, 6, 7)); \
        half8 fb1 = __builtin_bit_cast(half8, __builtin_shufflevector(l1, h1, 0, 1, 2, 3, 4, 5, 6, 7)); \
        asm volatile("s_waitcnt vmcnt(%1)" : "+v"(ra[(G) % RD]) : "n"(((G) > 127 - RD ? 127 - (G) : RD - 1) + ((G) >= 64 && (G) < 64 + RD ? NST : 0))); \
        half8 fa = __builtin_bit_cast(half8, ra[(G) % RD]); \
        acc[0] = __builtin_amdgcn_mfma_f32_32x32x16_f16(fa, fb0, acc[0], 0, 0, 0); \
        acc[1] = __builtin_amdgcn_mfma_f32_32x32x16_f16(fa, fb1, acc[1], 0, 0, 0); \
        if ((G) + RD < 128) GLOAD(ra[(G) % RD], a_voff, a_base + AOFF((G) + RD)); \
    } while (0)
#define SB() __builtin_amdgcn_sched_barrier(0)
#define PS2 16
#define PS1 0
#define PS2A 0
#define PS2B 0
#define PZ() do { if (aggr) { if (wid < 4) { if (PS2A) __builtin_amdgcn_s_sleep(PS2A); } else { if (PS2B) __builtin_amdgcn_s_sleep(PS2B); } } else { if (PS2) __builtin_amdgcn_s_sleep(PS2); } } while (0)
#define PY() do { if (PS1 && !aggr) __builtin_amdgcn_s_sleep(PS1); } while (0)
#define STEP4(G0, J0) do { STEPO((G0), (J0)); STEPO((G0) + 1, (J0) + 1); STEPO((G0) + 2, (J0) + 2); STEPO((G0) + 3, (J0) + 3); SB(); } while (0)
#define BLOCK1(B) do { BBASE(B); \
        BWAIT(4 * (B));     PY(); STEP4(16 * (B), 0); \
        BWAIT(4 * (B) + 1); PY(); STEP4(16 * (B) + 4, 4); \
        BWAIT(4 * (B) + 2); PY(); STEP4(16 * (B) + 8, 8); \
        BWAIT(4 * (B) + 3); PY(); STEP4(16 * (B) + 12, 12); } while (0)
#define BLOCK2(B) do { BBASE(B); \
        PZ(); STEP4(64 + 16 * (B), 0); PZ(); STEP4(64 + 16 * (B) + 4, 4); PZ(); STEP4(64 + 16 * (B) + 8, 8); PZ(); STEP4(64 + 16 * (B) + 12, 12); } while (0)

#define SK 16
#define SK2 1
    if (aggr && wid >= 2 && wid < 4) BWAIT(SK2 - 1);
    if (LSL > 2 && aggr && wid >= 4) {
        unsigned long long t0 = rtc();
        while (true) {
            BPOLL();
            if (seen > (unsigned)(SK - 1)) break;
            if (rtc() - t0 > DEADLINE_TICKS) { bad = true; seen = 64; break; }
            __builtin_amdgcn_s_sleep(LSL);
        }
    }
    if (aggr && wid >= 4) BWAIT(SK - 1);
#define WG 2
#define WGW 4
#define GATE() do { \
        unsigned long long t0 = __builtin_amdgcn_s_memrealtime(); \
        while (true) { \
            unsigned c; LDSRD(c, &sv[7]); \
            if (__builtin_amdgcn_readfirstlane(c) >= (unsigned)WG) break; \
            if (__builtin_amdgcn_s_memrealtime() - t0 > DEADLINE_TICKS) { bad = true; break; } \
            __builtin_amdgcn_s_sleep(LSL); \
        } } while (0)
    if (GMODE == 1 && WG && aggr && wid >= WGW) GATE();
    BWAIT(0);
    BLOCK1(0); BLOCK1(1); BLOCK1(2); BLOCK1(3);
    RST(2);
    char* mypart = part + (size_t)((grp * 4 + q) * 8 + wid) * 16384 + lane * 16;
    const unsigned tag = call & 1u;
#pragma unroll
    for (int G = 0; G < 4; ++G) {
        if (OWNL && G == q) {
#pragma unroll
            for (int t = 0; t < 2; ++t) {
                f32x4 v; v[0] = acc[t][4 * G]; v[1] = acc[t][4 * G + 1]; v[2] = acc[t][4 * G + 2]; v[3] = acc[t][4 * G + 3];
                *(f32x4*)(Ts[wid] + t * 256 + lane * 4) = v;
            }
        } else {
#pragma unroll
            for (int t = 0; t < 2; ++t) {
                f32x4 v; v[0] = TAGGED(acc[t][4 * G]); v[1] = TAGGED(acc[t][4 * G + 1]); v[2] = TAGGED(acc[t][4 * G + 2]); v[3] = TAGGED(acc[t][4 * G + 3]);
                GSTOREA(mypart + G * 2048 + t * 1024, v);
            }
        }
    }
    asm volatile("s_waitcnt lgkmcnt(0)" ::: "memory");
    if (lane == 0) *(volatile unsigned*)&sv[8 + wid] = 1u;
    if (FM && lane == 0) __hip_atomic_fetch_add(&sv[3], 1u, __ATOMIC_RELAXED, __HIP_MEMORY_SCOPE_WORKGROUP);
#pragma unroll
    for (int t = 0; t < 2; ++t)
#pragma unroll
        for (int i = 0; i < 16; ++i) acc[t][i] = 0.f;

    if (GMODE == 2 && WG && wid < 4) GATE();
    BLOCK2(0); BLOCK2(1); BLOCK2(2); BLOCK2(3);
    RST(3);

    f32x4 own[2];
    {
#pragma unroll
        for (int G = 0; G < 4; ++G) {
            if (G == q) {
#pragma unroll
                for (int t = 0; t < 2; ++t) { own[t][0] = acc[t][4 * G]; own[t][1] = acc[t][4 * G + 1]; own[t][2] = acc[t][4 * G + 2]; own[t][3] = acc[t][4 * G + 3]; }
            } else {
                char* d = mypart + 8192 + ((G - q - 1) & 3) * 2048;
#pragma unroll
                for (int t = 0; t < 2; ++t) {
                    f32x4 v; v[0] = TAGGED(acc[t][4 * G]); v[1] = TAGGED(acc[t][4 * G + 1]); v[2] = TAGGED(acc[t][4 * G + 2]); v[3] = TAGGED(acc[t][4 * G + 3]);
                    GSTOREA(d + t * 1024, v);
                }
            }
        }
        SXT(0);
        u32x4 pv[6];
        unsigned pend = 7u;
        unsigned long long t0 = __builtin_amdgcn_s_memrealtime();
        if (CDELAY) __builtin_amdgcn_s_sleep(CDELAY);
#define CPOLL1 8
#define CPC 0
        if ((aggr || CPC) && wid < CPOLL1) {
            const int p = (q + 1 + (lane & 3)) & 3;
            const unsigned po = (unsigned)(((grp * 4 + p) * 8 + wid) * 16384 + 8192 + ((q - p - 1) & 3) * 2048) + pw;
            while (true) {
                unsigned fv = tag;
                if (lane < 3) fv = __builtin_amdgcn_raw_buffer_load_b32(prs, po, 0, LAUX);
                if (__builtin_amdgcn_ballot_w64(((fv ^ tag) & 1u) != 0u) == 0ull) break;
                if (__builtin_amdgcn_s_memrealtime() - t0 > DEADLINE_TICKS) { bad = true; break; }
                __builtin_amdgcn_s_sleep(FSLEEP);
                asm volatile("" ::: "memory");
            }
        }
        while (true) {
#pragma unroll
            for (int d = 0; d < 3; ++d)
                if (pend & (1u << d)) {
                    const int p = (q + 1 + d) & 3;
                    const unsigned o = (unsigned)(((grp * 4 + p) * 8 + wid) * 16384 + 8192 + ((q - p - 1) & 3) * 2048 + lane * 16);
                    pv[2 * d] = __builtin_amdgcn_raw_buffer_load_b128(prs, o, 0, LAUX);
                    pv[2 * d + 1] = __builtin_amdgcn_raw_buffer_load_b128(prs, o + 1024u, 0, LAUX);
                }
#pragma unroll
            for (int d = 0; d < 3; ++d)
                if (pend & (1u << d)) {
                    if (TAGOK(pv[2 * d], pv[2 * d + 1])) pend &= ~(1u << d);
                }
            if (pend == 0u) break;
            if (__builtin_amdgcn_s_memrealtime() - t0 > DEADLINE_TICKS) { bad = true; break; }
            __builtin_amdgcn_s_sleep(CSLEEP);
            asm volatile("" ::: "memory");
        }
        SXT(1); SXT(2);
#pragma unroll
        for (int t = 0; t < 2; ++t) own[t] = (own[t] + AF(pv[t])) + (AF(pv[2 + t]) + AF(pv[4 + t]));
    }
    if (OWNL) {
        unsigned long long t0 = __builtin_amdgcn_s_memrealtime();
        while (true) {
            unsigned c; LDSRD(c, &sv[8 + wid]);
            if (__builtin_amdgcn_readfirstlane(c) == 2u) break;
            if (__builtin_amdgcn_s_memrealtime() - t0 > DEADLINE_TICKS) { bad = true; break; }
            __builtin_amdgcn_s_sleep(1);
        }
    }
    RST(4);
    finish_unit<PACE1>(own[0], own[1], Ts[wid], dst + (size_t)(64 * RG(wid) + 32 + 8 * q) * NLOC, lane, bad, &sv[4], aggr ? (wid < 4 ? NT1 : NT2) : NTOK);
    asm volatile("s_waitcnt vmcnt(0)" ::: "memory");
    SXT(3);
    __syncthreads();
    RST(5);
#define TL 1
    if (wid == 0) {
        if (tid == 0) {
            if (peer != rank) {
                __builtin_amdgcn_fence(__ATOMIC_RELEASE, "");
                asm volatile("s_waitcnt vmcnt(0)" ::: "memory");
                __hip_atomic_fetch_add(ws_word((void*)wpp, 2 + rank), 1u, __ATOMIC_RELAXED, __HIP_MEMORY_SCOPE_SYSTEM);
            }
            __hip_atomic_fetch_add(ws_word(myws, 0), 1u, __ATOMIC_RELAXED, __HIP_MEMORY_SCOPE_AGENT);
        }
        if (b == 0) {
            const unsigned need = call * (unsigned)BLOCKS_PER_PEER;
            unsigned long long t0 = __builtin_amdgcn_s_memrealtime();
            bool timed_out = false;
            if (AGG == 2 && tid == 0) __hip_atomic_store((unsigned long long*)ws_word(myws, 12), t0, __ATOMIC_RELAXED, __HIP_MEMORY_SCOPE_AGENT);
            while (true) {
                unsigned f = need;
                if (lane < NR && lane != rank) f = __hip_atomic_load(ws_word(myws, 2 + lane), __ATOMIC_RELAXED, __HIP_MEMORY_SCOPE_SYSTEM);
                if (__builtin_amdgcn_ballot_w64(f < need) == 0ull) break;
                if (__builtin_amdgcn_s_memrealtime() - t0 > DEADLINE_TICKS) { timed_out = true; break; }
                __builtin_amdgcn_s_sleep(2);
            }
            __builtin_amdgcn_fence(__ATOMIC_ACQUIRE, "");
            if (timed_out && tid == 0) {
                __hip_atomic_store(ws_word(myws, 1), call, __ATOMIC_RELAXED, __HIP_MEMORY_SCOPE_AGENT);
                ((float*)out_all[rank])[(size_t)(rank * MLOC) * NLOC] = __builtin_nanf("");
            }
        }
    }
}

extern "C" void kernel_launch(int rank, int nranks, ncclComm_t comm,
                              hipStream_t stream,
                              void* const* d_in, const int* in_sizes, int n_in,
                              void* d_out, int out_size,
                              const int* in_dtypes, int out_dtype,
                              void* const* d_out_all,
                              void* const* d_ws_all, size_t ws_bytes) {
    const float* x = (const float*)d_in[0];
    const float* w = (const float*)d_in[1];
    hipLaunchKernelGGL(gemm_a2a_kernel, dim3(256), dim3(NBLOCK), 0, stream,
                       x, w, d_out_all, d_ws_all, rank);
}
```
